```python
import numpy as np

kernel_name = "gpu_gemm_ar_m4096_k4096_n4096_bf16_relu"


ABSMAX_RTOL = 0.008

COMPUTE_DTYPE = "bfloat16"
EPILOGUE = "relu"
IN_QUANT = None
OUT_QUANT = None
IN_DTYPES = {"x": "bfloat16", "w_mat": "bfloat16"}
WORLD_SIZE = 8

def _to_bf16(x):
    u = np.ascontiguousarray(x, dtype=np.float32).view(np.uint32)
    return ((u + 0x7FFF + ((u >> 16) & 1)) & 0xFFFF0000).view(np.float32)

def setup_inputs():

    x = _to_bf16(np.random.randn(4096, 4096).astype(np.float32))
    w_mat = _to_bf16((np.random.randn(4096, 4096) / np.sqrt(4096)).astype(np.float32))
    return {"x": x, "w_mat": w_mat}

def _epilogue(y):
    return np.maximum(y, 0.0).astype(np.float32)

def reference(x, w_mat):
    y = (x @ w_mat).astype(np.float32)
    return _epilogue(y)
```

```cpp
#include <hip/hip_runtime.h>
#include <rccl/rccl.h>
#include <cstdio>
#include <cstdint>

#define HIP_CHECK(expr) do { hipError_t _e = (expr); if (_e != hipSuccess) { \
    fprintf(stderr, "HIP error %s at %s:%d\n", hipGetErrorString(_e), __FILE__, __LINE__); \
    return; } } while (0)

namespace {

typedef short bf16x8 __attribute__((ext_vector_type(8)));
typedef float f32x4 __attribute__((ext_vector_type(4)));
typedef unsigned u32x4 __attribute__((ext_vector_type(4)));
typedef unsigned u32x2 __attribute__((ext_vector_type(2)));
typedef unsigned long long u64;

constexpr int NR = 8;
constexpr int DIM = 4096;
constexpr int KSL = DIM / NR;
constexpr int RT = 16, CT = 8;
constexpr int NKS = 64;
constexpr unsigned KSB = 16384;
constexpr unsigned SEG = 9216;

constexpr unsigned XS_OFF = 0;
constexpr unsigned WS_OFF = XS_OFF + RT * NKS * KSB;
constexpr unsigned SX_OFF = WS_OFF + CT * NKS * KSB;
constexpr unsigned SW_OFF = SX_OFF + RT * 8 * KSB;
constexpr unsigned OB_OFF = SW_OFF + 32 * 8 * KSB;
constexpr unsigned IF_OFF = OB_OFF + RT * CT * 4 * SEG;
constexpr unsigned PF_OFF = IF_OFF + 24 * 256;
constexpr unsigned OF_OFF = PF_OFF + NR * 192 * 4;
constexpr unsigned TK_OFF = OF_OFF + NR * RT * CT * 64;
constexpr unsigned TMO_OFF = TK_OFF + 32 * 128;
constexpr unsigned DBG_OFF = TMO_OFF + 4096;
constexpr unsigned WS_USED = DBG_OFF + 256 * 4 * 16 * 8;

#define NIN 16
#define DEPTH 4
#define NOUT 48

constexpr u64 DEADLINE_TICKS = 50000000ull;

constexpr int LDS_STAGE = 32768;
constexpr int LDS_CTRL = 4 * LDS_STAGE;
constexpr int LDS_BYTES = LDS_CTRL + 256;

struct Params {
  const unsigned short* x;
  const unsigned short* w;
  unsigned short* out;
  char* ws[NR];
  int rank;
};

#define LDS_AS __attribute__((address_space(3)))
#define GLB_AS __attribute__((address_space(1)))

__device__ __forceinline__ __amdgpu_buffer_rsrc_t make_rsrc(const void* p, unsigned bytes) {
  return __builtin_amdgcn_make_buffer_rsrc((void*)p, 0, bytes, 0x00020000);
}
constexpr int AUX_SYS = 17;

#define STAMP(i) do { } while (0)

__device__ __forceinline__ bool timed_out(unsigned* tmo, unsigned call, u64 t0) {
  if (__hip_atomic_load(tmo, __ATOMIC_RELAXED, __HIP_MEMORY_SCOPE_AGENT) == call ||
      __builtin_amdgcn_s_memrealtime() - t0 > DEADLINE_TICKS) {
    __hip_atomic_store(tmo, call, __ATOMIC_RELAXED, __HIP_MEMORY_SCOPE_AGENT);
    return true;
  }
  return false;
}

__device__ __forceinline__ unsigned pack_bf16(float a, float b) {
  typedef __bf16 bf2 __attribute__((ext_vector_type(2)));
  typedef float f2 __attribute__((ext_vector_type(2)));
  f2 v = {a, b};
  bf2 r = __builtin_convertvector(v, bf2);
  return __builtin_bit_cast(unsigned, r);
}

__global__ void __launch_bounds__(256, 1) gemm_ar_kernel(Params p) {
  __shared__ __attribute__((aligned(16))) char lds[LDS_BYTES];
  const int tid = threadIdx.x;
  const int lane = tid & 63;
  const int wv = __builtin_amdgcn_readfirstlane(tid >> 6);
  const int rank = p.rank;
  char* myws = p.ws[rank];
  unsigned* tmo = (unsigned*)(myws + TMO_OFF);
  const int r16 = lane & 15, g4 = lane >> 4;
  STAMP(0);

  const int pi = rank >> 2, pj = rank & 3;
  __amdgpu_buffer_rsrc_t mrs = make_rsrc(myws, WS_USED);
  LDS_AS unsigned* cnt = (LDS_AS unsigned*)(lds + LDS_CTRL);
  unsigned call;
  {
    unsigned ticket = 0;
    if (lane == 0)
      ticket = __hip_atomic_fetch_add((unsigned*)(myws + TK_OFF + (blockIdx.x & 31) * 128), 1u, __ATOMIC_RELAXED, __HIP_MEMORY_SCOPE_AGENT);
    const int gt = blockIdx.x >> 3, ksl = blockIdx.x & 7;
    const int cbr = 8 * ((r16 & 3) >> 1) + 2 * (r16 >> 2) + (r16 & 1);
    const unsigned short* wsrc = p.w + (size_t)(ksl * 64 + (wv >> 1) * 32 + 8 * g4) * DIM + gt * 128 + 64 * (wv & 1) + 4 * cbr;
    const unsigned short* xsrc = p.x + (size_t)(gt * 128 + wv * 32 + r16) * KSL + ksl * 64 + 8 * g4;
    u32x2 wr[8];
    u32x4 xr[4];
#pragma unroll
    for (int h = 0; h < 8; ++h) wr[h] = *(const u32x2*)(wsrc + (size_t)h * DIM);
#pragma unroll
    for (int h = 0; h < 4; ++h) xr[h] = *(const u32x4*)(xsrc + (size_t)(h >> 1) * 16 * KSL + (h & 1) * 32);
    if (tid < 16) cnt[tid] = 0;
    asm volatile("s_waitcnt lgkmcnt(0)" ::: "memory");
    __builtin_amdgcn_s_barrier();
    const unsigned wo = ((gt >> 3) == pj ? WS_OFF + (unsigned)((gt & 7) * NKS + 56 + ksl) * KSB : SW_OFF + (unsigned)(gt * 8 + ksl) * KSB) + wv * 4096 + lane * 16;
    const unsigned xo = ((gt >> 4) == pi ? XS_OFF + (unsigned)((gt & 15) * NKS + 56 + ksl) * KSB : SX_OFF + (unsigned)((gt & 15) * 8 + ksl) * KSB) + wv * 4096 + lane * 16;
#pragma unroll
    for (int t = 0; t < 4; ++t) {
      u32x4 v;
#pragma unroll
      for (int h = 0; h < 4; ++h)
        v[h] = __builtin_amdgcn_perm(wr[2 * h + 1][t >> 1], wr[2 * h][t >> 1], (t & 1) ? 0x07060302u : 0x05040100u);
      __builtin_amdgcn_raw_buffer_store_b128(v, mrs, wo + t * 1024, 0, AUX_SYS);
    }
    asm volatile("" ::: "memory");
#pragma unroll
    for (int h = 0; h < 4; ++h) __builtin_amdgcn_raw_buffer_store_b128(xr[h], mrs, xo + h * 1024, 0, AUX_SYS);
    call = __builtin_amdgcn_readfirstlane(ticket) / 32u + 1u;
#pragma unroll
    for (int ph = 0; ph < 2; ++ph) {
      if (ph == 0) asm volatile("s_waitcnt vmcnt(4)" ::: "memory");
      else asm volatile("s_waitcnt vmcnt(0)" ::: "memory");
      unsigned old = 0;
      if (lane == 0) old = __hip_atomic_fetch_add(cnt + ph, 1u, __ATOMIC_RELAXED, __HIP_MEMORY_SCOPE_WORKGROUP);
      old = __builtin_amdgcn_readfirstlane(old);
      const bool need = ph == 0 ? ((lane & 3) == (gt >> 3)) : ((lane >> 2) == (gt >> 4));
      if (old == 3 && lane < NR && need) {
        const int unit = ph == 0 ? (gt & 7) : 8 + (gt & 15);
        unsigned* fl = lane == rank ? (unsigned*)(myws + IF_OFF) + unit * 64 + 56 + ksl
                                    : (unsigned*)(p.ws[lane] + PF_OFF) + rank * 192 + unit * 8 + ksl;
        __hip_atomic_store(fl, call, __ATOMIC_RELAXED, __HIP_MEMORY_SCOPE_SYSTEM);
      }
    }
  }
  STAMP(1);

  if (blockIdx.x < 128) {
    const int mb = blockIdx.x;
    if (mb >= 112) return;
    const int bi = __builtin_amdgcn_readfirstlane((mb * 9363) >> 16);
    const int lk = __builtin_amdgcn_readfirstlane(mb - 7 * bi);
    const int src = (rank + 1 + lk) & 7;
    const int wi = bi + 16 * wv;
    __amdgpu_buffer_rsrc_t srs = make_rsrc(p.ws[src], WS_USED);
    bool bad = false;
    if (wi < NIN) {
      constexpr int NT = 192 / NIN;
      {
        unsigned* fl = (unsigned*)(myws + PF_OFF) + src * 192 + wi + (lane < NT ? lane : 0) * NIN;
        u64 t0 = __builtin_amdgcn_s_memrealtime();
        for (;;) {
          unsigned v = __hip_atomic_load(fl, __ATOMIC_RELAXED, __HIP_MEMORY_SCOPE_SYSTEM);
          if (__all(v >= call)) break;
          if (__builtin_amdgcn_readfirstlane(timed_out(tmo, call, t0))) { bad = true; break; }
          __builtin_amdgcn_s_sleep(1);
        }
      }
      STAMP(2);
      if (!bad) {
        const int ksl = wi & 7;
        auto src_off = [&](int j) -> unsigned {
          const int unit = (wi >> 3) + j * (NIN / 8);
          if (unit >= 24) return 0xF0000000u;
          if (unit < 8) return (src & 3) == pj ? WS_OFF + (unsigned)(unit * NKS + 56 + ksl) * KSB : SW_OFF + (unsigned)((pj * 8 + unit) * 8 + ksl) * KSB;
          return (src >> 2) == pi ? XS_OFF + (unsigned)((unit - 8) * NKS + 56 + ksl) * KSB : SX_OFF + (unsigned)((unit - 8) * 8 + ksl) * KSB;
        };
        auto dst_off = [&](int j) -> unsigned {
          const int unit = (wi >> 3) + j * (NIN / 8);
          return (unit < 8 ? WS_OFF + (unsigned)(unit * NKS) * KSB : XS_OFF + (unsigned)((unit - 8) * NKS) * KSB) + (unsigned)(lk * 8 + ksl) * KSB;
        };
        unsigned* ifl = (unsigned*)(myws + IF_OFF) + (wi >> 3) * 64 + lk * 8 + ksl;
        u32x4 v[DEPTH];
        const unsigned l16 = lane * 16;
        const unsigned oob = 0xF0000000u;
#define LD(r, vo, so, imm) asm volatile("buffer_load_dwordx4 %0, %1, %2, 0 offen offset:%3 sc0 sc1" : "=v"(r) : "v"((vo) + (so)), "s"(srs), "i"(imm) : "memory")
#define ST(r, vo, so, imm) asm volatile("buffer_store_dwordx4 %0, %1, %2, 0 offen offset:%3 sc1\n\ts_nop 1" :: "v"(r), "v"((vo) + (so)), "s"(mrs), "i"(imm) : "memory")
        unsigned so = src_off(0);
        asm volatile("s_nop 4" ::: "memory");
        {
          u32x4 z = {0, 0, 0, 0};
#pragma unroll
          for (int u = 0; u < DEPTH; ++u) { ST(z, l16, oob, 0); LD(v[u], l16, so + (u >> 2) * 4096, (u & 3) * 1024); }
        }
        for (int j = 0; j < NT; ++j) {
          const unsigned sn = src_off(j + 1), dn = dst_off(j);
          asm volatile("s_nop 4" ::: "memory");
#pragma unroll
          for (int q = 0; q < 16; ++q) {
            asm volatile("s_waitcnt vmcnt(%1)" : "+v"(v[q % DEPTH]) : "i"(2 * (DEPTH - 1)) : "memory");
            if (q == DEPTH - 1 && j > 0 && lane == 0)
              __hip_atomic_store(ifl + (j - 1) * (NIN / 8) * 64, call, __ATOMIC_RELAXED, __HIP_MEMORY_SCOPE_AGENT);
            ST(v[q % DEPTH], l16, dn + (q >> 2) * 4096, (q & 3) * 1024);
            const int qn = q + DEPTH;
            if (qn < 16) LD(v[q % DEPTH], l16, so + (qn >> 2) * 4096, (qn & 3) * 1024);
            else LD(v[q % DEPTH], l16, sn + ((qn - 16) >> 2) * 4096, (qn & 3) * 1024);
          }
          so = sn;
        }
        asm volatile("s_waitcnt vmcnt(0)" : "+v"(v[0]) :: "memory");
#pragma unroll
        for (int u = 1; u < DEPTH; ++u) asm volatile("" : "+v"(v[u]) :: "memory");
        if (lane == 0) __hip_atomic_store(ifl + (NT - 1) * (NIN / 8) * 64, call, __ATOMIC_RELAXED, __HIP_MEMORY_SCOPE_AGENT);
      }
      if (lane == 0) __hip_atomic_fetch_add(cnt + 2, 1u, __ATOMIC_RELAXED, __HIP_MEMORY_SCOPE_WORKGROUP);
    }
    if (wi >= NOUT) return;
    while (__hip_atomic_load(cnt + 2, __ATOMIC_RELAXED, __HIP_MEMORY_SCOPE_WORKGROUP) < (unsigned)(NIN / 16)) __builtin_amdgcn_s_sleep(8);
    STAMP(3);

    LDS_AS char* seg = (LDS_AS char*)(lds + wv * SEG);
    for (int sg = wi; sg < 512; sg += NOUT) {
      const int tile = sg >> 2, sw = sg & 3;
      const int rt = tile >> 3, ct = tile & 7;
      u64* flag = (u64*)(myws + OF_OFF + (unsigned)(src * 128 + tile) * 64);
      u64 f;
      {
        u64 t0 = __builtin_amdgcn_s_memrealtime();
        for (;;) {
          f = __hip_atomic_load(flag, __ATOMIC_RELAXED, __HIP_MEMORY_SCOPE_SYSTEM);
          if ((unsigned)(f >> 32) >= call) break;
          if (timed_out(tmo, call, t0)) { bad = true; break; }
          __builtin_amdgcn_s_sleep(4);
        }
      }
      bad = __builtin_amdgcn_readfirstlane(bad);
      unsigned short* o = p.out + (size_t)((src >> 2) * 2048 + rt * 128 + (sw >> 1) * 64 + r16) * DIM + (src & 3) * 1024 + ct * 128 + 64 * (sw & 1) + 8 * g4;
      if (bad) {
        const u32x4 nanv = {0x7fc07fc0u, 0x7fc07fc0u, 0x7fc07fc0u, 0x7fc07fc0u};
#pragma unroll
        for (int c = 0; c < 8; ++c) *(u32x4*)(o + (size_t)(c >> 1) * 16 * DIM + (c & 1) * 32) = nanv;
        continue;
      }
      const unsigned n64 = (__builtin_amdgcn_readfirstlane((unsigned)f) >> (8 * sw)) & 255u;
      const unsigned so = OB_OFF + (unsigned)sg * SEG + lane * 16;
      u32x4 pc[9];
#pragma unroll
      for (int i = 0; i < 9; ++i)
        pc[i] = __builtin_amdgcn_raw_buffer_load_b128(srs, (unsigned)(i * 64 + lane) < n64 * 4 ? so + i * 1024 : 0xF0000000u, 0, AUX_SYS);
#pragma unroll
      for (int i = 0; i < 9; ++i) *(LDS_AS u32x4*)(seg + i * 1024 + lane * 16) = pc[i];
      const u32x2 mk = *(LDS_AS u32x2*)(seg + lane * 8);
      unsigned base = 256;
#pragma unroll
      for (int c = 0; c < 8; ++c) {
        unsigned h[8];
#pragma unroll
        for (int e = 0; e < 8; ++e) {
          const unsigned lo = __builtin_amdgcn_readlane(mk.x, c * 8 + e);
          const unsigned hi = __builtin_amdgcn_readlane(mk.y, c * 8 + e);
          const unsigned pos = __builtin_amdgcn_mbcnt_hi(hi, __builtin_amdgcn_mbcnt_lo(lo, base));
          const unsigned v = *(LDS_AS unsigned short*)(seg + pos * 2);
          const u64 m = ((u64)hi << 32) | lo;
          h[e] = __builtin_amdgcn_inverse_ballot_w64(m) ? v : 0u;
          base += __builtin_popcountll(m);
        }
        u32x4 v = {h[0] | (h[1] << 16), h[2] | (h[3] << 16), h[4] | (h[5] << 16), h[6] | (h[7] << 16)};
        *(u32x4*)(o + (size_t)(c >> 1) * 16 * DIM + (c & 1) * 32) = v;
      }
    }
    if (__hip_atomic_load(tmo, __ATOMIC_RELAXED, __HIP_MEMORY_SCOPE_AGENT) == call && lane == 0)
      p.out[(size_t)((src >> 2) * 2048) * DIM + (src & 3) * 1024 + wi] = 0x7fc0;
    return;
  }

  const int gb = blockIdx.x - 128;
  const int ct = gb & 7, rt = gb >> 3;
  const int wm = wv >> 1, hf = wv & 1;
  LDS_AS u64* ctl = (LDS_AS u64*)(lds + LDS_CTRL + 32);
  if (wv < 2) {
    unsigned* fl = (unsigned*)(myws + IF_OFF) + (wv == 0 ? ct : 8 + rt) * 64 + lane;
    u64 t0 = __builtin_amdgcn_s_memrealtime();
    for (;;) {
      unsigned v = __hip_atomic_load(fl, __ATOMIC_RELAXED, __HIP_MEMORY_SCOPE_SYSTEM);
      if (__all(v >= call)) break;
      if (__builtin_amdgcn_readfirstlane(timed_out(tmo, call, t0))) break;
      __builtin_amdgcn_s_sleep(8);
    }
    __builtin_amdgcn_fence(__ATOMIC_ACQUIRE, "");
    asm volatile("s_waitcnt vmcnt(0)" ::: "memory");
  }
  __syncthreads();
  const bool bad = __builtin_amdgcn_readfirstlane(__hip_atomic_load(tmo, __ATOMIC_RELAXED, __HIP_MEMORY_SCOPE_AGENT) == call);
  unsigned short* o = p.out + (size_t)((rank >> 2) * 2048 + rt * 128 + wm * 64 + r16) * DIM + (rank & 3) * 1024 + ct * 128 + 64 * hf + 8 * g4;
  if (bad) {
    const u32x4 nanv = {0x7fc07fc0u, 0x7fc07fc0u, 0x7fc07fc0u, 0x7fc07fc0u};
#pragma unroll
    for (int c = 0; c < 8; ++c) *(u32x4*)(o + (size_t)(c >> 1) * 16 * DIM + (c & 1) * 32) = nanv;
    return;
  }
  STAMP(2);

  f32x4 acc[8][8];
  {
    const unsigned bx = XS_OFF + (unsigned)rt * (NKS * KSB) + lane * 16, bw = WS_OFF + (unsigned)ct * (NKS * KSB) + lane * 16;
    const unsigned vx0 = bx + wm * 8192, vx1 = bx + (wm ^ 1) * 8192;
    const unsigned vw0 = bw + hf * 4096, vw1 = bw + (hf ^ 1) * 4096;
#pragma unroll
    for (int a = 0; a < 8; ++a)
#pragma unroll
      for (int b = 0; b < 8; ++b) acc[a][b] = f32x4{0.f, 0.f, 0.f, 0.f};
    u32x4 fw[3][8], fx[3][8];
#define LDW(dst, vo, so, imm) asm volatile("buffer_load_dwordx4 %0, %1, %2, %3 offen offset:%4" : "=v"(dst) : "v"(vo), "s"(mrs), "s"(so), "i"(imm))
#define LDX(dst, vo, so, imm) asm volatile("buffer_load_dwordx4 %0, %1, %2, %3 offen offset:%4" : "=v"(dst) : "v"(vo), "s"(mrs), "s"(so), "i"(imm))
#define MF(T, M, w_, x_) asm volatile("v_mfma_f32_16x16x32_bf16 %0, %1, %2, %0" : "+a"(acc[T][M]) : "v"(w_[T]), "v"(x_[M]))
#define WAITF(w_, x_, n) asm volatile("s_waitcnt vmcnt(" #n ")" : "+v"(w_[0]), "+v"(w_[1]), "+v"(w_[2]), "+v"(w_[3]), "+v"(w_[4]), "+v"(w_[5]), "+v"(w_[6]), "+v"(w_[7]), \
                   "+v"(x_[0]), "+v"(x_[1]), "+v"(x_[2]), "+v"(x_[3]), "+v"(x_[4]), "+v"(x_[5]), "+v"(x_[6]), "+v"(x_[7]))
    auto load1 = [&](int i, unsigned sb, int kk, u32x4* w_, u32x4* x_) {
      const int q = i >> 2, j = i & 3;
      if (q == 0) LDW(w_[j], vw0, sb + kk * 8192, j * 1024);
      else if (q == 1) LDX(x_[j], vx0, sb + (j >> 1) * 4096, (j & 1) * 2048 + kk * 1024);
      else if (q == 2) LDW(w_[4 + j], vw1, sb + kk * 8192, j * 1024);
      else LDX(x_[4 + j], vx1, sb + (j >> 1) * 4096, (j & 1) * 2048 + kk * 1024);
    };
    auto half = [&](u32x4* w_, u32x4* x_, bool doload, unsigned sb, int kk, u32x4* wn, u32x4* xn) {
#pragma unroll
      for (int i = 0; i < 16; ++i) {
        const int qa = i >> 3, qb = (i >> 2) & 1, m = i & 3;
#pragma unroll
        for (int t = 0; t < 4; ++t) MF((qa ^ qb) * 4 + t, qa * 4 + m, w_, x_);
        if (doload) load1(i, sb, kk, wn, xn);
      }
    };
    const unsigned sb0 = (unsigned)wv * KSB;
    asm volatile("s_waitcnt vmcnt(0)\n\ts_nop 4" ::: "memory");
#pragma unroll
    for (int i = 0; i < 16; ++i) load1(i, sb0, 0, fw[0], fx[0]);
#pragma unroll
    for (int i = 0; i < 16; ++i) load1(i, sb0, 1, fw[1], fx[1]);
    for (int s = 0; s < 30; s += 6) {
#pragma unroll
      for (int u = 0; u < 6; ++u) {
        WAITF(fw[u % 3], fx[u % 3], 16);
        half(fw[u % 3], fx[u % 3], true, sb0 + (unsigned)((s + u + 2) >> 1) * (4 * KSB), u & 1, fw[(u + 2) % 3], fx[(u + 2) % 3]);
      }
    }
    WAITF(fw[0], fx[0], 16);
    half(fw[0], fx[0], false, 0, 0, fw[0], fx[0]);
    WAITF(fw[1], fx[1], 0);
    half(fw[1], fx[1], false, 0, 0, fw[0], fx[0]);
    asm volatile("s_nop 15\n\ts_nop 15" ::: "memory");
    __builtin_amdgcn_sched_barrier(0);
  }
  {
    LDS_AS f32x4* ex = (LDS_AS f32x4*)lds;
#pragma unroll
    for (int t = 0; t < 4; ++t)
#pragma unroll
      for (int m = 0; m < 8; ++m) ex[(wv * 32 + t * 8 + m) * 64 + lane] = acc[4 + t][m];
    __syncthreads();
#pragma unroll
    for (int t = 0; t < 4; ++t)
#pragma unroll
      for (int m = 0; m < 8; ++m) acc[t][m] += ex[((wv ^ 1) * 32 + t * 8 + m) * 64 + lane];
    __syncthreads();
#pragma unroll
    for (int t = 0; t < 4; ++t)
#pragma unroll
      for (int m = 0; m < 4; ++m) ex[(64 + wv * 16 + t * 4 + m) * 64 + lane] = acc[t][4 + m];
    __syncthreads();
#pragma unroll
    for (int t = 0; t < 4; ++t)
#pragma unroll
      for (int m = 0; m < 4; ++m) acc[t][m] += ex[(64 + (wv ^ 2) * 16 + t * 4 + m) * 64 + lane];
  }
  STAMP(3);

  {
    LDS_AS char* seg = (LDS_AS char*)(lds + wv * SEG);
    unsigned base = 256;
    unsigned mlo = 0, mhi = 0;
#pragma unroll
    for (int c = 0; c < 8; ++c) {
      const int mt = c >> 1, pc = c & 1;
      float v[8];
#pragma unroll
      for (int e = 0; e < 8; ++e) {
        const float a = acc[e & 3][mt][2 * pc + (e >> 2)];
        v[e] = (a <= 0.f) ? 0.f : a;
      }
      u32x4 pk = {pack_bf16(v[0], v[1]), pack_bf16(v[2], v[3]), pack_bf16(v[4], v[5]), pack_bf16(v[6], v[7])};
      *(u32x4*)(o + (size_t)mt * 16 * DIM + pc * 32) = pk;
#pragma unroll
      for (int e = 0; e < 8; ++e) {
        const unsigned h = (e & 1) ? (pk[e >> 1] >> 16) : (pk[e >> 1] & 0xffffu);
        const u64 m = __ballot(h != 0);
        const unsigned lo = (unsigned)m, hi = (unsigned)(m >> 32);
        const unsigned pos = __builtin_amdgcn_mbcnt_hi(hi, __builtin_amdgcn_mbcnt_lo(lo, base));
        if (h != 0) *(LDS_AS unsigned short*)(seg + pos * 2) = (unsigned short)h;
        base += __builtin_popcountll(m);
        mlo = (lane == c * 8 + e) ? lo : mlo;
        mhi = (lane == c * 8 + e) ? hi : mhi;
      }
    }
    *(LDS_AS u32x2*)(seg + lane * 8) = u32x2{mlo, mhi};
    const unsigned n64 = (base * 2 + 63) >> 6;
    const unsigned so = OB_OFF + (unsigned)(gb * 4 + wv) * SEG + lane * 16;
#pragma unroll
    for (int i = 0; i < 9; ++i) {
      const u32x4 v = *(LDS_AS u32x4*)(seg + i * 1024 + lane * 16);
      __builtin_amdgcn_raw_buffer_store_b128(v, mrs, (unsigned)(i * 64 + lane) < n64 * 4 ? so + i * 1024 : 0xF0000000u, 0, AUX_SYS);
    }
    STAMP(4);
    asm volatile("s_waitcnt vmcnt(0)" ::: "memory");
    u64 old = 0;
    if (lane == 0)
      old = __hip_atomic_fetch_add(ctl, (1ull << 32) | ((u64)n64 << (8 * wv)), __ATOMIC_RELAXED, __HIP_MEMORY_SCOPE_WORKGROUP);
    const unsigned olo = __builtin_amdgcn_readfirstlane((unsigned)old);
    const unsigned ohi = __builtin_amdgcn_readfirstlane((unsigned)(old >> 32));
    if (ohi == 3 && lane < NR && lane != rank) {
      const u64 f = ((u64)call << 32) | (olo + (n64 << (8 * wv)));
      __hip_atomic_store((u64*)(p.ws[lane] + OF_OFF + (unsigned)(rank * 128 + gb) * 64), f, __ATOMIC_RELAXED, __HIP_MEMORY_SCOPE_SYSTEM);
    }
    STAMP(5);
  }
}

}

extern "C" void kernel_launch(int rank, int nranks, ncclComm_t comm,
                              hipStream_t stream,
                              void* const* d_in, const int* in_sizes, int n_in,
                              void* d_out, int out_size,
                              const int* in_dtypes, int out_dtype,
                              void* const* d_out_all,
                              void* const* d_ws_all, size_t ws_bytes) {
  (void)comm; (void)in_sizes; (void)n_in; (void)out_size; (void)in_dtypes; (void)out_dtype; (void)d_out_all;
  if (nranks != NR || (unsigned)rank >= (unsigned)NR || ws_bytes < (size_t)WS_USED) {
    fprintf(stderr, "unsupported configuration\n");
    return;
  }
  Params p;
  p.x = (const unsigned short*)d_in[0];
  p.w = (const unsigned short*)d_in[1];
  p.out = (unsigned short*)d_out;
  for (int i = 0; i < NR; ++i) p.ws[i] = (char*)d_ws_all[i];
  p.rank = rank;
  gemm_ar_kernel<<<256, 256, 0, stream>>>(p);
  HIP_CHECK(hipGetLastError());
}
```

```cpp
#include <hip/hip_runtime.h>
#include <rccl/rccl.h>
#include <cstdio>
#include <cstdint>

#define HIP_CHECK(expr) do { hipError_t _e = (expr); if (_e != hipSuccess) { \
    fprintf(stderr, "HIP error %s at %s:%d\n", hipGetErrorString(_e), __FILE__, __LINE__); \
    return; } } while (0)

namespace {

typedef short bf16x8 __attribute__((ext_vector_type(8)));
typedef float f32x4 __attribute__((ext_vector_type(4)));
typedef unsigned u32x4 __attribute__((ext_vector_type(4)));
typedef unsigned u32x2 __attribute__((ext_vector_type(2)));
typedef unsigned long long u64;

constexpr int NR = 8;
constexpr int DIM = 4096;
constexpr int KSL = DIM / NR;
constexpr int RT = 16, CT = 8;
constexpr int NKS = 64;
constexpr unsigned KSB = 16384;
constexpr unsigned SEG = 9216;

constexpr unsigned XS_OFF = 0;
constexpr unsigned WS_OFF = XS_OFF + RT * NKS * KSB;
constexpr unsigned SX_OFF = WS_OFF + CT * NKS * KSB;
constexpr unsigned SW_OFF = SX_OFF + RT * 8 * KSB;
constexpr unsigned OB_OFF = SW_OFF + 32 * 8 * KSB;
constexpr unsigned IF_OFF = OB_OFF + RT * CT * 4 * SEG;
constexpr unsigned PF_OFF = IF_OFF + 24 * 256;
constexpr unsigned OF_OFF = PF_OFF + NR * 192 * 4;
constexpr unsigned TK_OFF = OF_OFF + NR * RT * CT * 64;
constexpr unsigned QC_OFF = TK_OFF + 32 * 128;
constexpr unsigned TMO_OFF = QC_OFF + 16 * 128;
constexpr unsigned DBG_OFF = TMO_OFF + 4096;
constexpr unsigned WS_USED = DBG_OFF + 256 * 4 * 16 * 8;

#define DEPTH 4
#define ODEPTH 5
constexpr int NIN = 16;
constexpr int NSLOT = 12;

constexpr u64 DEADLINE_TICKS = 50000000ull;

constexpr int LDS_STAGE = 32768;
constexpr int LDS_CTRL = 4 * LDS_STAGE;
constexpr int LDS_BYTES = LDS_CTRL + 256;

#define MU4 4
struct OrderTable { unsigned char t[192]; };
constexpr OrderTable make_order() {
  OrderTable o{};
  int n = 0;
  for (int key = 0; key <= 4 * 23 + MU4 * 7; ++key)
    for (int k = 7; k >= 0; --k)
      for (int s = 0; s < 24; ++s)
        if (4 * s + MU4 * k == key) o.t[n++] = (unsigned char)((s < 16 ? ((s & 1) ? (s >> 1) : 8 + (s >> 1)) : s) * 8 + k);
  return o;
}

struct Params {
  const unsigned short* x;
  const unsigned short* w;
  unsigned short* out;
  char* ws[NR];
  int rank;
  OrderTable order;
};

#define LDS_AS __attribute__((address_space(3)))
#define GLB_AS __attribute__((address_space(1)))

__device__ __forceinline__ __amdgpu_buffer_rsrc_t make_rsrc(const void* p, unsigned bytes) {
  return __builtin_amdgcn_make_buffer_rsrc((void*)p, 0, bytes, 0x00020000);
}
constexpr int AUX_SYS = 17;

#define STAMP(i) do { } while (0)

__device__ __forceinline__ bool timed_out(unsigned* tmo, unsigned call, u64 t0) {
  if (__hip_atomic_load(tmo, __ATOMIC_RELAXED, __HIP_MEMORY_SCOPE_AGENT) == call ||
      __builtin_amdgcn_s_memrealtime() - t0 > DEADLINE_TICKS) {
    __hip_atomic_store(tmo, call, __ATOMIC_RELAXED, __HIP_MEMORY_SCOPE_AGENT);
    return true;
  }
  return false;
}

__device__ __forceinline__ unsigned pack_bf16(float a, float b) {
  typedef __bf16 bf2 __attribute__((ext_vector_type(2)));
  typedef float f2 __attribute__((ext_vector_type(2)));
  f2 v = {a, b};
  bf2 r = __builtin_convertvector(v, bf2);
  return __builtin_bit_cast(unsigned, r);
}

__device__ __forceinline__ unsigned wave_scan(unsigned x) {
  x += __builtin_amdgcn_update_dpp(0u, x, 0x111, 0xf, 0xf, false);
  x += __builtin_amdgcn_update_dpp(0u, x, 0x112, 0xf, 0xf, false);
  x += __builtin_amdgcn_update_dpp(0u, x, 0x114, 0xf, 0xf, false);
  x += __builtin_amdgcn_update_dpp(0u, x, 0x118, 0xf, 0xf, false);
  x += __builtin_amdgcn_update_dpp(0u, x, 0x142, 0xa, 0xf, false);
  x += __builtin_amdgcn_update_dpp(0u, x, 0x143, 0xc, 0xf, false);
  return x;
}
typedef unsigned short u16x2 __attribute__((ext_vector_type(2)));
typedef unsigned __attribute__((aligned(2))) u32_a2;

__device__ __forceinline__ int tile_of(int i) {
  if (i >= 64) return i;
  const int n = (i >= 1) + (i >= 4) + (i >= 9) + (i >= 16) + (i >= 25) + (i >= 36) + (i >= 49);
  const int rem = i - n * n;
  return rem < n ? n * 8 + rem : (rem - n) * 8 + n;
}

__global__ void __launch_bounds__(256, 1) gemm_ar_kernel(Params p) {
  __shared__ __attribute__((aligned(16))) char lds[LDS_BYTES];
  const int tid = threadIdx.x;
  const int lane = tid & 63;
  const int wv = __builtin_amdgcn_readfirstlane(tid >> 6);
  const int rank = p.rank;
  char* myws = p.ws[rank];
  unsigned* tmo = (unsigned*)(myws + TMO_OFF);
  const int r16 = lane & 15, g4 = lane >> 4;
  STAMP(0);

  const int pi = rank >> 2, pj = rank & 3;
  __amdgpu_buffer_rsrc_t mrs = make_rsrc(myws, WS_USED);
  LDS_AS unsigned* cnt = (LDS_AS unsigned*)(lds + LDS_CTRL);
  unsigned call;
  {
    unsigned ticket = 0;
    if (lane == 0)
      ticket = __hip_atomic_fetch_add((unsigned*)(myws + TK_OFF + (blockIdx.x & 31) * 128), 1u, __ATOMIC_RELAXED, __HIP_MEMORY_SCOPE_AGENT);
    const int gt = blockIdx.x >> 3, ksl = blockIdx.x & 7;
    const int cbr = 8 * ((r16 & 3) >> 1) + 2 * (r16 >> 2) + (r16 & 1);
    const unsigned short* wsrc = p.w + (size_t)(ksl * 64 + (wv >> 1) * 32 + 8 * g4) * DIM + gt * 128 + 64 * (wv & 1) + 4 * cbr;
    const unsigned short* xsrc = p.x + (size_t)(gt * 128 + wv * 32 + r16) * KSL + ksl * 64 + 8 * g4;
    u32x2 wr[8];
    u32x4 xr[4];
#pragma unroll
    for (int h = 0; h < 8; ++h) wr[h] = *(const u32x2*)(wsrc + (size_t)h * DIM);
#pragma unroll
    for (int h = 0; h < 4; ++h) xr[h] = *(const u32x4*)(xsrc + (size_t)(h >> 1) * 16 * KSL + (h & 1) * 32);
    if (tid < 64) cnt[tid] = 0;
    asm volatile("s_waitcnt lgkmcnt(0)" ::: "memory");
    __builtin_amdgcn_s_barrier();
    const unsigned wo = ((gt >> 3) == pj ? WS_OFF + (unsigned)((gt & 7) * NKS + 56 + ksl) * KSB : SW_OFF + (unsigned)(gt * 8 + ksl) * KSB) + wv * 4096 + lane * 16;
    const unsigned xo = ((gt >> 4) == pi ? XS_OFF + (unsigned)((gt & 15) * NKS + 56 + ksl) * KSB : SX_OFF + (unsigned)((gt & 15) * 8 + ksl) * KSB) + wv * 4096 + lane * 16;
#pragma unroll
    for (int t = 0; t < 4; ++t) {
      u32x4 v;
#pragma unroll
      for (int h = 0; h < 4; ++h)
        v[h] = __builtin_amdgcn_perm(wr[2 * h + 1][t >> 1], wr[2 * h][t >> 1], (t & 1) ? 0x07060302u : 0x05040100u);
      __builtin_amdgcn_raw_buffer_store_b128(v, mrs, wo + t * 1024, 0, AUX_SYS);
    }
    asm volatile("" ::: "memory");
#pragma unroll
    for (int h = 0; h < 4; ++h) __builtin_amdgcn_raw_buffer_store_b128(xr[h], mrs, xo + h * 1024, 0, AUX_SYS);
    call = __builtin_amdgcn_readfirstlane(ticket) / 32u + 1u;
#pragma unroll
    for (int ph = 0; ph < 2; ++ph) {
      if (ph == 0) asm volatile("s_waitcnt vmcnt(4)" ::: "memory");
      else asm volatile("s_waitcnt vmcnt(0)" ::: "memory");
      unsigned old = 0;
      if (lane == 0) old = __hip_atomic_fetch_add(cnt + ph, 1u, __ATOMIC_RELAXED, __HIP_MEMORY_SCOPE_WORKGROUP);
      old = __builtin_amdgcn_readfirstlane(old);
      const bool need = ph == 0 ? ((lane & 3) == (gt >> 3)) : ((lane >> 2) == (gt >> 4));
      if (old == 3 && lane < NR && need) {
        const int unit = ph == 0 ? (gt & 7) : 8 + (gt & 15);
        unsigned* fl = lane == rank ? (unsigned*)(myws + IF_OFF) + unit * 64 + 56 + ksl
                                    : (unsigned*)(p.ws[lane] + PF_OFF) + rank * 192 + unit * 8 + ksl;
        __hip_atomic_store(fl, call, __ATOMIC_RELAXED, __HIP_MEMORY_SCOPE_SYSTEM);
      }
    }
  }
  STAMP(1);

  if (blockIdx.x < 128) {
    const int mb = blockIdx.x;
    if (mb >= 112) return;
    const int bi = __builtin_amdgcn_readfirstlane((mb * 9363) >> 16);
    const int lk = __builtin_amdgcn_readfirstlane(mb - 7 * bi);
    const int src = (rank + 1 + lk) & 7;
    const int wi = bi;
    const unsigned l16 = lane * 16;
    const unsigned oob = 0xF0000000u;
    LDS_AS u64* rdesc = (LDS_AS u64*)(lds + LDS_CTRL + 64);
    LDS_AS unsigned* rfree = (LDS_AS unsigned*)(lds + LDS_CTRL + 160);
    LDS_AS unsigned* rland = (LDS_AS unsigned*)(lds + LDS_CTRL + 208);
#define LDS_LD(ptr) __hip_atomic_load(ptr, __ATOMIC_RELAXED, __HIP_MEMORY_SCOPE_WORKGROUP)
#define LDS_ST(ptr, val) __hip_atomic_store(ptr, val, __ATOMIC_RELAXED, __HIP_MEMORY_SCOPE_WORKGROUP)
    if (wv != 0) {
      unsigned short* ob = p.out + (size_t)((src >> 2) * 2048 + r16) * DIM + (src & 3) * 1024 + 8 * g4;
      int slot = wv - 1;
      for (unsigned sq = wv - 1;; sq += 3) {
        u64 d;
        for (;;) {
          d = LDS_LD(rdesc + slot);
          if ((unsigned)(d >> 48) == sq + 1) break;
          __builtin_amdgcn_s_sleep(4);
        }
        const unsigned dhi = __builtin_amdgcn_readfirstlane((unsigned)(d >> 32)), dend = __builtin_amdgcn_readfirstlane((unsigned)d);
        const unsigned sg = dhi & 0x3ffu;
        if (sg >= 512) break;
        while (LDS_LD(rland) < dend) __builtin_amdgcn_s_sleep(1);
        asm volatile("" ::: "memory");
        LDS_AS char* seg = (LDS_AS char*)(lds + slot * SEG);
        const u32x2 mk = *(LDS_AS u32x2*)(seg + lane * 8);
        const unsigned c0 = __builtin_popcount(mk.x), cn = c0 + __builtin_popcount(mk.y);
        const unsigned b0 = 510 + 2 * (wave_scan(cn) - cn), b1 = b0 + 2 * c0;
        u32x4 ov[8];
#pragma unroll
        for (int c = 0; c < 8; ++c)
#pragma unroll
          for (int dd = 0; dd < 4; ++dd) {
            const int dq = (c & 3) * 4 + dd;
            const unsigned E = c < 4 ? mk.x : mk.y;
            const unsigned pp = __builtin_popcount(E & (((2u << dq) - 1u) | (((1u << dq) - 1u) << 16)));
            ov[c][dd] = *(LDS_AS u32_a2*)(seg + (c < 4 ? b0 : b1) + 2 * pp);
          }
        asm volatile("s_waitcnt lgkmcnt(0)" : "+v"(ov[0]), "+v"(ov[1]), "+v"(ov[2]), "+v"(ov[3]), "+v"(ov[4]), "+v"(ov[5]), "+v"(ov[6]), "+v"(ov[7]) :: "memory");
        LDS_ST(rfree + slot, sq + 1);
#pragma unroll
        for (int c = 0; c < 8; ++c)
#pragma unroll
          for (int dd = 0; dd < 4; ++dd) {
            const unsigned m = ((c < 4 ? mk.x : mk.y) >> ((c & 3) * 4 + dd)) & 0x00010001u;
            const unsigned t = ov[c][dd];
            const u16x2 r = __builtin_bit_cast(u16x2, t) * __builtin_bit_cast(u16x2, m);
            ov[c][dd] = __builtin_bit_cast(unsigned, r);
          }
        if (dhi & 0x8000u) {
#pragma unroll
          for (int c = 0; c < 8; ++c) ov[c] = u32x4{0x7fc07fc0u, 0x7fc07fc0u, 0x7fc07fc0u, 0x7fc07fc0u};
          asm volatile("" ::: "memory");
        }
        const int tile = sg >> 2, sw = sg & 3;
        unsigned short* o = ob + (size_t)((tile >> 3) * 128 + (sw >> 1) * 64) * DIM + (tile & 7) * 128 + 64 * (sw & 1);
#pragma unroll
        for (int c = 0; c < 8; ++c) *(u32x4*)(o + (size_t)(c >> 1) * 16 * DIM + (c & 1) * 32) = ov[c];
        slot += 3;
        if (slot >= NSLOT) slot -= NSLOT;
      }
      STAMP(11);
      return;
    }
    __amdgpu_buffer_rsrc_t srs = make_rsrc(p.ws[src], WS_USED);
    bool bad = false;
    if (mb == 0 && tid < 7)
      __hip_atomic_store((unsigned*)(myws + QC_OFF + (((call + 1) & 1) * 8 + tid) * 128), 0u, __ATOMIC_RELAXED, __HIP_MEMORY_SCOPE_AGENT);
    {
      constexpr int NT = 192 / NIN;
      const unsigned mytask = p.order.t[wi + (lane < NT ? lane : 0) * NIN];
      unsigned* pfl = (unsigned*)(myws + PF_OFF) + src * 192 + mytask;
      u64 ready = 0;
      auto wait_ready = [&](int j) {
        if ((ready >> j) & 1) return;
        u64 t0 = __builtin_amdgcn_s_memrealtime();
        for (;;) {
          unsigned v = __hip_atomic_load(pfl, __ATOMIC_RELAXED, __HIP_MEMORY_SCOPE_SYSTEM);
          ready = __ballot(v >= call);
          if ((ready >> j) & 1) break;
          if (__builtin_amdgcn_readfirstlane(timed_out(tmo, call, t0))) { bad = true; ready = ~0ull; break; }
          __builtin_amdgcn_s_sleep(1);
        }
      };
      auto src_off = [&](int j) -> unsigned {
        if (j >= NT) return oob;
        const int tb = __builtin_amdgcn_readlane(mytask, j), unit = tb >> 3, ksl = tb & 7;
        if (unit < 8) return (src & 3) == pj ? WS_OFF + (unsigned)(unit * NKS + 56 + ksl) * KSB : SW_OFF + (unsigned)((pj * 8 + unit) * 8 + ksl) * KSB;
        return (src >> 2) == pi ? XS_OFF + (unsigned)((unit - 8) * NKS + 56 + ksl) * KSB : SX_OFF + (unsigned)((unit - 8) * 8 + ksl) * KSB;
      };
      unsigned* ifl = (unsigned*)(myws + IF_OFF) + lk * 8;
      u32x4 v[DEPTH];
#define LD(r, vo, so, imm) asm volatile("buffer_load_dwordx4 %0, %1, %2, 0 offen offset:%3 sc0 sc1" : "=v"(r) : "v"((vo) + (so)), "s"(srs), "i"(imm) : "memory")
#define ST(r, vo, so, imm) asm volatile("buffer_store_dwordx4 %0, %1, %2, 0 offen offset:%3 sc1\n\ts_nop 1" :: "v"(r), "v"((vo) + (so)), "s"(mrs), "i"(imm) : "memory")
      wait_ready(0);
      STAMP(2);
      unsigned so = src_off(0);
      asm volatile("s_nop 4" ::: "memory");
      {
        u32x4 z = {0, 0, 0, 0};
#pragma unroll
        for (int u = 0; u < DEPTH; ++u) { ST(z, l16, oob, 0); LD(v[u], l16, so + (u >> 2) * 4096, (u & 3) * 1024); }
      }
      int uprev = 0;
      for (int j = 0; j < NT; ++j) {
        const int tb = __builtin_amdgcn_readlane(mytask, j), unit = tb >> 3, ksl = tb & 7;
        const unsigned sn = src_off(j + 1);
        const unsigned dn = (unit < 8 ? WS_OFF + (unsigned)(unit * NKS) * KSB : XS_OFF + (unsigned)((unit - 8) * NKS) * KSB) + (unsigned)(lk * 8 + ksl) * KSB;
        asm volatile("s_nop 4" ::: "memory");
#pragma unroll
        for (int q = 0; q < 16; ++q) {
          asm volatile("s_waitcnt vmcnt(%1)" : "+v"(v[q % DEPTH]) : "i"(2 * (DEPTH - 1)) : "memory");
          if (q == DEPTH - 1 && j > 0 && lane == 0 && !bad)
            __hip_atomic_store(ifl + uprev, call, __ATOMIC_RELAXED, __HIP_MEMORY_SCOPE_AGENT);
          ST(v[q % DEPTH], l16, dn + (q >> 2) * 4096, (q & 3) * 1024);
          const int qn = q + DEPTH;
          if (qn == 16) wait_ready(j + 1);
          if (qn < 16) LD(v[q % DEPTH], l16, so + (qn >> 2) * 4096, (qn & 3) * 1024);
          else LD(v[q % DEPTH], l16, sn + ((qn - 16) >> 2) * 4096, (qn & 3) * 1024);
        }
        so = sn;
        uprev = unit * 64 + ksl;
      }
      asm volatile("s_waitcnt vmcnt(0)" : "+v"(v[0]) :: "memory");
#pragma unroll
      for (int u = 1; u < DEPTH; ++u) asm volatile("" : "+v"(v[u]) :: "memory");
      if (lane == 0 && !bad) __hip_atomic_store(ifl + uprev, call, __ATOMIC_RELAXED, __HIP_MEMORY_SCOPE_AGENT);
    }
    STAMP(3);

    {
      const unsigned qoff = lane == 0 ? QC_OFF + (unsigned)((call & 1) * 8 + lk) * 128 : oob;
      const unsigned ldsbase = (unsigned)(size_t)(LDS_AS char*)lds;
      u32x2 fl = {0u, 0u};
      unsigned tk = 0;
      int k = 0, land = 0;
      unsigned sq = 0;
      int slot = 0;
      int cur = wi, nxt = wi + NIN;
      auto drain = [&]() {
        asm volatile("s_waitcnt vmcnt(0)" : "+v"(fl), "+v"(tk) :: "memory");
        land = k;
        LDS_ST(rland, (unsigned)k);
      };
      auto next_slot = [&]() {
        if (sq >= (unsigned)NSLOT && LDS_LD(rfree + slot) != sq - NSLOT + 1) {
          drain();
          while (LDS_LD(rfree + slot) != sq - NSLOT + 1) __builtin_amdgcn_s_sleep(1);
        }
      };
      unsigned flo = 0, fhi = 0;
      while (cur < 128) {
        const int tile = tile_of(cur);
        if (fhi < call && !bad) {
          drain();
          u64* flag = (u64*)(myws + OF_OFF + (unsigned)(src * 128 + tile) * 64);
          u64 t0 = __builtin_amdgcn_s_memrealtime();
          for (;;) {
            const u64 f = __hip_atomic_load(flag, __ATOMIC_RELAXED, __HIP_MEMORY_SCOPE_SYSTEM);
            flo = __builtin_amdgcn_readfirstlane((unsigned)f);
            fhi = __builtin_amdgcn_readfirstlane((unsigned)(f >> 32));
            if (fhi >= call) break;
            if (__builtin_amdgcn_readfirstlane(timed_out(tmo, call, t0))) { bad = true; break; }
            __builtin_amdgcn_s_sleep(2);
          }
        }
        if (bad) flo = 0x01010101u;
        tk = 1;
        asm volatile("s_nop 4\n\tbuffer_atomic_add %0, %1, %2, 0 offen sc0" : "+v"(tk) : "v"(nxt < 128 ? qoff : oob), "s"(mrs) : "memory");
        asm volatile("buffer_load_dwordx2 %0, %1, %2, 0 offen sc0 sc1" : "=v"(fl) : "v"(nxt < 128 ? OF_OFF + (unsigned)(src * 128 + tile_of(nxt)) * 64 : oob), "s"(mrs) : "memory");
        k += 2;
        for (int sw = 0; sw < 4; ++sw) {
          next_slot();
          const unsigned n64 = (flo >> (8 * sw)) & 255u;
          const unsigned np = (n64 + 15) >> 4;
          LDS_ST(rdesc + slot, ((u64)(((sq + 1) << 16) | (bad ? 0x8000u : 0u) | (unsigned)(tile * 4 + sw)) << 32) | (unsigned)(k + np));
          unsigned vo = OB_OFF + (unsigned)(tile * 4 + sw) * SEG + l16;
          const unsigned vend = OB_OFF + (unsigned)(tile * 4 + sw) * SEG + n64 * 64;
          unsigned la = ldsbase + slot * SEG;
          for (unsigned i = 0; i < np; ++i) {
            unsigned keep;
            asm volatile("s_waitcnt vmcnt(%0)" :: "i"(ODEPTH - 1) : "memory");
            land = max(land, k - (ODEPTH - 1));
            LDS_ST(rland, (unsigned)land);
            asm volatile("s_mov_b32 %0, m0\n\ts_mov_b32 m0, %3\n\ts_nop 0\n\tbuffer_load_dwordx4 %1, %2, 0 offen sc0 sc1 lds\n\ts_mov_b32 m0, %0"
                         : "=&s"(keep) : "v"(vo < vend ? vo : oob), "s"(srs), "s"(la) : "memory");
            vo += 1024; la += 1024; ++k;
          }
          ++sq;
          if (++slot == NSLOT) slot = 0;
        }
        asm volatile("s_waitcnt vmcnt(4)" : "+v"(fl), "+v"(tk) :: "memory");
        flo = __builtin_amdgcn_readfirstlane(fl.x);
        fhi = __builtin_amdgcn_readfirstlane(fl.y);
        const int nn = nxt < 128 ? 2 * NIN + (int)min(__builtin_amdgcn_readfirstlane(tk), 128u) : 128;
        cur = nxt;
        nxt = nn;
      }
      drain();
      for (int e = 0; e < 3; ++e) {
        next_slot();
        LDS_ST(rdesc + slot, (u64)(((sq + 1) << 16) | 0x3ffu) << 32);
        ++sq;
        if (++slot == NSLOT) slot = 0;
      }
    }
    if (__hip_atomic_load(tmo, __ATOMIC_RELAXED, __HIP_MEMORY_SCOPE_AGENT) == call && lane == 0)
      p.out[(size_t)((src >> 2) * 2048) * DIM + (src & 3) * 1024 + wi] = 0x7fc0;
    return;
  }

  const int gb = blockIdx.x - 128;
  const int ct = gb & 7, rt = gb >> 3;
  const int wm = wv >> 1, hf = wv & 1;
  LDS_AS u64* ctl = (LDS_AS u64*)(lds + LDS_CTRL + 32);
  unsigned short* o = p.out + (size_t)((rank >> 2) * 2048 + rt * 128 + wm * 64 + r16) * DIM + (rank & 3) * 1024 + ct * 128 + 64 * hf + 8 * g4;

  f32x4 acc[8][8];
  {
    const unsigned bx = XS_OFF + (unsigned)rt * (NKS * KSB) + lane * 16, bw = WS_OFF + (unsigned)ct * (NKS * KSB) + lane * 16;
    const unsigned vx0 = bx + wm * 8192, vx1 = bx + (wm ^ 1) * 8192;
    const unsigned vw0 = bw + hf * 4096, vw1 = bw + (hf ^ 1) * 4096;
#pragma unroll
    for (int a = 0; a < 8; ++a)
#pragma unroll
      for (int b = 0; b < 8; ++b) acc[a][b] = f32x4{0.f, 0.f, 0.f, 0.f};
    u32x4 fw[3][8], fx[3][8];
#define LDW(dst, vo, so, imm) asm volatile("buffer_load_dwordx4 %0, %1, %2, %3 offen offset:%4 sc1" : "=v"(dst) : "v"(vo), "s"(mrs), "s"(so), "i"(imm))
#define LDX(dst, vo, so, imm) asm volatile("buffer_load_dwordx4 %0, %1, %2, %3 offen offset:%4 sc1" : "=v"(dst) : "v"(vo), "s"(mrs), "s"(so), "i"(imm))
#define MF(T, M, w_, x_) asm volatile("v_mfma_f32_16x16x32_bf16 %0, %1, %2, %0" : "+a"(acc[T][M]) : "v"(w_[T]), "v"(x_[M]))
#define WAITF(w_, x_, n) asm volatile("s_waitcnt vmcnt(" #n ")" : "+v"(w_[0]), "+v"(w_[1]), "+v"(w_[2]), "+v"(w_[3]), "+v"(w_[4]), "+v"(w_[5]), "+v"(w_[6]), "+v"(w_[7]), \
                   "+v"(x_[0]), "+v"(x_[1]), "+v"(x_[2]), "+v"(x_[3]), "+v"(x_[4]), "+v"(x_[5]), "+v"(x_[6]), "+v"(x_[7]))
    auto load1 = [&](int i, unsigned sb, int kk, u32x4* w_, u32x4* x_) {
      const int q = i >> 2, j = i & 3;
      if (q == 0) LDW(w_[j], vw0, sb + kk * 8192, j * 1024);
      else if (q == 1) LDX(x_[j], vx0, sb + (j >> 1) * 4096, (j & 1) * 2048 + kk * 1024);
      else if (q == 2) LDW(w_[4 + j], vw1, sb + kk * 8192, j * 1024);
      else LDX(x_[4 + j], vx1, sb + (j >> 1) * 4096, (j & 1) * 2048 + kk * 1024);
    };
    auto half = [&](u32x4* w_, u32x4* x_, bool doload, unsigned sb, int kk, u32x4* wn, u32x4* xn) {
#pragma unroll
      for (int i = 0; i < 16; ++i) {
        const int qa = i >> 3, qb = (i >> 2) & 1, m = i & 3;
#pragma unroll
        for (int t = 0; t < 4; ++t) MF((qa ^ qb) * 4 + t, qa * 4 + m, w_, x_);
        if (doload) load1(i, sb, kk, wn, xn);
      }
    };
    const unsigned foff = IF_OFF + (unsigned)(((lane & 16) ? ct : 8 + rt) * 64 + ((((wv - lane) & 3) + ((lane & 8) >> 1)) << 3) + (lane & 7)) * 4;
    unsigned todo = 0xffffu, fv = 0;
    auto pick = [&]() -> unsigned {
      u64 rd = __ballot(fv >= call);
      unsigned m = (unsigned)rd & (unsigned)(rd >> 16) & todo;
      if (m == 0) {
        u64 t0 = __builtin_amdgcn_s_memrealtime();
        for (;;) {
          const unsigned v = __hip_atomic_load((unsigned*)(myws + foff), __ATOMIC_RELAXED, __HIP_MEMORY_SCOPE_SYSTEM);
          rd = __ballot(v >= call);
          m = (unsigned)rd & (unsigned)(rd >> 16) & todo;
          if (m) break;
          if (__builtin_amdgcn_readfirstlane(timed_out(tmo, call, t0))) { m = todo; break; }
          __builtin_amdgcn_s_sleep(4);
        }
      }
      const unsigned j = __builtin_ctz(m);
      todo &= ~(1u << j);
      return ((((((unsigned)wv - j) & 3u) + ((j & 8u) >> 1)) << 3) + (j & 7u)) * KSB;
    };
#define LDF() asm volatile("buffer_load_dword %0, %1, %2, 0 offen sc0 sc1" : "=v"(fv) : "v"(foff), "s"(mrs))
    {
      const unsigned sb = pick();
      STAMP(2);
      asm volatile("s_waitcnt vmcnt(0)\n\ts_nop 4" ::: "memory");
      LDF();
#pragma unroll
      for (int i = 0; i < 16; ++i) load1(i, sb, 0, fw[0], fx[0]);
#pragma unroll
      for (int i = 0; i < 16; ++i) load1(i, sb, 1, fw[1], fx[1]);
    }
    for (int s = 0; s < 5; ++s) {
#pragma unroll
      for (int u = 0; u < 6; u += 2) {
        WAITF(fw[u % 3], fx[u % 3], 16);
        asm volatile("" : "+v"(fv));
        const unsigned sb = pick();
        asm volatile("s_nop 4");
        LDF();
        half(fw[u % 3], fx[u % 3], true, sb, 0, fw[(u + 2) % 3], fx[(u + 2) % 3]);
        WAITF(fw[(u + 1) % 3], fx[(u + 1) % 3], 17);
        half(fw[(u + 1) % 3], fx[(u + 1) % 3], true, sb, 1, fw[u % 3], fx[u % 3]);
      }
    }
    WAITF(fw[0], fx[0], 16);
    half(fw[0], fx[0], false, 0, 0, fw[0], fx[0]);
    WAITF(fw[1], fx[1], 0);
    half(fw[1], fx[1], false, 0, 0, fw[0], fx[0]);
    asm volatile("" : "+v"(fv));
    asm volatile("s_nop 15\n\ts_nop 15" ::: "memory");
    __builtin_amdgcn_sched_barrier(0);
  }
  {
    LDS_AS f32x4* ex = (LDS_AS f32x4*)lds;
#pragma unroll
    for (int t = 0; t < 4; ++t)
#pragma unroll
      for (int m = 0; m < 8; ++m) ex[(wv * 32 + t * 8 + m) * 64 + lane] = acc[4 + t][m];
    if (__hip_atomic_load(tmo, __ATOMIC_RELAXED, __HIP_MEMORY_SCOPE_AGENT) == call) cnt[4] = 1;
    __syncthreads();
    if (cnt[4]) {
      const u32x4 nanv = {0x7fc07fc0u, 0x7fc07fc0u, 0x7fc07fc0u, 0x7fc07fc0u};
#pragma unroll
      for (int c = 0; c < 8; ++c) *(u32x4*)(o + (size_t)(c >> 1) * 16 * DIM + (c & 1) * 32) = nanv;
      return;
    }
#pragma unroll
    for (int t = 0; t < 4; ++t)
#pragma unroll
      for (int m = 0; m < 8; ++m) acc[t][m] += ex[((wv ^ 1) * 32 + t * 8 + m) * 64 + lane];
    __syncthreads();
#pragma unroll
    for (int t = 0; t < 4; ++t)
#pragma unroll
      for (int m = 0; m < 4; ++m) ex[(64 + wv * 16 + t * 4 + m) * 64 + lane] = acc[t][4 + m];
    __syncthreads();
#pragma unroll
    for (int t = 0; t < 4; ++t)
#pragma unroll
      for (int m = 0; m < 4; ++m) acc[t][m] += ex[(64 + (wv ^ 2) * 16 + t * 4 + m) * 64 + lane];
  }
  STAMP(3);

  {
    LDS_AS char* seg = (LDS_AS char*)(lds + wv * SEG);
    u32x4 pk[8];
    unsigned E[2] = {0u, 0u};
#pragma unroll
    for (int c = 0; c < 8; ++c) {
      const int mt = c >> 1, pc = c & 1;
      float v[8];
#pragma unroll
      for (int e = 0; e < 8; ++e) {
        const float a = acc[e & 3][mt][2 * pc + (e >> 2)];
        v[e] = (a <= 0.f) ? 0.f : a;
      }
      pk[c] = u32x4{pack_bf16(v[0], v[1]), pack_bf16(v[2], v[3]), pack_bf16(v[4], v[5]), pack_bf16(v[6], v[7])};
      *(u32x4*)(o + (size_t)mt * 16 * DIM + pc * 32) = pk[c];
#pragma unroll
      for (int dd = 0; dd < 4; ++dd) {
        const unsigned t = pk[c][dd];
        unsigned nz;
        asm("v_pk_min_u16 %0, %1, 1 op_sel_hi:[1,0]" : "=v"(nz) : "v"(t));
        E[c >> 2] |= nz << ((c & 3) * 4 + dd);
      }
    }
    const unsigned cn = __builtin_popcount(E[0]) + __builtin_popcount(E[1]);
    const unsigned inc = wave_scan(cn);
    const unsigned total = __builtin_amdgcn_readlane(inc, 63);
    *(LDS_AS u32x2*)(seg + lane * 8) = u32x2{E[0], E[1]};
    unsigned wa = (unsigned)(size_t)seg + 512 + 2 * (inc - cn);
#pragma unroll
    for (int c = 0; c < 8; ++c)
#pragma unroll
      for (int dd = 0; dd < 4; ++dd)
        asm volatile("v_cmpx_ne_u16 vcc, 0, %1\n\t"
                     "ds_write_b16 %0, %1\n\t"
                     "v_add_u32 %0, 2, %0\n\t"
                     "s_mov_b64 exec, -1\n\t"
                     "v_cmpx_lt_u32 vcc, %2, %1\n\t"
                     "ds_write_b16_d16_hi %0, %1\n\t"
                     "v_add_u32 %0, 2, %0\n\t"
                     "s_mov_b64 exec, -1"
                     : "+v"(wa) : "v"(pk[c][dd]), "s"(0xffffu) : "vcc", "memory");
    asm volatile("s_waitcnt lgkmcnt(0)" ::: "memory");
    const unsigned n64 = (512 + total * 2 + 63) >> 6;
    const unsigned so = OB_OFF + (unsigned)(gb * 4 + wv) * SEG + lane * 16;
#pragma unroll
    for (int i = 0; i < 9; ++i) {
      const u32x4 v = *(LDS_AS u32x4*)(seg + i * 1024 + lane * 16);
      __builtin_amdgcn_raw_buffer_store_b128(v, mrs, (unsigned)(i * 64 + lane) < n64 * 4 ? so + i * 1024 : 0xF0000000u, 0, AUX_SYS);
    }
    STAMP(4);
    asm volatile("s_waitcnt vmcnt(0)" ::: "memory");
    u64 old = 0;
    if (lane == 0)
      old = __hip_atomic_fetch_add(ctl, (1ull << 32) | ((u64)n64 << (8 * wv)), __ATOMIC_RELAXED, __HIP_MEMORY_SCOPE_WORKGROUP);
    const unsigned olo = __builtin_amdgcn_readfirstlane((unsigned)old);
    const unsigned ohi = __builtin_amdgcn_readfirstlane((unsigned)(old >> 32));
    if (ohi == 3 && lane < NR && lane != rank) {
      const u64 f = ((u64)call << 32) | (olo + (n64 << (8 * wv)));
      __hip_atomic_store((u64*)(p.ws[lane] + OF_OFF + (unsigned)(rank * 128 + gb) * 64), f, __ATOMIC_RELAXED, __HIP_MEMORY_SCOPE_SYSTEM);
    }
    STAMP(5);
  }
}

}

extern "C" void kernel_launch(int rank, int nranks, ncclComm_t comm,
                              hipStream_t stream,
                              void* const* d_in, const int* in_sizes, int n_in,
                              void* d_out, int out_size,
                              const int* in_dtypes, int out_dtype,
                              void* const* d_out_all,
                              void* const* d_ws_all, size_t ws_bytes) {
  (void)comm; (void)in_sizes; (void)n_in; (void)out_size; (void)in_dtypes; (void)out_dtype; (void)d_out_all;
  if (nranks != NR || (unsigned)rank >= (unsigned)NR || ws_bytes < (size_t)WS_USED) {
    fprintf(stderr, "unsupported configuration\n");
    return;
  }
  Params p;
  p.x = (const unsigned short*)d_in[0];
  p.w = (const unsigned short*)d_in[1];
  p.out = (unsigned short*)d_out;
  for (int i = 0; i < NR; ++i) p.ws[i] = (char*)d_ws_all[i];
  p.rank = rank;
  p.order = make_order();
  gemm_ar_kernel<<<256, 256, 0, stream>>>(p);
  HIP_CHECK(hipGetLastError());
}
```

```cpp
#include <hip/hip_runtime.h>
#include <rccl/rccl.h>
#include <cstdio>
#include <cstdint>

#define HIP_CHECK(expr) do { hipError_t _e = (expr); if (_e != hipSuccess) { \
    fprintf(stderr, "HIP error %s at %s:%d\n", hipGetErrorString(_e), __FILE__, __LINE__); \
    return; } } while (0)

namespace {

typedef short bf16x8 __attribute__((ext_vector_type(8)));
typedef float f32x4 __attribute__((ext_vector_type(4)));
typedef unsigned u32x4 __attribute__((ext_vector_type(4)));
typedef unsigned u32x2 __attribute__((ext_vector_type(2)));
typedef unsigned long long u64;

constexpr int NR = 8;
constexpr int DIM = 4096;
constexpr int KSL = DIM / NR;
constexpr int RT = 16, CT = 8;
constexpr int NKS = 64;
constexpr unsigned KSB = 16384;
constexpr unsigned SEG = 9216;
constexpr unsigned SEG_TBL = 8704;
__host__ __device__ constexpr unsigned seg_bytes(unsigned n) { return ((2 * n + 3) & ~3u) + 4 * (128 - (n >> 5)); }

constexpr unsigned XS_OFF = 0;
constexpr unsigned WS_OFF = XS_OFF + RT * NKS * KSB;
constexpr unsigned SX_OFF = WS_OFF + CT * NKS * KSB;
constexpr unsigned SW_OFF = SX_OFF + RT * 8 * KSB;
constexpr unsigned OB_OFF = SW_OFF + 32 * 8 * KSB;
constexpr unsigned IF_OFF = OB_OFF + RT * CT * 4 * SEG;
constexpr unsigned PF_OFF = IF_OFF + 24 * 256;
constexpr unsigned OF_OFF = PF_OFF + NR * 192 * 4;
constexpr unsigned TK_OFF = OF_OFF + NR * RT * CT * 64;
constexpr unsigned QC_OFF = TK_OFF + 32 * 128;
constexpr unsigned TMO_OFF = QC_OFF + 16 * 128;
constexpr unsigned DBG_OFF = TMO_OFF + 4096;
constexpr unsigned WS_USED = DBG_OFF + 256 * 4 * 16 * 8;

#define DEPTH 4
#define ODEPTH 5
#define OUT_AUX 16
constexpr int NIN = 16;
constexpr int NSLOT = 12;

constexpr u64 DEADLINE_TICKS = 50000000ull;

constexpr int LDS_STAGE = 32768;
constexpr int LDS_CTRL = 4 * LDS_STAGE;
constexpr int LDS_BYTES = LDS_CTRL + 256;

#define KGAP 0
#define MU4 4
struct alignas(4) OrderTable { unsigned char t[192]; };
constexpr OrderTable make_order() {
  OrderTable o{};
  int n = 0;
  for (int key = 0; key <= 4 * 23 + MU4 * 7 + KGAP; ++key)
    for (int k = 7; k >= 0; --k)
      for (int s = 0; s < 24; ++s)
        if (4 * s + MU4 * k + (k >= 4 ? KGAP : 0) == key) o.t[n++] = (unsigned char)((s < 16 ? ((s & 1) ? (s >> 1) : 8 + (s >> 1)) : s) * 8 + k);
  return o;
}

struct Params {
  const unsigned short* x;
  const unsigned short* w;
  unsigned short* out;
  char* myws;
  char* ws[NR];
  int rank;
  OrderTable order;
};

#define LDS_AS __attribute__((address_space(3)))
#define GLB_AS __attribute__((address_space(1)))

__device__ __forceinline__ __amdgpu_buffer_rsrc_t make_rsrc(const void* p, unsigned bytes) {
  return __builtin_amdgcn_make_buffer_rsrc((void*)p, 0, bytes, 0x00020000);
}
constexpr int AUX_SYS = 17;

#define STAMP(i) do { } while (0)

__device__ __forceinline__ bool timed_out(unsigned* tmo, unsigned call, u64 t0) {
  if (__hip_atomic_load(tmo, __ATOMIC_RELAXED, __HIP_MEMORY_SCOPE_AGENT) == call ||
      __builtin_amdgcn_s_memrealtime() - t0 > DEADLINE_TICKS) {
    __hip_atomic_store(tmo, call, __ATOMIC_RELAXED, __HIP_MEMORY_SCOPE_AGENT);
    return true;
  }
  return false;
}

__device__ __forceinline__ unsigned pack_bf16(float a, float b) {
  typedef __bf16 bf2 __attribute__((ext_vector_type(2)));
  typedef float f2 __attribute__((ext_vector_type(2)));
  f2 v = {a, b};
  bf2 r = __builtin_convertvector(v, bf2);
  return __builtin_bit_cast(unsigned, r);
}

__device__ __forceinline__ unsigned wave_scan(unsigned x) {
  x += __builtin_amdgcn_update_dpp(0u, x, 0x111, 0xf, 0xf, false);
  x += __builtin_amdgcn_update_dpp(0u, x, 0x112, 0xf, 0xf, false);
  x += __builtin_amdgcn_update_dpp(0u, x, 0x114, 0xf, 0xf, false);
  x += __builtin_amdgcn_update_dpp(0u, x, 0x118, 0xf, 0xf, false);
  x += __builtin_amdgcn_update_dpp(0u, x, 0x142, 0xa, 0xf, false);
  x += __builtin_amdgcn_update_dpp(0u, x, 0x143, 0xc, 0xf, false);
  return x;
}
typedef unsigned short u16x2 __attribute__((ext_vector_type(2)));
typedef unsigned __attribute__((aligned(2))) u32_a2;

__device__ __forceinline__ int tile_of(int i) {
  if (i >= 64) return i;
  const int n = (i >= 1) + (i >= 4) + (i >= 9) + (i >= 16) + (i >= 25) + (i >= 36) + (i >= 49);
  const int rem = i - n * n;
  return rem < n ? n * 8 + rem : (rem - n) * 8 + n;
}

__global__ void __launch_bounds__(256, 1) gemm_ar_kernel(Params p) {
  __shared__ __attribute__((aligned(16))) char lds[LDS_BYTES];
  const int tid = threadIdx.x;
  const int lane = tid & 63;
  const int wv = __builtin_amdgcn_readfirstlane(tid >> 6);
  const int rank = p.rank;
  char* myws = p.myws;
  unsigned* tmo = (unsigned*)(myws + TMO_OFF);
  const int r16 = lane & 15, g4 = lane >> 4;
  STAMP(0);

  const int pi = rank >> 2, pj = rank & 3;
  __amdgpu_buffer_rsrc_t mrs = make_rsrc(myws, WS_USED);
  __amdgpu_buffer_rsrc_t ors = make_rsrc(p.out, (unsigned)DIM * DIM * 2);
  LDS_AS unsigned* cnt = (LDS_AS unsigned*)(lds + LDS_CTRL);
  unsigned call;
  {
    unsigned ticket = 1;
    asm volatile("buffer_atomic_add %0, %1, %2, 0 offen sc0" : "+v"(ticket) : "v"(lane == 0 ? TK_OFF + (blockIdx.x & 31) * 128 : 0xF0000000u), "s"(mrs) : "memory");
    char* const wsl = p.ws[lane & 7];
    const int cbr = 8 * ((r16 & 3) >> 1) + 2 * (r16 >> 2) + (r16 & 1);
    const unsigned short* wsrc = p.w + (size_t)((wv >> 1) * 32 + 8 * g4) * DIM + 64 * (wv & 1) + 4 * cbr;
    const unsigned short* xsrc = p.x + (size_t)(wv * 32 + r16) * KSL + 8 * g4;
    auto load_w = [&](u32x4* r, int t, int k) {
      const unsigned short* q = wsrc + (size_t)(k * 64) * DIM + t * 128;
#pragma unroll
      for (int h = 0; h < 4; ++h) {
        const u32x2 a = *(const u32x2*)(q + (size_t)(2 * h) * DIM), b = *(const u32x2*)(q + (size_t)(2 * h + 1) * DIM);
        r[h] = u32x4{a.x, a.y, b.x, b.y};
      }
    };
    auto load_x = [&](u32x4* r, int t, int k) {
      const unsigned short* q = xsrc + (size_t)(t * 128) * KSL + k * 64;
#pragma unroll
      for (int h = 0; h < 4; ++h) r[h] = *(const u32x4*)(q + (size_t)(h >> 1) * 16 * KSL + (h & 1) * 32);
    };
    auto transpose_w = [&](u32x4* r) {
      u32x4 v[4];
#pragma unroll
      for (int t = 0; t < 4; ++t)
#pragma unroll
        for (int h = 0; h < 4; ++h) {
          const unsigned lo = r[h][t >> 1], hi = r[h][2 + (t >> 1)];
          v[t][h] = __builtin_amdgcn_perm(hi, lo, (t & 1) ? 0x07060302u : 0x05040100u);
        }
#pragma unroll
      for (int t = 0; t < 4; ++t) r[t] = v[t];
    };
    auto store_pair = [&](u32x4* wr, u32x4* xr, int gt, int ksl) {
      const unsigned wo = ((gt >> 3) == pj ? WS_OFF + (unsigned)((gt & 7) * NKS + 56 + ksl) * KSB : SW_OFF + (unsigned)(gt * 8 + ksl) * KSB) + wv * 4096 + lane * 16;
      const unsigned xo = ((gt >> 4) == pi ? XS_OFF + (unsigned)((gt & 15) * NKS + 56 + ksl) * KSB : SX_OFF + (unsigned)((gt & 15) * 8 + ksl) * KSB) + wv * 4096 + lane * 16;
      transpose_w(wr);
#pragma unroll
      for (int t = 0; t < 4; ++t) __builtin_amdgcn_raw_buffer_store_b128(wr[t], mrs, wo + t * 1024, 0, AUX_SYS);
#pragma unroll
      for (int h = 0; h < 4; ++h) __builtin_amdgcn_raw_buffer_store_b128(xr[h], mrs, xo + h * 1024, 0, AUX_SYS);
    };
    auto publish_pair = [&](int ci, int gt, int ksl) {
      unsigned old = 0;
      if (lane == 0) old = __hip_atomic_fetch_add(cnt + ci, 1u, __ATOMIC_RELAXED, __HIP_MEMORY_SCOPE_WORKGROUP);
      old = __builtin_amdgcn_readfirstlane(old);
      const int q = lane & 7;
      const bool need = lane < 8 ? ((q & 3) == (gt >> 3)) : ((q >> 2) == (gt >> 4));
      if (old == 3 && lane < 16 && need) {
        const int unit = lane < 8 ? (gt & 7) : 8 + (gt & 15);
        unsigned* fl = q == rank ? (unsigned*)(myws + IF_OFF) + unit * 64 + 56 + ksl
                                 : (unsigned*)(wsl + PF_OFF) + rank * 192 + unit * 8 + ksl;
        __hip_atomic_store(fl, call, __ATOMIC_RELAXED, __HIP_MEMORY_SCOPE_SYSTEM);
      }
    };
    const int gt = blockIdx.x >> 3, ksl = blockIdx.x & 7;
    u32x4 wr[4], xr[4];
    load_w(wr, gt, ksl);
    load_x(xr, gt, ksl);
    if (tid < 64) cnt[tid] = 0;
    asm volatile("s_waitcnt lgkmcnt(0)" ::: "memory");
    __builtin_amdgcn_s_barrier();
    store_pair(wr, xr, gt, ksl);
    asm volatile("s_waitcnt vmcnt(0)" : "+v"(ticket) :: "memory");
    STAMP(13);
    call = (unsigned)__builtin_amdgcn_readfirstlane(ticket) / 32u + 1u;
    publish_pair(0, gt, ksl);
  }
  STAMP(1);

  if (blockIdx.x < 128) {
    const int mb = blockIdx.x;
    if (mb >= 112) return;
    const int bi = __builtin_amdgcn_readfirstlane((mb * 9363) >> 16);
    const int lk = __builtin_amdgcn_readfirstlane(mb - 7 * bi);
    const int src = (rank + 1 + lk) & 7;
    const int wi = bi;
    const unsigned l16 = lane * 16;
    const unsigned oob = 0xF0000000u;
    LDS_AS u64* rdesc = (LDS_AS u64*)(lds + LDS_CTRL + 64);
    LDS_AS unsigned* rfree = (LDS_AS unsigned*)(lds + LDS_CTRL + 160);
    LDS_AS unsigned* rland = (LDS_AS unsigned*)(lds + LDS_CTRL + 208);
#define LDS_LD(ptr) __hip_atomic_load(ptr, __ATOMIC_RELAXED, __HIP_MEMORY_SCOPE_WORKGROUP)
#define LDS_ST(ptr, val) __hip_atomic_store(ptr, val, __ATOMIC_RELAXED, __HIP_MEMORY_SCOPE_WORKGROUP)
    if (wv != 0) {
      const unsigned ob = (unsigned)(((src >> 2) * 2048 + r16) * DIM + (src & 3) * 1024 + 8 * g4) * 2;
      int slot = wv - 1;
      for (unsigned sq = wv - 1;; sq += 3) {
        u64 d;
        for (;;) {
          d = LDS_LD(rdesc + slot);
          if ((unsigned)(d >> 48) == sq + 1) break;
          __builtin_amdgcn_s_sleep(4);
        }
        const unsigned dhi = __builtin_amdgcn_readfirstlane((unsigned)(d >> 32)), dend = __builtin_amdgcn_readfirstlane((unsigned)d);
        const unsigned sg = dhi & 0x3ffu;
        if (sg >= 512) break;
        const unsigned nv = dend >> 16, H = nv >> 5;
        while (LDS_LD(rland) < (dend & 0xffffu)) __builtin_amdgcn_s_sleep(1);
        asm volatile("" ::: "memory");
        LDS_AS char* seg = (LDS_AS char*)(lds + 16 + slot * SEG);
        STAMP(4);
        auto unhide = [&](int i0, int n) {
          u32x4 v[5];
#pragma unroll
          for (int i = 0; i < n; ++i) v[i] = *(LDS_AS u32x4*)(seg + (i0 + i) * 1024 + l16);
#pragma unroll
          for (int i = 0; i < n; ++i) {
            unsigned e = ((v[i][0] >> 15) & 0x00010001u) | ((v[i][1] >> 14) & 0x00020002u) | ((v[i][2] >> 13) & 0x00040004u) | ((v[i][3] >> 12) & 0x00080008u);
            e <<= (lane & 3) * 4;
            e |= __builtin_amdgcn_update_dpp(0u, e, 0xb1, 0xf, 0xf, true);
            e |= __builtin_amdgcn_update_dpp(0u, e, 0x4e, 0xf, 0xf, true);
            *(LDS_AS unsigned*)(seg + SEG_TBL + (i0 + i) * 64 + (lane >> 2) * 4) = e;
          }
        };
        unhide(0, 5);
        if (H > 80) unhide(5, 3);
        const unsigned xo = ((2 * nv + 3) & ~3u) - 4 * H;
        u32x2 mk;
        mk.x = *(LDS_AS unsigned*)(seg + (2u * lane < H ? SEG_TBL : xo) + lane * 8);
        mk.y = *(LDS_AS unsigned*)(seg + (2u * lane + 1 < H ? SEG_TBL : xo) + lane * 8 + 4);
        const unsigned c0 = __builtin_popcount(mk.x), cn = c0 + __builtin_popcount(mk.y);
        const int b0 = 2 * (int)(wave_scan(cn) - cn) - 2, b1 = b0 + 2 * (int)c0;
        u32x4 ov[8], oh[8];
#pragma unroll
        for (int c = 0; c < 8; ++c)
#pragma unroll
          for (int dd = 0; dd < 4; ++dd) {
            const int dq = (c & 3) * 4 + dd;
            const unsigned E = c < 4 ? mk.x : mk.y;
            const unsigned pp = __builtin_popcount(E & (((2u << dq) - 1u) | (((1u << dq) - 1u) << 16)));
            const volatile LDS_AS unsigned short* q = (const volatile LDS_AS unsigned short*)(seg + (c < 4 ? b0 : b1) + 2 * (int)pp);
            ov[c][dd] = q[0];
            oh[c][dd] = q[1];
          }
        asm volatile("s_waitcnt lgkmcnt(0)" : "+v"(ov[0]), "+v"(ov[1]), "+v"(ov[2]), "+v"(ov[3]), "+v"(ov[4]), "+v"(ov[5]), "+v"(ov[6]), "+v"(ov[7]),
                     "+v"(oh[0]), "+v"(oh[1]), "+v"(oh[2]), "+v"(oh[3]), "+v"(oh[4]), "+v"(oh[5]), "+v"(oh[6]), "+v"(oh[7]) :: "memory");
        LDS_ST(rfree + slot, sq + 1);
        STAMP(6);
#pragma unroll
        for (int c = 0; c < 8; ++c)
#pragma unroll
          for (int dd = 0; dd < 4; ++dd) {
            const unsigned m = ((c < 4 ? mk.x : mk.y) >> ((c & 3) * 4 + dd)) & 0x00010001u;
            const unsigned t = (ov[c][dd] | (oh[c][dd] << 16)) & 0x7fff7fffu;
            const u16x2 r = __builtin_bit_cast(u16x2, t) * __builtin_bit_cast(u16x2, m);
            ov[c][dd] = __builtin_bit_cast(unsigned, r);
          }
        if (dhi & 0x8000u) {
#pragma unroll
          for (int c = 0; c < 8; ++c) ov[c] = u32x4{0x7fc07fc0u, 0x7fc07fc0u, 0x7fc07fc0u, 0x7fc07fc0u};
          asm volatile("" ::: "memory");
        }
        const int tile = sg >> 2, sw = sg & 3;
        const unsigned oo = ob + (unsigned)(((tile >> 3) * 128 + (sw >> 1) * 64) * DIM + (tile & 7) * 128 + 64 * (sw & 1)) * 2;
#pragma unroll
        for (int c = 0; c < 8; ++c) __builtin_amdgcn_raw_buffer_store_b128(ov[c], ors, oo + (c & 1) * 64, (c >> 1) * (16 * DIM * 2), OUT_AUX);
        STAMP(7);
        slot += 3;
        if (slot >= NSLOT) slot -= NSLOT;
      }
      STAMP(11);
      return;
    }
    __amdgpu_buffer_rsrc_t srs = make_rsrc(p.ws[src], WS_USED);
    bool bad = false;
    if (mb == 0 && tid < 7)
      __hip_atomic_store((unsigned*)(myws + QC_OFF + (((call + 1) & 1) * 8 + tid) * 128), 0u, __ATOMIC_RELAXED, __HIP_MEMORY_SCOPE_AGENT);
    {
      constexpr int NT = 192 / NIN;
      const unsigned mytask = p.order.t[wi + (lane < NT ? lane : 0) * NIN];
      unsigned* pfl = (unsigned*)(myws + PF_OFF) + src * 192 + mytask;
      u64 ready = 0;
      auto wait_ready = [&](int j) {
        if ((ready >> j) & 1) return;
        u64 t0 = __builtin_amdgcn_s_memrealtime();
        for (;;) {
          unsigned v = __hip_atomic_load(pfl, __ATOMIC_RELAXED, __HIP_MEMORY_SCOPE_SYSTEM);
          ready = __ballot(v >= call);
          if ((ready >> j) & 1) break;
          if (__builtin_amdgcn_readfirstlane(timed_out(tmo, call, t0))) { bad = true; ready = ~0ull; break; }
          __builtin_amdgcn_s_sleep(1);
        }
      };
      auto src_off = [&](int j) -> unsigned {
        if (j >= NT) return oob;
        const int tb = __builtin_amdgcn_readlane(mytask, j), unit = tb >> 3, ksl = tb & 7;
        if (unit < 8) return (src & 3) == pj ? WS_OFF + (unsigned)(unit * NKS + 56 + ksl) * KSB : SW_OFF + (unsigned)((pj * 8 + unit) * 8 + ksl) * KSB;
        return (src >> 2) == pi ? XS_OFF + (unsigned)((unit - 8) * NKS + 56 + ksl) * KSB : SX_OFF + (unsigned)((unit - 8) * 8 + ksl) * KSB;
      };
      unsigned* ifl = (unsigned*)(myws + IF_OFF) + lk * 8;
      u32x4 v[DEPTH];
#define LD(r, vo, so, imm) asm volatile("buffer_load_dwordx4 %0, %1, %2, 0 offen offset:%3 sc0 sc1" : "=v"(r) : "v"((vo) + (so)), "s"(srs), "i"(imm) : "memory")
#define ST(r, vo, so, imm) asm volatile("buffer_store_dwordx4 %0, %1, %2, 0 offen offset:%3 sc1\n\ts_nop 1" :: "v"(r), "v"((vo) + (so)), "s"(mrs), "i"(imm) : "memory")
      wait_ready(0);
      STAMP(2);
      unsigned so = src_off(0);
      asm volatile("s_nop 4" ::: "memory");
      {
        u32x4 z = {0, 0, 0, 0};
#pragma unroll
        for (int u = 0; u < DEPTH; ++u) { ST(z, l16, oob, 0); LD(v[u], l16, so + (u >> 2) * 4096, (u & 3) * 1024); }
      }
      int uprev = 0;
      for (int j = 0; j < NT; ++j) {
        const int tb = __builtin_amdgcn_readlane(mytask, j), unit = tb >> 3, ksl = tb & 7;
        const unsigned sn = src_off(j + 1);
        const unsigned dn = (unit < 8 ? WS_OFF + (unsigned)(unit * NKS) * KSB : XS_OFF + (unsigned)((unit - 8) * NKS) * KSB) + (unsigned)(lk * 8 + ksl) * KSB;
        asm volatile("s_nop 4" ::: "memory");
#pragma unroll
        for (int q = 0; q < 16; ++q) {
          asm volatile("s_waitcnt vmcnt(%1)" : "+v"(v[q % DEPTH]) : "i"(2 * (DEPTH - 1)) : "memory");
          if (q == DEPTH - 1 && j > 0 && lane == 0 && !bad)
            __hip_atomic_store(ifl + uprev, call, __ATOMIC_RELAXED, __HIP_MEMORY_SCOPE_AGENT);
          ST(v[q % DEPTH], l16, dn + (q >> 2) * 4096, (q & 3) * 1024);
          const int qn = q + DEPTH;
          if (qn == 16) wait_ready(j + 1);
          if (qn < 16) LD(v[q % DEPTH], l16, so + (qn >> 2) * 4096, (qn & 3) * 1024);
          else LD(v[q % DEPTH], l16, sn + ((qn - 16) >> 2) * 4096, (qn & 3) * 1024);
        }
        so = sn;
        uprev = unit * 64 + ksl;
      }
      asm volatile("s_waitcnt vmcnt(0)" : "+v"(v[0]) :: "memory");
#pragma unroll
      for (int u = 1; u < DEPTH; ++u) asm volatile("" : "+v"(v[u]) :: "memory");
      if (lane == 0 && !bad) __hip_atomic_store(ifl + uprev, call, __ATOMIC_RELAXED, __HIP_MEMORY_SCOPE_AGENT);
    }
    STAMP(3);

    {
      const unsigned qoff = lane == 0 ? QC_OFF + (unsigned)((call & 1) * 8 + lk) * 128 : oob;
      const unsigned ldsbase = (unsigned)(size_t)(LDS_AS char*)lds + 16;
      u32x2 fl = {0u, 0u};
      unsigned tk = 0;
      int k = 0, land = 0;
      unsigned sq = 0;
      int slot = 0;
      int cur = wi, nxt = wi + NIN;
      auto drain = [&]() {
        asm volatile("s_waitcnt vmcnt(0)" : "+v"(fl), "+v"(tk) :: "memory");
        land = k;
        LDS_ST(rland, (unsigned)k);
      };
      auto next_slot = [&]() {
        if (sq >= (unsigned)NSLOT && LDS_LD(rfree + slot) != sq - NSLOT + 1) {
          drain();
          while (LDS_LD(rfree + slot) != sq - NSLOT + 1) __builtin_amdgcn_s_sleep(1);
        }
      };
      unsigned flo = 0, fhi = 0;
      while (cur < 512) {
        const int tile = tile_of(cur >> 2), sw = cur & 3;
        if (fhi < call && !bad) {
          drain();
          u64* flag = (u64*)(myws + OF_OFF + (unsigned)(src * 128 + tile) * 64 + (sw >> 1) * 8);
          u64 t0 = __builtin_amdgcn_s_memrealtime();
          for (;;) {
            const u64 f = __hip_atomic_load(flag, __ATOMIC_RELAXED, __HIP_MEMORY_SCOPE_SYSTEM);
            flo = __builtin_amdgcn_readfirstlane((unsigned)f);
            fhi = __builtin_amdgcn_readfirstlane((unsigned)(f >> 32));
            if (fhi >= call) break;
            if (__builtin_amdgcn_readfirstlane(timed_out(tmo, call, t0))) { bad = true; break; }
            __builtin_amdgcn_s_sleep(2);
          }
        }
        if (bad) flo = 0;
        tk = 1;
        asm volatile("s_nop 4\n\tbuffer_atomic_add %0, %1, %2, 0 offen sc0" : "+v"(tk) : "v"(nxt < 512 ? qoff : oob), "s"(mrs) : "memory");
        asm volatile("buffer_load_dwordx2 %0, %1, %2, 0 offen sc0 sc1" : "=v"(fl) : "v"(nxt < 512 ? OF_OFF + (unsigned)(src * 128 + tile_of(nxt >> 2)) * 64 + (nxt & 2) * 4 : oob), "s"(mrs) : "memory");
        k += 2;
        next_slot();
        const unsigned nv = min((flo >> (14 * (sw & 1))) & 0x3fffu, 4096u);
        const unsigned nb = seg_bytes(nv);
        const unsigned np = (nb + 1023) >> 10;
        LDS_ST(rdesc + slot, ((u64)(((sq + 1) << 16) | (bad ? 0x8000u : 0u) | (unsigned)(tile * 4 + sw)) << 32) | (nv << 16) | (unsigned)(k + np));
        unsigned vo = OB_OFF + (unsigned)(tile * 4 + sw) * SEG + l16;
        const unsigned vend = OB_OFF + (unsigned)(tile * 4 + sw) * SEG + nb;
        unsigned la = ldsbase + slot * SEG;
        for (unsigned i = 0; i < np; ++i) {
          unsigned keep;
          asm volatile("s_waitcnt vmcnt(%0)" :: "i"(ODEPTH - 1) : "memory");
          land = max(land, k - (ODEPTH - 1));
          LDS_ST(rland, (unsigned)land);
          asm volatile("s_mov_b32 %0, m0\n\ts_mov_b32 m0, %3\n\ts_nop 0\n\tbuffer_load_dwordx4 %1, %2, 0 offen sc0 sc1 lds\n\ts_mov_b32 m0, %0"
                       : "=&s"(keep) : "v"(vo < vend ? vo : oob), "s"(srs), "s"(la) : "memory");
          vo += 1024; la += 1024; ++k;
        }
        ++sq;
        if (++slot == NSLOT) slot = 0;
        if (np >= 5) asm volatile("s_waitcnt vmcnt(5)" ::: "memory");
        else if (np == 4) asm volatile("s_waitcnt vmcnt(4)" ::: "memory");
        else asm volatile("s_waitcnt vmcnt(1)" ::: "memory");
        asm volatile("" : "+v"(fl), "+v"(tk) :: "memory");
        flo = __builtin_amdgcn_readfirstlane(fl.x);
        fhi = __builtin_amdgcn_readfirstlane(fl.y);
        const unsigned tku = (unsigned)__builtin_amdgcn_readfirstlane(tk);
        const int nn = (nxt < 512 && tku < 512u) ? 2 * NIN + (int)tku : 512;
        cur = nxt;
        nxt = nn;
      }
      STAMP(10);
      drain();
      for (int e = 0; e < 3; ++e) {
        next_slot();
        LDS_ST(rdesc + slot, (u64)(((sq + 1) << 16) | 0x3ffu) << 32);
        ++sq;
        if (++slot == NSLOT) slot = 0;
      }
    }
    if (__hip_atomic_load(tmo, __ATOMIC_RELAXED, __HIP_MEMORY_SCOPE_AGENT) == call && lane == 0)
      p.out[(size_t)((src >> 2) * 2048) * DIM + (src & 3) * 1024 + wi] = 0x7fc0;
    return;
  }

  const int gb = blockIdx.x - 128;
  const int ct = gb & 7, rt = gb >> 3;
  const int wm = wv >> 1, hf = wv & 1;
  LDS_AS u64* ctl = (LDS_AS u64*)(lds + LDS_CTRL + 32);
  unsigned short* o = p.out + (size_t)((rank >> 2) * 2048 + rt * 128 + wm * 64 + r16) * DIM + (rank & 3) * 1024 + ct * 128 + 64 * hf + 8 * g4;

  f32x4 acc[8][8];
  {
    const unsigned bx = XS_OFF + (unsigned)rt * (NKS * KSB) + lane * 16, bw = WS_OFF + (unsigned)ct * (NKS * KSB) + lane * 16;
    const unsigned vx0 = bx + wm * 8192, vx1 = bx + (wm ^ 1) * 8192;
    const unsigned vw0 = bw + hf * 4096, vw1 = bw + (hf ^ 1) * 4096;
#pragma unroll
    for (int a = 0; a < 8; ++a)
#pragma unroll
      for (int b = 0; b < 8; ++b) acc[a][b] = f32x4{0.f, 0.f, 0.f, 0.f};
    u32x4 fw[3][8], fx[3][8];
#define LDW(dst, vo, so, imm) asm volatile("buffer_load_dwordx4 %0, %1, %2, %3 offen offset:%4 sc1" : "=v"(dst) : "v"(vo), "s"(mrs), "s"(so), "i"(imm))
#define LDX(dst, vo, so, imm) asm volatile("buffer_load_dwordx4 %0, %1, %2, %3 offen offset:%4 sc1" : "=v"(dst) : "v"(vo), "s"(mrs), "s"(so), "i"(imm))
#define MF(T, M, w_, x_) asm volatile("v_mfma_f32_16x16x32_bf16 %0, %1, %2, %0" : "+a"(acc[T][M]) : "v"(w_[T]), "v"(x_[M]))
#define WAITF(w_, x_, n) asm volatile("s_waitcnt vmcnt(" #n ")" : "+v"(w_[0]), "+v"(w_[1]), "+v"(w_[2]), "+v"(w_[3]), "+v"(w_[4]), "+v"(w_[5]), "+v"(w_[6]), "+v"(w_[7]), \
                   "+v"(x_[0]), "+v"(x_[1]), "+v"(x_[2]), "+v"(x_[3]), "+v"(x_[4]), "+v"(x_[5]), "+v"(x_[6]), "+v"(x_[7]))
    auto load1 = [&](int i, unsigned sb, int kk, u32x4* w_, u32x4* x_) {
      const int q = i >> 2, j = i & 3;
      if (q == 0) LDW(w_[j], vw0, sb + kk * 8192, j * 1024);
      else if (q == 1) LDX(x_[j], vx0, sb + (j >> 1) * 4096, (j & 1) * 2048 + kk * 1024);
      else if (q == 2) LDW(w_[4 + j], vw1, sb + kk * 8192, j * 1024);
      else LDX(x_[4 + j], vx1, sb + (j >> 1) * 4096, (j & 1) * 2048 + kk * 1024);
    };
    auto half = [&](u32x4* w_, u32x4* x_, bool doload, unsigned sb, int kk, u32x4* wn, u32x4* xn) {
#pragma unroll
      for (int i = 0; i < 16; ++i) {
        const int qa = i >> 3, qb = (i >> 2) & 1, m = i & 3;
#pragma unroll
        for (int t = 0; t < 4; ++t) MF((qa ^ qb) * 4 + t, qa * 4 + m, w_, x_);
        if (doload) load1(i, sb, kk, wn, xn);
      }
    };
    const unsigned foff = IF_OFF + (unsigned)(((lane & 16) ? ct : 8 + rt) * 64 + ((((wv - lane) & 3) + ((lane & 8) >> 1)) << 3) + (lane & 7)) * 4;
    unsigned todo = 0xffffu, fv = 0;
    auto pick = [&]() -> unsigned {
      u64 rd = __ballot(fv >= call);
      unsigned m = (unsigned)rd & (unsigned)(rd >> 16) & todo;
      if (m == 0) {
        u64 t0 = __builtin_amdgcn_s_memrealtime();
        for (;;) {
          const unsigned v = __hip_atomic_load((unsigned*)(myws + foff), __ATOMIC_RELAXED, __HIP_MEMORY_SCOPE_SYSTEM);
          rd = __ballot(v >= call);
          m = (unsigned)rd & (unsigned)(rd >> 16) & todo;
          if (m) break;
          if (__builtin_amdgcn_readfirstlane(timed_out(tmo, call, t0))) { m = todo; break; }
          __builtin_amdgcn_s_sleep(4);
        }
      }
      const unsigned j = __builtin_ctz(m);
      todo &= ~(1u << j);
      return ((((((unsigned)wv - j) & 3u) + ((j & 8u) >> 1)) << 3) + (j & 7u)) * KSB;
    };
#define LDF() asm volatile("buffer_load_dword %0, %1, %2, 0 offen sc0 sc1" : "=v"(fv) : "v"(foff), "s"(mrs))
    {
      const unsigned sb = pick();
      STAMP(2);
      asm volatile("s_waitcnt vmcnt(0)\n\ts_nop 4" ::: "memory");
      LDF();
#pragma unroll
      for (int i = 0; i < 16; ++i) load1(i, sb, 0, fw[0], fx[0]);
#pragma unroll
      for (int i = 0; i < 16; ++i) load1(i, sb, 1, fw[1], fx[1]);
    }
    for (int s = 0; s < 5; ++s) {
#pragma unroll
      for (int u = 0; u < 6; u += 2) {
        WAITF(fw[u % 3], fx[u % 3], 16);
        asm volatile("" : "+v"(fv));
        const unsigned sb = pick();
        asm volatile("s_nop 4");
        LDF();
        half(fw[u % 3], fx[u % 3], true, sb, 0, fw[(u + 2) % 3], fx[(u + 2) % 3]);
        WAITF(fw[(u + 1) % 3], fx[(u + 1) % 3], 17);
        half(fw[(u + 1) % 3], fx[(u + 1) % 3], true, sb, 1, fw[u % 3], fx[u % 3]);
      }
    }
    WAITF(fw[0], fx[0], 16);
    half(fw[0], fx[0], false, 0, 0, fw[0], fx[0]);
    WAITF(fw[1], fx[1], 0);
    half(fw[1], fx[1], false, 0, 0, fw[0], fx[0]);
    asm volatile("" : "+v"(fv));
    asm volatile("s_nop 15\n\ts_nop 15" ::: "memory");
    __builtin_amdgcn_sched_barrier(0);
  }
  {
    LDS_AS f32x4* ex = (LDS_AS f32x4*)lds;
#pragma unroll
    for (int t = 0; t < 4; ++t)
#pragma unroll
      for (int m = 0; m < 8; ++m) ex[(wv * 32 + t * 8 + m) * 64 + lane] = acc[4 + t][m];
    if (__hip_atomic_load(tmo, __ATOMIC_RELAXED, __HIP_MEMORY_SCOPE_AGENT) == call) cnt[4] = 1;
    __syncthreads();
    if (cnt[4]) {
      const u32x4 nanv = {0x7fc07fc0u, 0x7fc07fc0u, 0x7fc07fc0u, 0x7fc07fc0u};
#pragma unroll
      for (int c = 0; c < 8; ++c) *(u32x4*)(o + (size_t)(c >> 1) * 16 * DIM + (c & 1) * 32) = nanv;
      return;
    }
#pragma unroll
    for (int t = 0; t < 4; ++t)
#pragma unroll
      for (int m = 0; m < 8; ++m) acc[t][m] += ex[((wv ^ 1) * 32 + t * 8 + m) * 64 + lane];
    __syncthreads();
#pragma unroll
    for (int t = 0; t < 4; ++t)
#pragma unroll
      for (int m = 0; m < 4; ++m) ex[(64 + wv * 16 + t * 4 + m) * 64 + lane] = acc[t][4 + m];
    __syncthreads();
#pragma unroll
    for (int t = 0; t < 4; ++t)
#pragma unroll
      for (int m = 0; m < 4; ++m) acc[t][m] += ex[(64 + (wv ^ 2) * 16 + t * 4 + m) * 64 + lane];
  }
  STAMP(3);

  {
    LDS_AS char* seg = (LDS_AS char*)(lds + wv * SEG);
    u32x4 pk[8];
    unsigned E[2] = {0u, 0u};
#pragma unroll
    for (int c = 0; c < 8; ++c) {
      const int mt = c >> 1, pc = c & 1;
      float v[8];
#pragma unroll
      for (int e = 0; e < 8; ++e) {
        const float a = acc[e & 3][mt][2 * pc + (e >> 2)];
        v[e] = (a <= 0.f) ? 0.f : a;
      }
      pk[c] = u32x4{pack_bf16(v[0], v[1]), pack_bf16(v[2], v[3]), pack_bf16(v[4], v[5]), pack_bf16(v[6], v[7])};
      pk[c] &= 0x7fff7fffu;
      __builtin_amdgcn_raw_buffer_store_b128(pk[c], ors, (unsigned)((char*)o - (char*)p.out) + pc * 64, mt * (16 * DIM * 2), OUT_AUX);
#pragma unroll
      for (int dd = 0; dd < 4; ++dd) {
        const unsigned t = pk[c][dd];
        unsigned nz;
        asm("v_pk_min_u16 %0, %1, 1 op_sel_hi:[1,0]" : "=v"(nz) : "v"(t));
        E[c >> 2] |= nz << ((c & 3) * 4 + dd);
      }
    }
    const unsigned cn = __builtin_popcount(E[0]) + __builtin_popcount(E[1]);
    const unsigned inc = wave_scan(cn);
    const unsigned total = __builtin_amdgcn_readlane(inc, 63);
    const unsigned H = total >> 5;
    {
      const unsigned xo = ((2 * total + 3) & ~3u) - 4 * H;
      *(LDS_AS u32x2*)(seg + SEG_TBL + lane * 8) = u32x2{2u * lane < H ? E[0] : 0u, 2u * lane + 1 < H ? E[1] : 0u};
      if (lane == 63) *(LDS_AS unsigned short*)(seg + 2 * total) = 0;
      if (2u * lane >= H) *(LDS_AS unsigned*)(seg + xo + lane * 8) = E[0];
      if (2u * lane + 1 >= H) *(LDS_AS unsigned*)(seg + xo + lane * 8 + 4) = E[1];
    }
    unsigned wa = (unsigned)(size_t)seg + 2 * (inc - cn);
#pragma unroll
    for (int c = 0; c < 8; ++c)
#pragma unroll
      for (int dd = 0; dd < 4; ++dd)
        asm volatile("v_cmpx_ne_u16 vcc, 0, %1\n\t"
                     "ds_write_b16 %0, %1\n\t"
                     "v_add_u32 %0, 2, %0\n\t"
                     "s_mov_b64 exec, -1\n\t"
                     "v_cmpx_lt_u32 vcc, %2, %1\n\t"
                     "ds_write_b16_d16_hi %0, %1\n\t"
                     "v_add_u32 %0, 2, %0\n\t"
                     "s_mov_b64 exec, -1"
                     : "+v"(wa) : "v"(pk[c][dd]), "s"(0xffffu) : "vcc", "memory");
    asm volatile("s_waitcnt lgkmcnt(0)" ::: "memory");
    const unsigned nb = seg_bytes(total);
    const unsigned so = OB_OFF + (unsigned)(gb * 4 + wv) * SEG + lane * 16;
#pragma unroll
    for (int i = 0; i < 9; ++i) {
      u32x4 v = *(LDS_AS u32x4*)(seg + i * 1024 + lane * 16);
      if (i < 8) {
        const unsigned e = *(LDS_AS unsigned*)(seg + SEG_TBL + i * 64 + (lane >> 2) * 4) >> ((lane & 3) * 4);
#pragma unroll
        for (int d = 0; d < 4; ++d) v[d] |= (e << (15 - d)) & 0x80008000u;
      }
      __builtin_amdgcn_raw_buffer_store_b128(v, mrs, (unsigned)(i * 1024 + lane * 16) < nb ? so + i * 1024 : 0xF0000000u, 0, AUX_SYS);
    }
    STAMP(4);
    asm volatile("s_waitcnt vmcnt(0)" ::: "memory");
    u64 old = 0;
    const u64 mine = (1ull << 56) | ((u64)total << (14 * wv));
    if (lane == 0) old = __hip_atomic_fetch_add(ctl, mine, __ATOMIC_RELAXED, __HIP_MEMORY_SCOPE_WORKGROUP);
    old = (((u64)(unsigned)__builtin_amdgcn_readfirstlane((unsigned)(old >> 32)) << 32) | (unsigned)__builtin_amdgcn_readfirstlane((unsigned)old)) + mine;
    if ((old >> 56) == 4 && lane < 16 && (lane & 7) != rank) {
      const u64 f = ((u64)call << 32) | (unsigned)((old >> (28 * (lane >> 3))) & 0xfffffffu);
      __hip_atomic_store((u64*)(p.ws[lane & 7] + OF_OFF + (unsigned)(rank * 128 + gb) * 64 + (lane >> 3) * 8), f, __ATOMIC_RELAXED, __HIP_MEMORY_SCOPE_SYSTEM);
    }
    STAMP(5);
  }
}

}

extern "C" void kernel_launch(int rank, int nranks, ncclComm_t comm,
                              hipStream_t stream,
                              void* const* d_in, const int* in_sizes, int n_in,
                              void* d_out, int out_size,
                              const int* in_dtypes, int out_dtype,
                              void* const* d_out_all,
                              void* const* d_ws_all, size_t ws_bytes) {
  (void)comm; (void)in_sizes; (void)n_in; (void)out_size; (void)in_dtypes; (void)out_dtype; (void)d_out_all;
  if (nranks != NR || (unsigned)rank >= (unsigned)NR || ws_bytes < (size_t)WS_USED) {
    fprintf(stderr, "unsupported configuration\n");
    return;
  }
  Params p;
  p.x = (const unsigned short*)d_in[0];
  p.w = (const unsigned short*)d_in[1];
  p.out = (unsigned short*)d_out;
  for (int i = 0; i < NR; ++i) p.ws[i] = (char*)d_ws_all[i];
  p.myws = p.ws[rank];
  p.rank = rank;
  p.order = make_order();
  gemm_ar_kernel<<<256, 256, 0, stream>>>(p);
  HIP_CHECK(hipGetLastError());
}
```

```cpp
#include <hip/hip_runtime.h>
#include <rccl/rccl.h>
#include <cstdio>
#include <cstdint>

#define HIP_CHECK(expr) do { hipError_t _e = (expr); if (_e != hipSuccess) { \
    fprintf(stderr, "HIP error %s at %s:%d\n", hipGetErrorString(_e), __FILE__, __LINE__); \
    return; } } while (0)

namespace {

typedef short bf16x8 __attribute__((ext_vector_type(8)));
typedef float f32x4 __attribute__((ext_vector_type(4)));
typedef unsigned u32x4 __attribute__((ext_vector_type(4)));
typedef unsigned u32x2 __attribute__((ext_vector_type(2)));
typedef unsigned long long u64;

constexpr int NR = 8;
constexpr int DIM = 4096;
constexpr int KSL = DIM / NR;
constexpr int RT = 16, CT = 8;
constexpr int NKS = 64;
constexpr unsigned KSB = 16384;
constexpr unsigned SEG = 9216;
constexpr unsigned SEG_TBL = 8704;
__host__ __device__ constexpr unsigned seg_bytes(unsigned n) { return ((2 * n + 3) & ~3u) + 4 * (128 - (n >> 5)); }

constexpr unsigned XS_OFF = 0;
constexpr unsigned WS_OFF = XS_OFF + RT * NKS * KSB;
constexpr unsigned SX_OFF = WS_OFF + CT * NKS * KSB;
constexpr unsigned SW_OFF = SX_OFF + RT * 8 * KSB;
constexpr unsigned OB_OFF = SW_OFF + 32 * 8 * KSB;
constexpr unsigned IF_OFF = OB_OFF + RT * CT * 4 * SEG;
constexpr unsigned PF_OFF = IF_OFF + 24 * 256;
constexpr unsigned OF_OFF = PF_OFF + NR * 192 * 4;
constexpr unsigned TK_OFF = OF_OFF + NR * RT * CT * 64;
constexpr unsigned QC_OFF = TK_OFF + 32 * 128;
constexpr unsigned TMO_OFF = QC_OFF + 16 * 128;
constexpr unsigned DBG_OFF = TMO_OFF + 4096;
constexpr unsigned WS_USED = DBG_OFF + 256 * 4 * 16 * 8;

#define DEPTH 4
#define ODEPTH 5
#define OUT_AUX 16
#define NINV 16
constexpr int NIN = NINV;
constexpr int NSLOT = 12;

constexpr u64 DEADLINE_TICKS = 50000000ull;

constexpr int LDS_STAGE = 32768;
constexpr int LDS_CTRL = 4 * LDS_STAGE;
constexpr int LDS_BYTES = LDS_CTRL + 256;

#define KGAP 0
#define MU4 4
struct alignas(4) OrderTable { unsigned char t[192]; };
constexpr OrderTable make_order() {
  OrderTable o{};
  int n = 0;
  for (int key = 0; key <= 4 * 23 + MU4 * 7 + KGAP; ++key)
    for (int k = 7; k >= 0; --k)
      for (int s = 0; s < 24; ++s)
        if (4 * s + MU4 * k + (k >= 4 ? KGAP : 0) == key) o.t[n++] = (unsigned char)((s < 16 ? ((s & 1) ? (s >> 1) : 8 + (s >> 1)) : s) * 8 + k);
  return o;
}

struct alignas(4) PrepTable { unsigned char t[256]; };
constexpr PrepTable make_prep() {
  PrepTable o{};
  int n = 0;
  for (int key = 0; key <= 4 * 23 + MU4 * 7 + KGAP; ++key)
    for (int gt = 0; gt < 32; ++gt)
      for (int k = 0; k < 8; ++k) {
        const int x = gt & 15, sw = 2 * (gt & 7) + 1, sx = x < 8 ? 2 * x : x + 8;
        if (4 * (sw < sx ? sw : sx) + MU4 * k + (k >= 4 ? KGAP : 0) == key) o.t[n++] = (unsigned char)(gt * 8 + k);
      }
  return o;
}
#define PSTEP 16
#define PK 2
constexpr int NPUSH = (PK + 1) * (PK + 2) / 2;
#define NEARLY 48
#define PDELAY 40

struct Params {
  const unsigned short* x;
  const unsigned short* w;
  unsigned short* out;
  char* myws;
  char* ws[NR];
  int rank;
  OrderTable order;
  PrepTable prep;
};

#define LDS_AS __attribute__((address_space(3)))
#define GLB_AS __attribute__((address_space(1)))

__device__ __forceinline__ __amdgpu_buffer_rsrc_t make_rsrc(const void* p, unsigned bytes) {
  return __builtin_amdgcn_make_buffer_rsrc((void*)p, 0, bytes, 0x00020000);
}
constexpr int AUX_SYS = 17;

#define STAMP(i) do { } while (0)
#define TMARK(v) do { } while (0)
#define TSTORE(i, v) do { } while (0)

__device__ __forceinline__ bool timed_out(unsigned* tmo, unsigned call, u64 t0) {
  if (__hip_atomic_load(tmo, __ATOMIC_RELAXED, __HIP_MEMORY_SCOPE_AGENT) == call ||
      __builtin_amdgcn_s_memrealtime() - t0 > DEADLINE_TICKS) {
    __hip_atomic_store(tmo, call, __ATOMIC_RELAXED, __HIP_MEMORY_SCOPE_AGENT);
    return true;
  }
  return false;
}

__device__ __forceinline__ unsigned pack_bf16(float a, float b) {
  typedef __bf16 bf2 __attribute__((ext_vector_type(2)));
  typedef float f2 __attribute__((ext_vector_type(2)));
  f2 v = {a, b};
  bf2 r = __builtin_convertvector(v, bf2);
  return __builtin_bit_cast(unsigned, r);
}

__device__ __forceinline__ unsigned wave_scan(unsigned x) {
  x += __builtin_amdgcn_update_dpp(0u, x, 0x111, 0xf, 0xf, false);
  x += __builtin_amdgcn_update_dpp(0u, x, 0x112, 0xf, 0xf, false);
  x += __builtin_amdgcn_update_dpp(0u, x, 0x114, 0xf, 0xf, false);
  x += __builtin_amdgcn_update_dpp(0u, x, 0x118, 0xf, 0xf, false);
  x += __builtin_amdgcn_update_dpp(0u, x, 0x142, 0xa, 0xf, false);
  x += __builtin_amdgcn_update_dpp(0u, x, 0x143, 0xc, 0xf, false);
  return x;
}
typedef unsigned short u16x2 __attribute__((ext_vector_type(2)));
typedef unsigned __attribute__((aligned(2))) u32_a2;

__device__ __forceinline__ int tile_of(int i) {
  if (i >= 64) return i;
  const int n = (i >= 1) + (i >= 4) + (i >= 9) + (i >= 16) + (i >= 25) + (i >= 36) + (i >= 49);
  const int rem = i - n * n;
  return rem < n ? n * 8 + rem : (rem - n) * 8 + n;
}

__global__ void __launch_bounds__(256, 1) gemm_ar_kernel(Params p) {
  __shared__ __attribute__((aligned(16))) char lds[LDS_BYTES];
  TMARK(tm0);
  const int tid = threadIdx.x;
  const int lane = tid & 63;
  const int wv = __builtin_amdgcn_readfirstlane(tid >> 6);
  const int rank = p.rank;
  char* myws = p.myws;
  unsigned* tmo = (unsigned*)(myws + TMO_OFF);
  const int r16 = lane & 15, g4 = lane >> 4;

  const int pi = rank >> 2, pj = rank & 3;
  __amdgpu_buffer_rsrc_t mrs = make_rsrc(myws, WS_USED);
  __amdgpu_buffer_rsrc_t ors = make_rsrc(p.out, (unsigned)DIM * DIM * 2);
  LDS_AS unsigned* cnt = (LDS_AS unsigned*)(lds + LDS_CTRL);
  unsigned call;
  {
    const unsigned job = blockIdx.x ^ 128u;
    const unsigned jobw = ((const unsigned*)p.prep.t)[job >> 2];
    const unsigned short* const px = p.x;
    const unsigned short* const pw = p.w;
    asm volatile("" :: "s"(px), "s"(pw), "s"(jobw), "s"(rank), "s"(myws));
    unsigned ticket = 1;
    asm volatile("buffer_atomic_add %0, %1, %2, 0 offen sc0" : "+v"(ticket) : "v"(lane == 0 ? TK_OFF + (blockIdx.x & 31) * 128 : 0xF0000000u), "s"(mrs) : "memory");
    char* wsl = p.ws[0];
#pragma unroll
    for (int i = 1; i < NR; ++i) wsl = (lane & 7) == i ? p.ws[i] : wsl;
    const int cbr = 8 * ((r16 & 3) >> 1) + 2 * (r16 >> 2) + (r16 & 1);
    auto load_w = [&](u32x4* r, int t, int k, int qt) {
      const unsigned short* q = pw + (size_t)(k * 64 + (qt >> 1) * 32 + 8 * g4) * DIM + t * 128 + 64 * (qt & 1) + 4 * cbr;
#pragma unroll
      for (int h = 0; h < 4; ++h) {
        const u32x2 a = *(const u32x2*)(q + (size_t)(2 * h) * DIM), b = *(const u32x2*)(q + (size_t)(2 * h + 1) * DIM);
        r[h] = u32x4{a.x, a.y, b.x, b.y};
      }
    };
    auto load_x = [&](u32x4* r, int t, int k, int qt) {
      const unsigned short* q = px + (size_t)(t * 128 + qt * 32 + r16) * KSL + k * 64 + 8 * g4;
#pragma unroll
      for (int h = 0; h < 4; ++h) r[h] = *(const u32x4*)(q + (size_t)(h >> 1) * 16 * KSL + (h & 1) * 32);
    };
    auto transpose_w = [&](u32x4* r) {
      u32x4 v[4];
#pragma unroll
      for (int t = 0; t < 4; ++t)
#pragma unroll
        for (int h = 0; h < 4; ++h) {
          const unsigned lo = r[h][t >> 1], hi = r[h][2 + (t >> 1)];
          v[t][h] = __builtin_amdgcn_perm(hi, lo, (t & 1) ? 0x07060302u : 0x05040100u);
        }
#pragma unroll
      for (int t = 0; t < 4; ++t) r[t] = v[t];
    };
    auto store_pair = [&](u32x4* wr, u32x4* xr, int gt, int ksl, int qt) {
      const unsigned wo = ((gt >> 3) == pj ? WS_OFF + (unsigned)((gt & 7) * NKS + 56 + ksl) * KSB : SW_OFF + (unsigned)(gt * 8 + ksl) * KSB) + qt * 4096 + lane * 16;
      const unsigned xo = ((gt >> 4) == pi ? XS_OFF + (unsigned)((gt & 15) * NKS + 56 + ksl) * KSB : SX_OFF + (unsigned)((gt & 15) * 8 + ksl) * KSB) + qt * 4096 + lane * 16;
      transpose_w(wr);
#pragma unroll
      for (int t = 0; t < 4; ++t) __builtin_amdgcn_raw_buffer_store_b128(wr[t], mrs, wo + t * 1024, 0, AUX_SYS);
#pragma unroll
      for (int h = 0; h < 4; ++h) __builtin_amdgcn_raw_buffer_store_b128(xr[h], mrs, xo + h * 1024, 0, AUX_SYS);
    };
    auto publish_pair = [&](int ci, int gt, int ksl) {
      unsigned old = 0;
      if (lane == 0) old = __hip_atomic_fetch_add(cnt + ci, 1u, __ATOMIC_RELAXED, __HIP_MEMORY_SCOPE_WORKGROUP);
      old = __builtin_amdgcn_readfirstlane(old);
      const int q = lane & 7;
      const bool need = lane < 8 ? ((q & 3) == (gt >> 3)) : ((q >> 2) == (gt >> 4));
      if (old == 3 && lane < 16 && need) {
        const int unit = lane < 8 ? (gt & 7) : 8 + (gt & 15);
        unsigned* fl = q == rank ? (unsigned*)(myws + IF_OFF) + unit * 64 + 56 + ksl
                                 : (unsigned*)(wsl + PF_OFF) + rank * 192 + unit * 8 + ksl;
        __hip_atomic_store(fl, call, __ATOMIC_RELAXED, __HIP_MEMORY_SCOPE_SYSTEM);
      }
    };
    const int jb = (jobw >> (8 * (job & 3))) & 255, gt = jb >> 3, ksl = jb & 7;
    const bool mover = blockIdx.x < 7 * NIN;
    const bool puller = mover && wv == 0;
    const bool pushw = 2 * (gt & 7) + 1 + ksl <= PK, pushx = 2 * (gt & 15) + ksl <= PK;
    if (tid < 64) cnt[tid] = 0;
    asm volatile("s_waitcnt lgkmcnt(0)" ::: "memory");
    __builtin_amdgcn_s_barrier();
    TMARK(tm12);
    TMARK(tm9);
    if (!puller) {
      if (job >= NEARLY) {
        __builtin_amdgcn_s_sleep(PDELAY);
        for (unsigned i = NEARLY; i < job; i += 8) __builtin_amdgcn_s_sleep(PSTEP);
      }
      u32x4 wr[4], xr[4];
      load_w(wr, gt, ksl, wv);
      load_x(xr, gt, ksl, wv);
      store_pair(wr, xr, gt, ksl, wv);
      if (pushw || pushx) {
        asm volatile("" ::: "memory");
#pragma unroll
        for (int i = 0; i < 6; ++i) {
          const int q = i < 2 ? (gt >> 3) + 4 * i : (gt >> 4) * 4 + i - 2;
          const bool on = (i < 2 ? pushw : pushx) && q != rank;
          __amdgpu_buffer_rsrc_t prs = make_rsrc(p.ws[q], WS_USED);
          const unsigned po = (i < 2 ? WS_OFF + (unsigned)((gt & 7) * NKS) * KSB : XS_OFF + (unsigned)((gt & 15) * NKS) * KSB)
                              + (unsigned)(((rank - q - 1) & 7) * 8 + ksl) * KSB + wv * 4096 + lane * 16;
#pragma unroll
          for (int h = 0; h < 4; ++h) __builtin_amdgcn_raw_buffer_store_b128(i < 2 ? wr[h] : xr[h], prs, on ? po + h * 1024 : 0xF0000000u, 0, AUX_SYS);
        }
      }
    }
    if (pushw || pushx) asm volatile("s_waitcnt vmcnt(24)" ::: "memory");
    else asm volatile("s_waitcnt vmcnt(0)" ::: "memory");
    asm volatile("" : "+v"(ticket) :: "memory");
    TMARK(tm13);
    call = (unsigned)__builtin_amdgcn_readfirstlane(ticket) / 32u + 1u;
    if (!puller) publish_pair(0, gt, ksl);
    if (pushw || pushx) {
      asm volatile("s_waitcnt vmcnt(0)" ::: "memory");
      unsigned old = 0;
      if (lane == 0) old = __hip_atomic_fetch_add(cnt + 1, 1u, __ATOMIC_RELAXED, __HIP_MEMORY_SCOPE_WORKGROUP);
      old = __builtin_amdgcn_readfirstlane(old);
      const int q = lane & 7;
      const bool need = lane < 8 ? (pushw && (q & 3) == (gt >> 3)) : (pushx && (q >> 2) == (gt >> 4));
      if (old == 3 && lane < 16 && need && q != rank)
        __hip_atomic_store((unsigned*)(wsl + IF_OFF) + (lane < 8 ? (gt & 7) : 8 + (gt & 15)) * 64 + ((rank - q - 1) & 7) * 8 + ksl, call, __ATOMIC_RELAXED, __HIP_MEMORY_SCOPE_SYSTEM);
    }
    if (mover && wv == 1) {
      u32x4 wr[4], xr[4];
      load_w(wr, gt, ksl, 0);
      load_x(xr, gt, ksl, 0);
      store_pair(wr, xr, gt, ksl, 0);
      asm volatile("s_waitcnt vmcnt(0)" ::: "memory");
      publish_pair(0, gt, ksl);
    }
  }
  if (blockIdx.x < 128) {
    const int mb = blockIdx.x;
    if (mb >= 7 * NIN) return;
    const int bi = __builtin_amdgcn_readfirstlane((mb * 9363) >> 16);
    const int lk = __builtin_amdgcn_readfirstlane(mb - 7 * bi);
    const int src = (rank + 1 + lk) & 7;
    const int wi = bi;
    const unsigned l16 = lane * 16;
    const unsigned oob = 0xF0000000u;
    LDS_AS u64* rdesc = (LDS_AS u64*)(lds + LDS_CTRL + 64);
    LDS_AS unsigned* rfree = (LDS_AS unsigned*)(lds + LDS_CTRL + 160);
    LDS_AS unsigned* rland = (LDS_AS unsigned*)(lds + LDS_CTRL + 208);
#define LDS_LD(ptr) __hip_atomic_load(ptr, __ATOMIC_RELAXED, __HIP_MEMORY_SCOPE_WORKGROUP)
#define LDS_ST(ptr, val) __hip_atomic_store(ptr, val, __ATOMIC_RELAXED, __HIP_MEMORY_SCOPE_WORKGROUP)
    if (wv != 0) {
      const unsigned ob = (unsigned)(((src >> 2) * 2048 + r16) * DIM + (src & 3) * 1024 + 8 * g4) * 2;
      int slot = wv - 1;
      for (unsigned sq = wv - 1;; sq += 3) {
        u64 d;
        for (;;) {
          d = LDS_LD(rdesc + slot);
          if ((unsigned)(d >> 48) == sq + 1) break;
          __builtin_amdgcn_s_sleep(4);
        }
        const unsigned dhi = __builtin_amdgcn_readfirstlane((unsigned)(d >> 32)), dend = __builtin_amdgcn_readfirstlane((unsigned)d);
        const unsigned sg = dhi & 0x3ffu;
        if (sg >= 512) break;
        const unsigned nv = dend >> 16, H = nv >> 5;
        while (LDS_LD(rland) < (dend & 0xffffu)) __builtin_amdgcn_s_sleep(1);
        asm volatile("" ::: "memory");
        LDS_AS char* seg = (LDS_AS char*)(lds + 16 + slot * SEG);
        STAMP(4);
        auto unhide = [&](int i0, int n) {
          u32x4 v[5];
#pragma unroll
          for (int i = 0; i < n; ++i) v[i] = *(LDS_AS u32x4*)(seg + (i0 + i) * 1024 + l16);
#pragma unroll
          for (int i = 0; i < n; ++i) {
            unsigned e = ((v[i][0] >> 15) & 0x00010001u) | ((v[i][1] >> 14) & 0x00020002u) | ((v[i][2] >> 13) & 0x00040004u) | ((v[i][3] >> 12) & 0x00080008u);
            e <<= (lane & 3) * 4;
            e |= __builtin_amdgcn_update_dpp(0u, e, 0xb1, 0xf, 0xf, true);
            e |= __builtin_amdgcn_update_dpp(0u, e, 0x4e, 0xf, 0xf, true);
            *(LDS_AS unsigned*)(seg + SEG_TBL + (i0 + i) * 64 + (lane >> 2) * 4) = e;
          }
        };
        unhide(0, 5);
        if (H > 80) unhide(5, 3);
        const unsigned xo = ((2 * nv + 3) & ~3u) - 4 * H;
        u32x2 mk;
        mk.x = *(LDS_AS unsigned*)(seg + (2u * lane < H ? SEG_TBL : xo) + lane * 8);
        mk.y = *(LDS_AS unsigned*)(seg + (2u * lane + 1 < H ? SEG_TBL : xo) + lane * 8 + 4);
        const unsigned c0 = __builtin_popcount(mk.x), cn = c0 + __builtin_popcount(mk.y);
        const int b0 = 2 * (int)(wave_scan(cn) - cn) - 2, b1 = b0 + 2 * (int)c0;
        u32x4 ov[8], oh[8];
#pragma unroll
        for (int c = 0; c < 8; ++c)
#pragma unroll
          for (int dd = 0; dd < 4; ++dd) {
            const int dq = (c & 3) * 4 + dd;
            const unsigned E = c < 4 ? mk.x : mk.y;
            const unsigned pp = __builtin_popcount(E & (((2u << dq) - 1u) | (((1u << dq) - 1u) << 16)));
            const volatile LDS_AS unsigned short* q = (const volatile LDS_AS unsigned short*)(seg + (c < 4 ? b0 : b1) + 2 * (int)pp);
            ov[c][dd] = q[0];
            oh[c][dd] = q[1];
          }
        LDS_ST(rfree + slot, sq + 1);
        STAMP(6);
        const int tile = sg >> 2, sw = sg & 3;
        const unsigned oo = ob + (unsigned)(((tile >> 3) * 128 + (sw >> 1) * 64) * DIM + (tile & 7) * 128 + 64 * (sw & 1)) * 2;
        const unsigned poison = (dhi & 0x8000u) ? 0x7fc07fc0u : 0u;
#pragma unroll
        for (int c = 0; c < 8; ++c) {
#pragma unroll
          for (int dd = 0; dd < 4; ++dd) {
            const unsigned m = ((c < 4 ? mk.x : mk.y) >> ((c & 3) * 4 + dd)) & 0x00010001u;
            const unsigned t = (ov[c][dd] | (oh[c][dd] << 16)) & 0x7fff7fffu;
            const u16x2 r = __builtin_bit_cast(u16x2, t) * __builtin_bit_cast(u16x2, m);
            ov[c][dd] = __builtin_bit_cast(unsigned, r);
          }
          ov[c][0] |= poison;
          __builtin_amdgcn_raw_buffer_store_b128(ov[c], ors, oo + (c & 1) * 64, (c >> 1) * (16 * DIM * 2), OUT_AUX);
          __builtin_amdgcn_sched_barrier(0);
        }
        STAMP(7);
        slot += 3;
        if (slot >= NSLOT) slot -= NSLOT;
      }
      STAMP(11);
      return;
    }
    __amdgpu_buffer_rsrc_t srs = make_rsrc(p.ws[src], WS_USED);
    bool bad = false;
    if (mb == 0 && tid < 7)
      __hip_atomic_store((unsigned*)(myws + QC_OFF + (((call + 1) & 1) * 8 + tid) * 128), 0u, __ATOMIC_RELAXED, __HIP_MEMORY_SCOPE_AGENT);
    {
      const int NT = (192 - NPUSH - wi + NIN - 1) / NIN;
      const unsigned mytask = p.order.t[NPUSH + wi + (lane < NT ? lane : 0) * NIN];
      unsigned* pfl = (unsigned*)(myws + PF_OFF) + src * 192 + mytask;
      u64 ready = 0;
      auto wait_ready = [&](int j) {
        if ((ready >> j) & 1) return;
        u64 t0 = __builtin_amdgcn_s_memrealtime();
        for (unsigned spin = 1;; ++spin) {
          unsigned v = __hip_atomic_load(pfl, __ATOMIC_RELAXED, __HIP_MEMORY_SCOPE_SYSTEM);
          ready = __ballot(v >= call);
          if ((ready >> j) & 1) break;
          if ((spin & 15) == 0 && __builtin_amdgcn_readfirstlane(timed_out(tmo, call, t0))) { bad = true; ready = ~0ull; break; }
          __builtin_amdgcn_s_sleep(1);
        }
      };
      auto src_off = [&](int j) -> unsigned {
        if (j >= NT) return oob;
        const int tb = __builtin_amdgcn_readlane(mytask, j), unit = tb >> 3, ksl = tb & 7;
        if (unit < 8) return (src & 3) == pj ? WS_OFF + (unsigned)(unit * NKS + 56 + ksl) * KSB : SW_OFF + (unsigned)((pj * 8 + unit) * 8 + ksl) * KSB;
        return (src >> 2) == pi ? XS_OFF + (unsigned)((unit - 8) * NKS + 56 + ksl) * KSB : SX_OFF + (unsigned)((unit - 8) * 8 + ksl) * KSB;
      };
      unsigned* ifl = (unsigned*)(myws + IF_OFF) + lk * 8;
      u32x4 v[DEPTH];
#define LD(r, vo, so, imm) asm volatile("buffer_load_dwordx4 %0, %1, %2, 0 offen offset:%3 sc0 sc1" : "=v"(r) : "v"((vo) + (so)), "s"(srs), "i"(imm) : "memory")
#define ST(r, vo, so, imm) asm volatile("buffer_store_dwordx4 %0, %1, %2, 0 offen offset:%3 sc1\n\ts_nop 1" :: "v"(r), "v"((vo) + (so)), "s"(mrs), "i"(imm) : "memory")
      wait_ready(0);
      TMARK(tm2);
      TMARK(tm14);
      unsigned so = src_off(0);
      asm volatile("s_nop 4" ::: "memory");
      {
        u32x4 z = {0, 0, 0, 0};
#pragma unroll
        for (int u = 0; u < DEPTH; ++u) { ST(z, l16, oob, 0); LD(v[u], l16, so + (u >> 2) * 4096, (u & 3) * 1024); }
      }
      int uprev = 0;
      for (int j = 0; j < NT; ++j) {
        const int tb = __builtin_amdgcn_readlane(mytask, j), unit = tb >> 3, ksl = tb & 7;
        const unsigned sn = src_off(j + 1);
        const unsigned dn = (unit < 8 ? WS_OFF + (unsigned)(unit * NKS) * KSB : XS_OFF + (unsigned)((unit - 8) * NKS) * KSB) + (unsigned)(lk * 8 + ksl) * KSB;
        asm volatile("s_nop 4" ::: "memory");
#pragma unroll
        for (int q = 0; q < 16; ++q) {
          asm volatile("s_waitcnt vmcnt(%1)" : "+v"(v[q % DEPTH]) : "i"(2 * (DEPTH - 1)) : "memory");
          if (q == DEPTH - 1 && j > 0 && lane == 0 && !bad)
            __hip_atomic_store(ifl + uprev, call, __ATOMIC_RELAXED, __HIP_MEMORY_SCOPE_AGENT);
          ST(v[q % DEPTH], l16, dn + (q >> 2) * 4096, (q & 3) * 1024);
          const int qn = q + DEPTH;
          if (qn == 16) wait_ready(j + 1);
          if (qn < 16) LD(v[q % DEPTH], l16, so + (qn >> 2) * 4096, (qn & 3) * 1024);
          else LD(v[q % DEPTH], l16, sn + ((qn - 16) >> 2) * 4096, (qn & 3) * 1024);
        }
        so = sn;
        uprev = unit * 64 + ksl;
      }
      asm volatile("s_waitcnt vmcnt(0)" : "+v"(v[0]) :: "memory");
#pragma unroll
      for (int u = 1; u < DEPTH; ++u) asm volatile("" : "+v"(v[u]) :: "memory");
      if (lane == 0 && !bad) __hip_atomic_store(ifl + uprev, call, __ATOMIC_RELAXED, __HIP_MEMORY_SCOPE_AGENT);
    }
    STAMP(3);

    {
      const unsigned qoff = lane == 0 ? QC_OFF + (unsigned)((call & 1) * 8 + lk) * 128 : oob;
      const unsigned ldsbase = (unsigned)(size_t)(LDS_AS char*)lds + 16;
      u32x2 fl = {0u, 0u};
      unsigned tk = 0;
      int k = 0, land = 0;
      unsigned sq = 0;
      int slot = 0;
      int cur = wi, nxt = wi + NIN;
      auto drain = [&]() {
        asm volatile("s_waitcnt vmcnt(0)" : "+v"(fl), "+v"(tk) :: "memory");
        land = k;
        LDS_ST(rland, (unsigned)k);
      };
      auto next_slot = [&]() {
        if (sq >= (unsigned)NSLOT && LDS_LD(rfree + slot) != sq - NSLOT + 1) {
          drain();
          while (LDS_LD(rfree + slot) != sq - NSLOT + 1) __builtin_amdgcn_s_sleep(1);
        }
      };
      unsigned flo = 0, fhi = 0;
      while (cur < 512) {
        const int tile = tile_of(cur >> 2), sw = cur & 3;
        if (fhi < call && !bad) {
          drain();
          u64* flag = (u64*)(myws + OF_OFF + (unsigned)(src * 128 + tile) * 64 + (sw >> 1) * 8);
          u64 t0 = __builtin_amdgcn_s_memrealtime();
          for (;;) {
            const u64 f = __hip_atomic_load(flag, __ATOMIC_RELAXED, __HIP_MEMORY_SCOPE_SYSTEM);
            flo = __builtin_amdgcn_readfirstlane((unsigned)f);
            fhi = __builtin_amdgcn_readfirstlane((unsigned)(f >> 32));
            if (fhi >= call) break;
            if (__builtin_amdgcn_readfirstlane(timed_out(tmo, call, t0))) { bad = true; break; }
            __builtin_amdgcn_s_sleep(2);
          }
        }
        if (bad) flo = 0;
        tk = 1;
        asm volatile("s_nop 4\n\tbuffer_atomic_add %0, %1, %2, 0 offen sc0" : "+v"(tk) : "v"(nxt < 512 ? qoff : oob), "s"(mrs) : "memory");
        asm volatile("buffer_load_dwordx2 %0, %1, %2, 0 offen sc0 sc1" : "=v"(fl) : "v"(nxt < 512 ? OF_OFF + (unsigned)(src * 128 + tile_of(nxt >> 2)) * 64 + (nxt & 2) * 4 : oob), "s"(mrs) : "memory");
        k += 2;
        next_slot();
        const unsigned nv = min((flo >> (14 * (sw & 1))) & 0x3fffu, 4096u);
        const unsigned nb = seg_bytes(nv);
        const unsigned np = (nb + 1023) >> 10;
        LDS_ST(rdesc + slot, ((u64)(((sq + 1) << 16) | (bad ? 0x8000u : 0u) | (unsigned)(tile * 4 + sw)) << 32) | (nv << 16) | (unsigned)(k + np));
        unsigned vo = OB_OFF + (unsigned)(tile * 4 + sw) * SEG + l16;
        const unsigned vend = OB_OFF + (unsigned)(tile * 4 + sw) * SEG + nb;
        unsigned la = ldsbase + slot * SEG;
        for (unsigned i = 0; i < np; ++i) {
          unsigned keep;
          asm volatile("s_waitcnt vmcnt(%0)" :: "i"(ODEPTH - 1) : "memory");
          land = max(land, k - (ODEPTH - 1));
          LDS_ST(rland, (unsigned)land);
          asm volatile("s_mov_b32 %0, m0\n\ts_mov_b32 m0, %3\n\ts_nop 0\n\tbuffer_load_dwordx4 %1, %2, 0 offen sc0 sc1 lds\n\ts_mov_b32 m0, %0"
                       : "=&s"(keep) : "v"(vo < vend ? vo : oob), "s"(srs), "s"(la) : "memory");
          vo += 1024; la += 1024; ++k;
        }
        ++sq;
        if (++slot == NSLOT) slot = 0;
        if (np >= 5) asm volatile("s_waitcnt vmcnt(5)" ::: "memory");
        else if (np == 4) asm volatile("s_waitcnt vmcnt(4)" ::: "memory");
        else asm volatile("s_waitcnt vmcnt(1)" ::: "memory");
        asm volatile("" : "+v"(fl), "+v"(tk) :: "memory");
        flo = __builtin_amdgcn_readfirstlane(fl.x);
        fhi = __builtin_amdgcn_readfirstlane(fl.y);
        const unsigned tku = (unsigned)__builtin_amdgcn_readfirstlane(tk);
        const int nn = (nxt < 512 && tku < 512u) ? 2 * NIN + (int)tku : 512;
        cur = nxt;
        nxt = nn;
      }
      STAMP(10);
      drain();
      for (int e = 0; e < 3; ++e) {
        next_slot();
        LDS_ST(rdesc + slot, (u64)(((sq + 1) << 16) | 0x3ffu) << 32);
        ++sq;
        if (++slot == NSLOT) slot = 0;
      }
    }
    if (__hip_atomic_load(tmo, __ATOMIC_RELAXED, __HIP_MEMORY_SCOPE_AGENT) == call && lane == 0)
      p.out[(size_t)((src >> 2) * 2048) * DIM + (src & 3) * 1024 + wi] = 0x7fc0;
    return;
  }

  const int gb = blockIdx.x - 128;
  const int ct = gb & 7, rt = gb >> 3;
  const int wm = wv >> 1, hf = wv & 1;
  LDS_AS u64* ctl = (LDS_AS u64*)(lds + LDS_CTRL + 32);
  unsigned short* o = p.out + (size_t)((rank >> 2) * 2048 + rt * 128 + wm * 64 + r16) * DIM + (rank & 3) * 1024 + ct * 128 + 64 * hf + 8 * g4;

  f32x4 acc[8][8];
  {
    const unsigned bx = XS_OFF + (unsigned)rt * (NKS * KSB) + lane * 16, bw = WS_OFF + (unsigned)ct * (NKS * KSB) + lane * 16;
    const unsigned vx0 = bx + wm * 8192, vx1 = bx + (wm ^ 1) * 8192;
    const unsigned vw0 = bw + hf * 4096, vw1 = bw + (hf ^ 1) * 4096;
#pragma unroll
    for (int a = 0; a < 8; ++a)
#pragma unroll
      for (int b = 0; b < 8; ++b) acc[a][b] = f32x4{0.f, 0.f, 0.f, 0.f};
    u32x4 fw[3][8], fx[3][8];
#define LDW(dst, vo, so, imm) asm volatile("buffer_load_dwordx4 %0, %1, %2, %3 offen offset:%4 sc1" : "=v"(dst) : "v"(vo), "s"(mrs), "s"(so), "i"(imm))
#define LDX(dst, vo, so, imm) asm volatile("buffer_load_dwordx4 %0, %1, %2, %3 offen offset:%4 sc1" : "=v"(dst) : "v"(vo), "s"(mrs), "s"(so), "i"(imm))
#define MF(T, M, w_, x_) asm volatile("v_mfma_f32_16x16x32_bf16 %0, %1, %2, %0" : "+a"(acc[T][M]) : "v"(w_[T]), "v"(x_[M]))
#define WAITF(w_, x_, n) asm volatile("s_waitcnt vmcnt(" #n ")" : "+v"(w_[0]), "+v"(w_[1]), "+v"(w_[2]), "+v"(w_[3]), "+v"(w_[4]), "+v"(w_[5]), "+v"(w_[6]), "+v"(w_[7]), \
                   "+v"(x_[0]), "+v"(x_[1]), "+v"(x_[2]), "+v"(x_[3]), "+v"(x_[4]), "+v"(x_[5]), "+v"(x_[6]), "+v"(x_[7]))
    auto load1 = [&](int i, unsigned sb, int kk, u32x4* w_, u32x4* x_) {
      const int q = i >> 2, j = i & 3;
      if (q == 0) LDW(w_[j], vw0, sb + kk * 8192, j * 1024);
      else if (q == 1) LDX(x_[j], vx0, sb + (j >> 1) * 4096, (j & 1) * 2048 + kk * 1024);
      else if (q == 2) LDW(w_[4 + j], vw1, sb + kk * 8192, j * 1024);
      else LDX(x_[4 + j], vx1, sb + (j >> 1) * 4096, (j & 1) * 2048 + kk * 1024);
    };
    auto half = [&](u32x4* w_, u32x4* x_, bool doload, unsigned sb, int kk, u32x4* wn, u32x4* xn) {
#pragma unroll
      for (int i = 0; i < 16; ++i) {
        const int qa = i >> 3, qb = (i >> 2) & 1, m = i & 3;
#pragma unroll
        for (int t = 0; t < 4; ++t) MF((qa ^ qb) * 4 + t, qa * 4 + m, w_, x_);
        if (doload) load1(i, sb, kk, wn, xn);
      }
    };
    const unsigned foff = IF_OFF + (unsigned)(((lane & 16) ? ct : 8 + rt) * 64 + ((((wv - lane) & 3) + ((lane & 8) >> 1)) << 3) + (lane & 7)) * 4;
    unsigned todo = 0xffffu, fv = 0;
    auto pick = [&]() -> unsigned {
      u64 rd = __ballot(fv >= call);
      unsigned m = (unsigned)rd & (unsigned)(rd >> 16) & todo;
      if (m == 0) {
        u64 t0 = __builtin_amdgcn_s_memrealtime();
        for (;;) {
          const unsigned v = __hip_atomic_load((unsigned*)(myws + foff), __ATOMIC_RELAXED, __HIP_MEMORY_SCOPE_SYSTEM);
          rd = __ballot(v >= call);
          m = (unsigned)rd & (unsigned)(rd >> 16) & todo;
          if (m) break;
          if (__builtin_amdgcn_readfirstlane(timed_out(tmo, call, t0))) { m = todo; break; }
          __builtin_amdgcn_s_sleep(4);
        }
      }
      const unsigned j = __builtin_ctz(m);
      todo &= ~(1u << j);
      return ((((((unsigned)wv - j) & 3u) + ((j & 8u) >> 1)) << 3) + (j & 7u)) * KSB;
    };
#define LDF() asm volatile("buffer_load_dword %0, %1, %2, 0 offen sc0 sc1" : "=v"(fv) : "v"(foff), "s"(mrs))
    {
      const unsigned sb = pick();
      STAMP(2);
      asm volatile("s_waitcnt vmcnt(0)\n\ts_nop 4" ::: "memory");
      LDF();
#pragma unroll
      for (int i = 0; i < 16; ++i) load1(i, sb, 0, fw[0], fx[0]);
#pragma unroll
      for (int i = 0; i < 16; ++i) load1(i, sb, 1, fw[1], fx[1]);
    }
    for (int s = 0; s < 5; ++s) {
#pragma unroll
      for (int u = 0; u < 6; u += 2) {
        WAITF(fw[u % 3], fx[u % 3], 16);
        asm volatile("" : "+v"(fv));
        const unsigned sb = pick();
        asm volatile("s_nop 4");
        LDF();
        half(fw[u % 3], fx[u % 3], true, sb, 0, fw[(u + 2) % 3], fx[(u + 2) % 3]);
        WAITF(fw[(u + 1) % 3], fx[(u + 1) % 3], 17);
        half(fw[(u + 1) % 3], fx[(u + 1) % 3], true, sb, 1, fw[u % 3], fx[u % 3]);
      }
    }
    WAITF(fw[0], fx[0], 16);
    half(fw[0], fx[0], false, 0, 0, fw[0], fx[0]);
    WAITF(fw[1], fx[1], 0);
    half(fw[1], fx[1], false, 0, 0, fw[0], fx[0]);
    asm volatile("" : "+v"(fv));
    asm volatile("s_nop 15\n\ts_nop 15" ::: "memory");
    __builtin_amdgcn_sched_barrier(0);
  }
  {
    LDS_AS f32x4* ex = (LDS_AS f32x4*)lds;
#pragma unroll
    for (int t = 0; t < 4; ++t)
#pragma unroll
      for (int m = 0; m < 8; ++m) ex[(wv * 32 + t * 8 + m) * 64 + lane] = acc[4 + t][m];
    if (__hip_atomic_load(tmo, __ATOMIC_RELAXED, __HIP_MEMORY_SCOPE_AGENT) == call) cnt[4] = 1;
    __syncthreads();
    if (cnt[4]) {
      const u32x4 nanv = {0x7fc07fc0u, 0x7fc07fc0u, 0x7fc07fc0u, 0x7fc07fc0u};
#pragma unroll
      for (int c = 0; c < 8; ++c) *(u32x4*)(o + (size_t)(c >> 1) * 16 * DIM + (c & 1) * 32) = nanv;
      return;
    }
#pragma unroll
    for (int t = 0; t < 4; ++t)
#pragma unroll
      for (int m = 0; m < 8; ++m) acc[t][m] += ex[((wv ^ 1) * 32 + t * 8 + m) * 64 + lane];
    __syncthreads();
#pragma unroll
    for (int t = 0; t < 4; ++t)
#pragma unroll
      for (int m = 0; m < 4; ++m) ex[(64 + wv * 16 + t * 4 + m) * 64 + lane] = acc[t][4 + m];
    __syncthreads();
#pragma unroll
    for (int t = 0; t < 4; ++t)
#pragma unroll
      for (int m = 0; m < 4; ++m) acc[t][m] += ex[(64 + (wv ^ 2) * 16 + t * 4 + m) * 64 + lane];
  }
  STAMP(3);

  {
    LDS_AS char* seg = (LDS_AS char*)(lds + wv * SEG);
    u32x4 pk[8];
    unsigned E[2] = {0u, 0u};
#pragma unroll
    for (int c = 0; c < 8; ++c) {
      const int mt = c >> 1, pc = c & 1;
      float v[8];
#pragma unroll
      for (int e = 0; e < 8; ++e) {
        const float a = acc[e & 3][mt][2 * pc + (e >> 2)];
        v[e] = (a <= 0.f) ? 0.f : a;
      }
      pk[c] = u32x4{pack_bf16(v[0], v[1]), pack_bf16(v[2], v[3]), pack_bf16(v[4], v[5]), pack_bf16(v[6], v[7])};
      pk[c] &= 0x7fff7fffu;
      __builtin_amdgcn_raw_buffer_store_b128(pk[c], ors, (unsigned)((char*)o - (char*)p.out) + pc * 64, mt * (16 * DIM * 2), OUT_AUX);
#pragma unroll
      for (int dd = 0; dd < 4; ++dd) {
        const unsigned t = pk[c][dd];
        unsigned nz;
        asm("v_pk_min_u16 %0, %1, 1 op_sel_hi:[1,0]" : "=v"(nz) : "v"(t));
        E[c >> 2] |= nz << ((c & 3) * 4 + dd);
      }
    }
    const unsigned cn = __builtin_popcount(E[0]) + __builtin_popcount(E[1]);
    const unsigned inc = wave_scan(cn);
    const unsigned total = __builtin_amdgcn_readlane(inc, 63);
    const unsigned H = total >> 5;
    {
      const unsigned xo = ((2 * total + 3) & ~3u) - 4 * H;
      *(LDS_AS u32x2*)(seg + SEG_TBL + lane * 8) = u32x2{2u * lane < H ? E[0] : 0u, 2u * lane + 1 < H ? E[1] : 0u};
      if (lane == 63) *(LDS_AS unsigned short*)(seg + 2 * total) = 0;
      if (2u * lane >= H) *(LDS_AS unsigned*)(seg + xo + lane * 8) = E[0];
      if (2u * lane + 1 >= H) *(LDS_AS unsigned*)(seg + xo + lane * 8 + 4) = E[1];
    }
    unsigned wa = (unsigned)(size_t)seg + 2 * (inc - cn);
#pragma unroll
    for (int c = 0; c < 8; ++c)
#pragma unroll
      for (int dd = 0; dd < 4; ++dd)
        asm volatile("v_cmpx_ne_u16 vcc, 0, %1\n\t"
                     "ds_write_b16 %0, %1\n\t"
                     "v_add_u32 %0, 2, %0\n\t"
                     "s_mov_b64 exec, -1\n\t"
                     "v_cmpx_lt_u32 vcc, %2, %1\n\t"
                     "ds_write_b16_d16_hi %0, %1\n\t"
                     "v_add_u32 %0, 2, %0\n\t"
                     "s_mov_b64 exec, -1"
                     : "+v"(wa) : "v"(pk[c][dd]), "s"(0xffffu) : "vcc", "memory");
    asm volatile("s_waitcnt lgkmcnt(0)" ::: "memory");
    const unsigned nb = seg_bytes(total);
    const unsigned so = OB_OFF + (unsigned)(gb * 4 + wv) * SEG + lane * 16;
#pragma unroll
    for (int i = 0; i < 9; ++i) {
      u32x4 v = *(LDS_AS u32x4*)(seg + i * 1024 + lane * 16);
      if (i < 8) {
        const unsigned e = *(LDS_AS unsigned*)(seg + SEG_TBL + i * 64 + (lane >> 2) * 4) >> ((lane & 3) * 4);
#pragma unroll
        for (int d = 0; d < 4; ++d) v[d] |= (e << (15 - d)) & 0x80008000u;
      }
      __builtin_amdgcn_raw_buffer_store_b128(v, mrs, (unsigned)(i * 1024 + lane * 16) < nb ? so + i * 1024 : 0xF0000000u, 0, AUX_SYS);
    }
    STAMP(4);
    asm volatile("s_waitcnt vmcnt(0)" ::: "memory");
    u64 old = 0;
    const u64 mine = (1ull << 56) | ((u64)total << (14 * wv));
    if (lane == 0) old = __hip_atomic_fetch_add(ctl, mine, __ATOMIC_RELAXED, __HIP_MEMORY_SCOPE_WORKGROUP);
    old = (((u64)(unsigned)__builtin_amdgcn_readfirstlane((unsigned)(old >> 32)) << 32) | (unsigned)__builtin_amdgcn_readfirstlane((unsigned)old)) + mine;
    if ((old >> 56) == 4 && lane < 16 && (lane & 7) != rank) {
      const u64 f = ((u64)call << 32) | (unsigned)((old >> (28 * (lane >> 3))) & 0xfffffffu);
      __hip_atomic_store((u64*)(p.ws[lane & 7] + OF_OFF + (unsigned)(rank * 128 + gb) * 64 + (lane >> 3) * 8), f, __ATOMIC_RELAXED, __HIP_MEMORY_SCOPE_SYSTEM);
    }
    STAMP(5);
  }
}

}

extern "C" void kernel_launch(int rank, int nranks, ncclComm_t comm,
                              hipStream_t stream,
                              void* const* d_in, const int* in_sizes, int n_in,
                              void* d_out, int out_size,
                              const int* in_dtypes, int out_dtype,
                              void* const* d_out_all,
                              void* const* d_ws_all, size_t ws_bytes) {
  (void)comm; (void)in_sizes; (void)n_in; (void)out_size; (void)in_dtypes; (void)out_dtype; (void)d_out_all;
  if (nranks != NR || (unsigned)rank >= (unsigned)NR || ws_bytes < (size_t)WS_USED) {
    fprintf(stderr, "unsupported configuration\n");
    return;
  }
  Params p;
  p.x = (const unsigned short*)d_in[0];
  p.w = (const unsigned short*)d_in[1];
  p.out = (unsigned short*)d_out;
  for (int i = 0; i < NR; ++i) p.ws[i] = (char*)d_ws_all[i];
  p.myws = p.ws[rank];
  p.rank = rank;
  constexpr OrderTable kOrder = make_order();
  constexpr PrepTable kPrep = make_prep();
  p.order = kOrder;
  p.prep = kPrep;
  gemm_ar_kernel<<<256, 256, 0, stream>>>(p);
  HIP_CHECK(hipGetLastError());
}
```

```cpp
#include <hip/hip_runtime.h>
#include <rccl/rccl.h>
#include <cstdio>
#include <cstdint>

#define HIP_CHECK(expr) do { hipError_t _e = (expr); if (_e != hipSuccess) { \
    fprintf(stderr, "HIP error %s at %s:%d\n", hipGetErrorString(_e), __FILE__, __LINE__); \
    return; } } while (0)

namespace {

typedef float f32x4 __attribute__((ext_vector_type(4)));
typedef unsigned u32x4 __attribute__((ext_vector_type(4)));
typedef unsigned u32x2 __attribute__((ext_vector_type(2)));
typedef unsigned long long u64;

constexpr int NR = 8;
constexpr int DIM = 4096;
constexpr int KSL = DIM / NR;
constexpr int RT = 16, CT = 8;
constexpr int NKS = 64;
constexpr unsigned KSB = 16384;
constexpr unsigned SEG = 9216;
constexpr unsigned SEG_TBL = 8704;
__host__ __device__ constexpr unsigned seg_bytes(unsigned n) { return ((2 * n + 3) & ~3u) + 4 * (128 - (n >> 5)); }

constexpr unsigned XS_OFF = 0;
constexpr unsigned WS_OFF = XS_OFF + RT * NKS * KSB;
constexpr unsigned SX_OFF = WS_OFF + CT * NKS * KSB;
constexpr unsigned SW_OFF = SX_OFF + RT * 8 * KSB;
constexpr unsigned OB_OFF = SW_OFF + 32 * 8 * KSB;
constexpr unsigned IF_OFF = OB_OFF + RT * CT * 4 * SEG;
constexpr unsigned PF_OFF = IF_OFF + 24 * 256;
constexpr unsigned OF_OFF = PF_OFF + NR * 192 * 4;
constexpr unsigned TK_OFF = OF_OFF + NR * RT * CT * 64;
constexpr unsigned QC_OFF = TK_OFF + 32 * 128;
constexpr unsigned TMO_OFF = QC_OFF + 16 * 128;
constexpr unsigned WS_USED = TMO_OFF + 128;

constexpr int DEPTH = 4;
constexpr int ODEPTH = 5;
constexpr int NIN = 16;
constexpr int NSLOT = 12;
constexpr int PK = 2;
constexpr int NPUSH = (PK + 1) * (PK + 2) / 2;
constexpr unsigned NEARLY = 48;
constexpr int PDELAY = 40, PSTEP = 16;

constexpr u64 DEADLINE_TICKS = 50000000ull;

constexpr int LDS_CTRL = 131072;
constexpr int LDS_BYTES = LDS_CTRL + 256;

struct alignas(4) OrderTable { unsigned char t[192]; };
constexpr OrderTable make_order() {
  OrderTable o{};
  int n = 0;
  for (int key = 0; key <= 23 + 7; ++key)
    for (int k = 7; k >= 0; --k)
      for (int s = 0; s < 24; ++s)
        if (s + k == key) o.t[n++] = (unsigned char)((s < 16 ? ((s & 1) ? (s >> 1) : 8 + (s >> 1)) : s) * 8 + k);
  return o;
}
struct alignas(4) PrepTable { unsigned char t[256]; };
constexpr PrepTable make_prep() {
  PrepTable o{};
  int n = 0;
  for (int key = 0; key <= 23 + 7; ++key)
    for (int gt = 0; gt < 32; ++gt)
      for (int k = 0; k < 8; ++k) {
        const int x = gt & 15, sw = 2 * (gt & 7) + 1, sx = x < 8 ? 2 * x : x + 8;
        if ((sw < sx ? sw : sx) + k == key) o.t[n++] = (unsigned char)(gt * 8 + k);
      }
  return o;
}

struct Params {
  const unsigned short* x;
  const unsigned short* w;
  unsigned short* out;
  char* myws;
  char* ws[NR];
  int rank;
  OrderTable order;
  PrepTable prep;
};

#define LDS_AS __attribute__((address_space(3)))

__device__ __forceinline__ __amdgpu_buffer_rsrc_t make_rsrc(const void* p, unsigned bytes) {
  return __builtin_amdgcn_make_buffer_rsrc((void*)p, 0, bytes, 0x00020000);
}
constexpr int AUX_SYS = 17;
constexpr int OUT_AUX = 16;

__device__ __forceinline__ bool timed_out(unsigned* tmo, unsigned call, u64 t0) {
  if (__hip_atomic_load(tmo, __ATOMIC_RELAXED, __HIP_MEMORY_SCOPE_AGENT) == call ||
      __builtin_amdgcn_s_memrealtime() - t0 > DEADLINE_TICKS) {
    __hip_atomic_store(tmo, call, __ATOMIC_RELAXED, __HIP_MEMORY_SCOPE_AGENT);
    return true;
  }
  return false;
}

__device__ __forceinline__ unsigned pack_bf16(float a, float b) {
  typedef __bf16 bf2 __attribute__((ext_vector_type(2)));
  typedef float f2 __attribute__((ext_vector_type(2)));
  f2 v = {a, b};
  bf2 r = __builtin_convertvector(v, bf2);
  return __builtin_bit_cast(unsigned, r);
}

__device__ __forceinline__ unsigned wave_scan(unsigned x) {
  x += __builtin_amdgcn_update_dpp(0u, x, 0x111, 0xf, 0xf, false);
  x += __builtin_amdgcn_update_dpp(0u, x, 0x112, 0xf, 0xf, false);
  x += __builtin_amdgcn_update_dpp(0u, x, 0x114, 0xf, 0xf, false);
  x += __builtin_amdgcn_update_dpp(0u, x, 0x118, 0xf, 0xf, false);
  x += __builtin_amdgcn_update_dpp(0u, x, 0x142, 0xa, 0xf, false);
  x += __builtin_amdgcn_update_dpp(0u, x, 0x143, 0xc, 0xf, false);
  return x;
}
typedef unsigned short u16x2 __attribute__((ext_vector_type(2)));

__device__ __forceinline__ int tile_of(int i) {
  if (i >= 64) return i;
  const int n = (i >= 1) + (i >= 4) + (i >= 9) + (i >= 16) + (i >= 25) + (i >= 36) + (i >= 49);
  const int rem = i - n * n;
  return rem < n ? n * 8 + rem : (rem - n) * 8 + n;
}

__global__ void __launch_bounds__(256, 1) gemm_ar_kernel(Params p) {
  __shared__ __attribute__((aligned(16))) char lds[LDS_BYTES];
  const int tid = threadIdx.x;
  const int lane = tid & 63;
  const int wv = __builtin_amdgcn_readfirstlane(tid >> 6);
  const int rank = p.rank;
  char* myws = p.myws;
  unsigned* tmo = (unsigned*)(myws + TMO_OFF);
  const int r16 = lane & 15, g4 = lane >> 4;

  const int pi = rank >> 2, pj = rank & 3;
  __amdgpu_buffer_rsrc_t mrs = make_rsrc(myws, WS_USED);
  __amdgpu_buffer_rsrc_t ors = make_rsrc(p.out, (unsigned)DIM * DIM * 2);
  LDS_AS unsigned* cnt = (LDS_AS unsigned*)(lds + LDS_CTRL);
  unsigned call;
  {
    const unsigned job = blockIdx.x ^ 128u;
    const unsigned jobw = ((const unsigned*)p.prep.t)[job >> 2];
    const unsigned short* const px = p.x;
    const unsigned short* const pw = p.w;
    asm volatile("" :: "s"(px), "s"(pw), "s"(jobw), "s"(rank), "s"(myws));
    unsigned ticket = 1;
    asm volatile("buffer_atomic_add %0, %1, %2, 0 offen sc0" : "+v"(ticket) : "v"(lane == 0 ? TK_OFF + (blockIdx.x & 31) * 128 : 0xF0000000u), "s"(mrs) : "memory");
    char* wsl = p.ws[0];
#pragma unroll
    for (int i = 1; i < NR; ++i) wsl = (lane & 7) == i ? p.ws[i] : wsl;
    const int cbr = 8 * ((r16 & 3) >> 1) + 2 * (r16 >> 2) + (r16 & 1);
    auto load_w = [&](u32x4* r, int t, int k, int qt) {
      const unsigned short* q = pw + (size_t)(k * 64 + (qt >> 1) * 32 + 8 * g4) * DIM + t * 128 + 64 * (qt & 1) + 4 * cbr;
#pragma unroll
      for (int h = 0; h < 4; ++h) {
        const u32x2 a = *(const u32x2*)(q + (size_t)(2 * h) * DIM), b = *(const u32x2*)(q + (size_t)(2 * h + 1) * DIM);
        r[h] = u32x4{a.x, a.y, b.x, b.y};
      }
    };
    auto load_x = [&](u32x4* r, int t, int k, int qt) {
      const unsigned short* q = px + (size_t)(t * 128 + qt * 32 + r16) * KSL + k * 64 + 8 * g4;
#pragma unroll
      for (int h = 0; h < 4; ++h) r[h] = *(const u32x4*)(q + (size_t)(h >> 1) * 16 * KSL + (h & 1) * 32);
    };
    auto transpose_w = [&](u32x4* r) {
      u32x4 v[4];
#pragma unroll
      for (int t = 0; t < 4; ++t)
#pragma unroll
        for (int h = 0; h < 4; ++h) {
          const unsigned lo = r[h][t >> 1], hi = r[h][2 + (t >> 1)];
          v[t][h] = __builtin_amdgcn_perm(hi, lo, (t & 1) ? 0x07060302u : 0x05040100u);
        }
#pragma unroll
      for (int t = 0; t < 4; ++t) r[t] = v[t];
    };
    auto store_pair = [&](u32x4* wr, u32x4* xr, int gt, int ksl, int qt) {
      const unsigned wo = ((gt >> 3) == pj ? WS_OFF + (unsigned)((gt & 7) * NKS + 56 + ksl) * KSB : SW_OFF + (unsigned)(gt * 8 + ksl) * KSB) + qt * 4096 + lane * 16;
      const unsigned xo = ((gt >> 4) == pi ? XS_OFF + (unsigned)((gt & 15) * NKS + 56 + ksl) * KSB : SX_OFF + (unsigned)((gt & 15) * 8 + ksl) * KSB) + qt * 4096 + lane * 16;
      transpose_w(wr);
#pragma unroll
      for (int t = 0; t < 4; ++t) __builtin_amdgcn_raw_buffer_store_b128(wr[t], mrs, wo + t * 1024, 0, AUX_SYS);
#pragma unroll
      for (int h = 0; h < 4; ++h) __builtin_amdgcn_raw_buffer_store_b128(xr[h], mrs, xo + h * 1024, 0, AUX_SYS);
    };
    auto publish_pair = [&](int ci, int gt, int ksl) {
      unsigned old = 0;
      if (lane == 0) old = __hip_atomic_fetch_add(cnt + ci, 1u, __ATOMIC_RELAXED, __HIP_MEMORY_SCOPE_WORKGROUP);
      old = __builtin_amdgcn_readfirstlane(old);
      const int q = lane & 7;
      const bool need = lane < 8 ? ((q & 3) == (gt >> 3)) : ((q >> 2) == (gt >> 4));
      if (old == 3 && lane < 16 && need) {
        const int unit = lane < 8 ? (gt & 7) : 8 + (gt & 15);
        unsigned* fl = q == rank ? (unsigned*)(myws + IF_OFF) + unit * 64 + 56 + ksl
                                 : (unsigned*)(wsl + PF_OFF) + rank * 192 + unit * 8 + ksl;
        __hip_atomic_store(fl, call, __ATOMIC_RELAXED, __HIP_MEMORY_SCOPE_SYSTEM);
      }
    };
    const int jb = (jobw >> (8 * (job & 3))) & 255, gt = jb >> 3, ksl = jb & 7;
    const bool mover = blockIdx.x < 7 * NIN;
    const bool puller = mover && wv == 0;
    const bool pushw = 2 * (gt & 7) + 1 + ksl <= PK, pushx = 2 * (gt & 15) + ksl <= PK;
    if (tid < 64) cnt[tid] = 0;
    asm volatile("s_waitcnt lgkmcnt(0)" ::: "memory");
    __builtin_amdgcn_s_barrier();
    if (!puller) {
      if (job >= NEARLY) {
        __builtin_amdgcn_s_sleep(PDELAY);
        for (unsigned i = NEARLY; i < job; i += 8) __builtin_amdgcn_s_sleep(PSTEP);
      }
      u32x4 wr[4], xr[4];
      load_w(wr, gt, ksl, wv);
      load_x(xr, gt, ksl, wv);
      store_pair(wr, xr, gt, ksl, wv);
      if (pushw || pushx) {
        asm volatile("" ::: "memory");
#pragma unroll
        for (int i = 0; i < 6; ++i) {
          const int q = i < 2 ? (gt >> 3) + 4 * i : (gt >> 4) * 4 + i - 2;
          const bool on = (i < 2 ? pushw : pushx) && q != rank;
          __amdgpu_buffer_rsrc_t prs = make_rsrc(p.ws[q], WS_USED);
          const unsigned po = (i < 2 ? WS_OFF + (unsigned)((gt & 7) * NKS) * KSB : XS_OFF + (unsigned)((gt & 15) * NKS) * KSB)
                              + (unsigned)(((rank - q - 1) & 7) * 8 + ksl) * KSB + wv * 4096 + lane * 16;
#pragma unroll
          for (int h = 0; h < 4; ++h) __builtin_amdgcn_raw_buffer_store_b128(i < 2 ? wr[h] : xr[h], prs, on ? po + h * 1024 : 0xF0000000u, 0, AUX_SYS);
        }
      }
    }
    if (pushw || pushx) asm volatile("s_waitcnt vmcnt(24)" ::: "memory");
    else asm volatile("s_waitcnt vmcnt(0)" ::: "memory");
    asm volatile("" : "+v"(ticket) :: "memory");
    call = (unsigned)__builtin_amdgcn_readfirstlane(ticket) / 32u + 1u;
    if (!puller) publish_pair(0, gt, ksl);
    if (pushw || pushx) {
      asm volatile("s_waitcnt vmcnt(0)" ::: "memory");
      unsigned old = 0;
      if (lane == 0) old = __hip_atomic_fetch_add(cnt + 1, 1u, __ATOMIC_RELAXED, __HIP_MEMORY_SCOPE_WORKGROUP);
      old = __builtin_amdgcn_readfirstlane(old);
      const int q = lane & 7;
      const bool need = lane < 8 ? (pushw && (q & 3) == (gt >> 3)) : (pushx && (q >> 2) == (gt >> 4));
      if (old == 3 && lane < 16 && need && q != rank)
        __hip_atomic_store((unsigned*)(wsl + IF_OFF) + (lane < 8 ? (gt & 7) : 8 + (gt & 15)) * 64 + ((rank - q - 1) & 7) * 8 + ksl, call, __ATOMIC_RELAXED, __HIP_MEMORY_SCOPE_SYSTEM);
    }
    if (mover && wv == 1) {
      u32x4 wr[4], xr[4];
      load_w(wr, gt, ksl, 0);
      load_x(xr, gt, ksl, 0);
      store_pair(wr, xr, gt, ksl, 0);
      asm volatile("s_waitcnt vmcnt(0)" ::: "memory");
      publish_pair(0, gt, ksl);
    }
  }
  if (blockIdx.x < 128) {
    const int mb = blockIdx.x;
    if (mb >= 7 * NIN) return;
    const int bi = __builtin_amdgcn_readfirstlane((mb * 9363) >> 16);
    const int lk = __builtin_amdgcn_readfirstlane(mb - 7 * bi);
    const int src = (rank + 1 + lk) & 7;
    const int wi = bi;
    const unsigned l16 = lane * 16;
    const unsigned oob = 0xF0000000u;
    LDS_AS u64* rdesc = (LDS_AS u64*)(lds + LDS_CTRL + 64);
    LDS_AS unsigned* rfree = (LDS_AS unsigned*)(lds + LDS_CTRL + 160);
    LDS_AS unsigned* rland = (LDS_AS unsigned*)(lds + LDS_CTRL + 208);
#define LDS_LD(ptr) __hip_atomic_load(ptr, __ATOMIC_RELAXED, __HIP_MEMORY_SCOPE_WORKGROUP)
#define LDS_ST(ptr, val) __hip_atomic_store(ptr, val, __ATOMIC_RELAXED, __HIP_MEMORY_SCOPE_WORKGROUP)
    const u64 tstart = __builtin_amdgcn_s_memrealtime();
    auto ring_dead = [&]() -> bool {
      if (__builtin_amdgcn_s_memrealtime() - tstart <= 3 * DEADLINE_TICKS) return false;
      __hip_atomic_store(tmo, call, __ATOMIC_RELAXED, __HIP_MEMORY_SCOPE_AGENT);
      if (lane == 0) p.out[(size_t)((src >> 2) * 2048) * DIM + (src & 3) * 1024 + wi] = 0x7fc0;
      return true;
    };
    if (wv != 0) {
      const unsigned ob = (unsigned)(((src >> 2) * 2048 + r16) * DIM + (src & 3) * 1024 + 8 * g4) * 2;
      int slot = wv - 1;
      for (unsigned sq = wv - 1;; sq += 3) {
        u64 d;
        for (;;) {
          d = LDS_LD(rdesc + slot);
          if ((unsigned)(d >> 48) == sq + 1) break;
          if (ring_dead()) return;
          __builtin_amdgcn_s_sleep(4);
        }
        const unsigned dhi = __builtin_amdgcn_readfirstlane((unsigned)(d >> 32)), dend = __builtin_amdgcn_readfirstlane((unsigned)d);
        const unsigned sg = dhi & 0x3ffu;
        if (sg >= 512) break;
        const unsigned nv = dend >> 16, H = nv >> 5;
        while (LDS_LD(rland) < (dend & 0xffffu)) {
          if (ring_dead()) return;
          __builtin_amdgcn_s_sleep(1);
        }
        asm volatile("" ::: "memory");
        LDS_AS char* seg = (LDS_AS char*)(lds + 16 + slot * SEG);
        auto unhide = [&](int i0, int n) {
          u32x4 v[5];
#pragma unroll
          for (int i = 0; i < n; ++i) v[i] = *(LDS_AS u32x4*)(seg + (i0 + i) * 1024 + l16);
#pragma unroll
          for (int i = 0; i < n; ++i) {
            unsigned e = ((v[i][0] >> 15) & 0x00010001u) | ((v[i][1] >> 14) & 0x00020002u) | ((v[i][2] >> 13) & 0x00040004u) | ((v[i][3] >> 12) & 0x00080008u);
            e <<= (lane & 3) * 4;
            e |= __builtin_amdgcn_update_dpp(0u, e, 0xb1, 0xf, 0xf, true);
            e |= __builtin_amdgcn_update_dpp(0u, e, 0x4e, 0xf, 0xf, true);
            *(LDS_AS unsigned*)(seg + SEG_TBL + (i0 + i) * 64 + (lane >> 2) * 4) = e;
          }
        };
        unhide(0, 5);
        if (H > 80) unhide(5, 3);
        const unsigned xo = ((2 * nv + 3) & ~3u) - 4 * H;
        u32x2 mk;
        mk.x = *(LDS_AS unsigned*)(seg + (2u * lane < H ? SEG_TBL : xo) + lane * 8);
        mk.y = *(LDS_AS unsigned*)(seg + (2u * lane + 1 < H ? SEG_TBL : xo) + lane * 8 + 4);
        const unsigned c0 = __builtin_popcount(mk.x), cn = c0 + __builtin_popcount(mk.y);
        const int b0 = 2 * (int)(wave_scan(cn) - cn) - 2, b1 = b0 + 2 * (int)c0;
        u32x4 ov[8], oh[8];
#pragma unroll
        for (int c = 0; c < 8; ++c)
#pragma unroll
          for (int dd = 0; dd < 4; ++dd) {
            const int dq = (c & 3) * 4 + dd;
            const unsigned E = c < 4 ? mk.x : mk.y;
            const unsigned pp = __builtin_popcount(E & (((2u << dq) - 1u) | (((1u << dq) - 1u) << 16)));
            const volatile LDS_AS unsigned short* q = (const volatile LDS_AS unsigned short*)(seg + (c < 4 ? b0 : b1) + 2 * (int)pp);
            ov[c][dd] = q[0];
            oh[c][dd] = q[1];
          }
        LDS_ST(rfree + slot, sq + 1);
        const int tile = sg >> 2, sw = sg & 3;
        const unsigned oo = ob + (unsigned)(((tile >> 3) * 128 + (sw >> 1) * 64) * DIM + (tile & 7) * 128 + 64 * (sw & 1)) * 2;
        const unsigned poison = (dhi & 0x8000u) ? 0x7fc07fc0u : 0u;
#pragma unroll
        for (int c = 0; c < 8; ++c) {
#pragma unroll
          for (int dd = 0; dd < 4; ++dd) {
            const unsigned m = ((c < 4 ? mk.x : mk.y) >> ((c & 3) * 4 + dd)) & 0x00010001u;
            const unsigned t = (ov[c][dd] | (oh[c][dd] << 16)) & 0x7fff7fffu;
            const u16x2 r = __builtin_bit_cast(u16x2, t) * __builtin_bit_cast(u16x2, m);
            ov[c][dd] = __builtin_bit_cast(unsigned, r);
          }
          ov[c][0] |= poison;
          __builtin_amdgcn_raw_buffer_store_b128(ov[c], ors, oo + (c & 1) * 64, (c >> 1) * (16 * DIM * 2), OUT_AUX);
          __builtin_amdgcn_sched_barrier(0);
        }
        slot += 3;
        if (slot >= NSLOT) slot -= NSLOT;
      }
      return;
    }
    __amdgpu_buffer_rsrc_t srs = make_rsrc(p.ws[src], WS_USED);
    bool bad = false;
    if (mb == 0 && tid < 7)
      __hip_atomic_store((unsigned*)(myws + QC_OFF + (((call + 1) & 1) * 8 + tid) * 128), 0u, __ATOMIC_RELAXED, __HIP_MEMORY_SCOPE_AGENT);
    u32x2 fl = {0u, 0u};
    const unsigned fo0 = OF_OFF + (unsigned)(src * 128 + tile_of(wi >> 2)) * 64 + (wi & 2) * 4;
    {
      const int NT = (192 - NPUSH - wi + NIN - 1) / NIN;
      const unsigned mytask = p.order.t[NPUSH + wi + (lane < NT ? lane : 0) * NIN];
      unsigned* pfl = (unsigned*)(myws + PF_OFF) + src * 192 + mytask;
      u64 ready = 0;
      auto wait_ready = [&](int j) {
        if ((ready >> j) & 1) return;
        u64 t0 = __builtin_amdgcn_s_memrealtime();
        for (unsigned spin = 1;; ++spin) {
          unsigned v = __hip_atomic_load(pfl, __ATOMIC_RELAXED, __HIP_MEMORY_SCOPE_SYSTEM);
          ready = __ballot(v >= call);
          if ((ready >> j) & 1) break;
          if ((spin & 15) == 0 && __builtin_amdgcn_readfirstlane(timed_out(tmo, call, t0))) { bad = true; ready = ~0ull; break; }
          __builtin_amdgcn_s_sleep(1);
        }
      };
      auto src_off = [&](int j) -> unsigned {
        if (j >= NT) return oob;
        const int tb = __builtin_amdgcn_readlane(mytask, j), unit = tb >> 3, ksl = tb & 7;
        if (unit < 8) return (src & 3) == pj ? WS_OFF + (unsigned)(unit * NKS + 56 + ksl) * KSB : SW_OFF + (unsigned)((pj * 8 + unit) * 8 + ksl) * KSB;
        return (src >> 2) == pi ? XS_OFF + (unsigned)((unit - 8) * NKS + 56 + ksl) * KSB : SX_OFF + (unsigned)((unit - 8) * 8 + ksl) * KSB;
      };
      unsigned* ifl = (unsigned*)(myws + IF_OFF) + lk * 8;
      u32x4 v[DEPTH];
#define LD(r, vo, so, imm) asm volatile("buffer_load_dwordx4 %0, %1, %2, 0 offen offset:%3 sc0 sc1" : "=v"(r) : "v"((vo) + (so)), "s"(srs), "i"(imm) : "memory")
#define ST(r, vo, so, imm) asm volatile("buffer_store_dwordx4 %0, %1, %2, 0 offen offset:%3 sc1\n\ts_nop 1" :: "v"(r), "v"((vo) + (so)), "s"(mrs), "i"(imm) : "memory")
      wait_ready(0);
      unsigned so = src_off(0);
      asm volatile("s_nop 4" ::: "memory");
      {
        u32x4 z = {0, 0, 0, 0};
#pragma unroll
        for (int u = 0; u < DEPTH; ++u) { ST(z, l16, oob, 0); LD(v[u], l16, so + (u >> 2) * 4096, (u & 3) * 1024); }
      }
      int uprev = 0;
      for (int j = 0; j < NT; ++j) {
        const int tb = __builtin_amdgcn_readlane(mytask, j), unit = tb >> 3, ksl = tb & 7;
        const unsigned sn = src_off(j + 1);
        const unsigned dn = (unit < 8 ? WS_OFF + (unsigned)(unit * NKS) * KSB : XS_OFF + (unsigned)((unit - 8) * NKS) * KSB) + (unsigned)(lk * 8 + ksl) * KSB;
        asm volatile("s_nop 4" ::: "memory");
#pragma unroll
        for (int q = 0; q < 16; ++q) {
          asm volatile("s_waitcnt vmcnt(%1)" : "+v"(v[q % DEPTH]) : "i"(2 * (DEPTH - 1)) : "memory");
          if (q == DEPTH - 1 && j > 0 && lane == 0 && !bad)
            __hip_atomic_store(ifl + uprev, call, __ATOMIC_RELAXED, __HIP_MEMORY_SCOPE_AGENT);
          ST(v[q % DEPTH], l16, dn + (q >> 2) * 4096, (q & 3) * 1024);
          const int qn = q + DEPTH;
          if (qn == 16) {
            wait_ready(j + 1);
            asm volatile("buffer_load_dwordx2 %0, %1, %2, 0 offen sc0 sc1" : "+v"(fl) : "v"(j + 1 == NT ? fo0 : oob), "s"(mrs) : "memory");
          }
          if (qn < 16) LD(v[q % DEPTH], l16, so + (qn >> 2) * 4096, (qn & 3) * 1024);
          else LD(v[q % DEPTH], l16, sn + ((qn - 16) >> 2) * 4096, (qn & 3) * 1024);
        }
        so = sn;
        uprev = unit * 64 + ksl;
      }
      asm volatile("s_waitcnt vmcnt(0)" : "+v"(v[0]), "+v"(fl) :: "memory");
#pragma unroll
      for (int u = 1; u < DEPTH; ++u) asm volatile("" : "+v"(v[u]) :: "memory");
      if (lane == 0 && !bad) __hip_atomic_store(ifl + uprev, call, __ATOMIC_RELAXED, __HIP_MEMORY_SCOPE_AGENT);
    }

    {
      const unsigned qoff = lane == 0 ? QC_OFF + (unsigned)((call & 1) * 8 + lk) * 128 : oob;
      const unsigned ldsbase = (unsigned)(size_t)(LDS_AS char*)lds + 16;
      unsigned tk = 0;
      int k = 0, land = 0;
      unsigned sq = 0;
      int slot = 0;
      int cur = wi, nxt = wi + NIN;
      auto drain = [&]() {
        asm volatile("s_waitcnt vmcnt(0)" : "+v"(fl), "+v"(tk) :: "memory");
        land = k;
        LDS_ST(rland, (unsigned)k);
      };
      auto next_slot = [&]() {
        if (sq >= (unsigned)NSLOT && LDS_LD(rfree + slot) != sq - NSLOT + 1) {
          drain();
          while (LDS_LD(rfree + slot) != sq - NSLOT + 1) {
            if (ring_dead()) { bad = true; break; }
            __builtin_amdgcn_s_sleep(1);
          }
        }
      };
      unsigned flo = __builtin_amdgcn_readfirstlane(fl.x), fhi = __builtin_amdgcn_readfirstlane(fl.y);
      while (cur < 512) {
        const int tile = tile_of(cur >> 2), sw = cur & 3;
        if (fhi < call && !bad) {
          drain();
          u64* flag = (u64*)(myws + OF_OFF + (unsigned)(src * 128 + tile) * 64 + (sw >> 1) * 8);
          u64 t0 = __builtin_amdgcn_s_memrealtime();
          for (;;) {
            const u64 f = __hip_atomic_load(flag, __ATOMIC_RELAXED, __HIP_MEMORY_SCOPE_SYSTEM);
            flo = __builtin_amdgcn_readfirstlane((unsigned)f);
            fhi = __builtin_amdgcn_readfirstlane((unsigned)(f >> 32));
            if (fhi >= call) break;
            if (__builtin_amdgcn_readfirstlane(timed_out(tmo, call, t0))) { bad = true; break; }
            __builtin_amdgcn_s_sleep(2);
          }
        }
        if (bad) flo = 0;
        tk = 1;
        asm volatile("s_nop 4\n\tbuffer_atomic_add %0, %1, %2, 0 offen sc0" : "+v"(tk) : "v"(nxt < 512 ? qoff : oob), "s"(mrs) : "memory");
        asm volatile("buffer_load_dwordx2 %0, %1, %2, 0 offen sc0 sc1" : "=v"(fl) : "v"(nxt < 512 ? OF_OFF + (unsigned)(src * 128 + tile_of(nxt >> 2)) * 64 + (nxt & 2) * 4 : oob), "s"(mrs) : "memory");
        k += 2;
        next_slot();
        const unsigned nv = min((flo >> (14 * (sw & 1))) & 0x3fffu, 4096u);
        const unsigned nb = seg_bytes(nv);
        const unsigned np = (nb + 1023) >> 10;
        LDS_ST(rdesc + slot, ((u64)(((sq + 1) << 16) | (bad ? 0x8000u : 0u) | (unsigned)(tile * 4 + sw)) << 32) | (nv << 16) | (unsigned)(k + np));
        unsigned vo = OB_OFF + (unsigned)(tile * 4 + sw) * SEG + l16;
        const unsigned vend = OB_OFF + (unsigned)(tile * 4 + sw) * SEG + nb;
        unsigned la = ldsbase + slot * SEG;
        for (unsigned i = 0; i < np; ++i) {
          unsigned keep;
          asm volatile("s_waitcnt vmcnt(%0)" :: "i"(ODEPTH - 1) : "memory");
          land = max(land, k - (ODEPTH - 1));
          LDS_ST(rland, (unsigned)land);
          asm volatile("s_mov_b32 %0, m0\n\ts_mov_b32 m0, %3\n\ts_nop 0\n\tbuffer_load_dwordx4 %1, %2, 0 offen sc0 sc1 lds\n\ts_mov_b32 m0, %0"
                       : "=&s"(keep) : "v"(vo < vend ? vo : oob), "s"(srs), "s"(la) : "memory");
          vo += 1024; la += 1024; ++k;
        }
        ++sq;
        if (++slot == NSLOT) slot = 0;
        if (np >= 5) asm volatile("s_waitcnt vmcnt(5)" ::: "memory");
        else if (np == 4) asm volatile("s_waitcnt vmcnt(4)" ::: "memory");
        else asm volatile("s_waitcnt vmcnt(1)" ::: "memory");
        asm volatile("" : "+v"(fl), "+v"(tk) :: "memory");
        flo = __builtin_amdgcn_readfirstlane(fl.x);
        fhi = __builtin_amdgcn_readfirstlane(fl.y);
        const unsigned tku = (unsigned)__builtin_amdgcn_readfirstlane(tk);
        const int nn = (nxt < 512 && tku < 512u) ? 2 * NIN + (int)tku : 512;
        cur = nxt;
        nxt = nn;
      }
      drain();
      for (int e = 0; e < 3; ++e) {
        next_slot();
        LDS_ST(rdesc + slot, (u64)(((sq + 1) << 16) | 0x3ffu) << 32);
        ++sq;
        if (++slot == NSLOT) slot = 0;
      }
    }
    if (__hip_atomic_load(tmo, __ATOMIC_RELAXED, __HIP_MEMORY_SCOPE_AGENT) == call && lane == 0)
      p.out[(size_t)((src >> 2) * 2048) * DIM + (src & 3) * 1024 + wi] = 0x7fc0;
    return;
  }

  const int gb = blockIdx.x - 128;
  const int ct = gb & 7, rt = gb >> 3;
  const int wm = wv >> 1, hf = wv & 1;
  LDS_AS u64* ctl = (LDS_AS u64*)(lds + LDS_CTRL + 32);
  unsigned short* o = p.out + (size_t)((rank >> 2) * 2048 + rt * 128 + wm * 64 + r16) * DIM + (rank & 3) * 1024 + ct * 128 + 64 * hf + 8 * g4;

  f32x4 acc[8][8];
  {
    const unsigned bx = XS_OFF + (unsigned)rt * (NKS * KSB) + lane * 16, bw = WS_OFF + (unsigned)ct * (NKS * KSB) + lane * 16;
    const unsigned vx0 = bx + wm * 8192, vx1 = bx + (wm ^ 1) * 8192;
    const unsigned vw0 = bw + hf * 4096, vw1 = bw + (hf ^ 1) * 4096;
#pragma unroll
    for (int a = 0; a < 8; ++a)
#pragma unroll
      for (int b = 0; b < 8; ++b) acc[a][b] = f32x4{0.f, 0.f, 0.f, 0.f};
    u32x4 fw[3][8], fx[3][8];
#define LDW(dst, vo, so, imm) asm volatile("buffer_load_dwordx4 %0, %1, %2, %3 offen offset:%4 sc1" : "=v"(dst) : "v"(vo), "s"(mrs), "s"(so), "i"(imm))
#define LDX(dst, vo, so, imm) asm volatile("buffer_load_dwordx4 %0, %1, %2, %3 offen offset:%4 sc1" : "=v"(dst) : "v"(vo), "s"(mrs), "s"(so), "i"(imm))
#define MF(T, M, w_, x_) asm volatile("v_mfma_f32_16x16x32_bf16 %0, %1, %2, %0" : "+a"(acc[T][M]) : "v"(w_[T]), "v"(x_[M]))
#define WAITF(w_, x_, n) asm volatile("s_waitcnt vmcnt(" #n ")" : "+v"(w_[0]), "+v"(w_[1]), "+v"(w_[2]), "+v"(w_[3]), "+v"(w_[4]), "+v"(w_[5]), "+v"(w_[6]), "+v"(w_[7]), \
                   "+v"(x_[0]), "+v"(x_[1]), "+v"(x_[2]), "+v"(x_[3]), "+v"(x_[4]), "+v"(x_[5]), "+v"(x_[6]), "+v"(x_[7]))
    auto load1 = [&](int i, unsigned sb, int kk, u32x4* w_, u32x4* x_) {
      const int q = i >> 2, j = i & 3;
      if (q == 0) LDW(w_[j], vw0, sb + kk * 8192, j * 1024);
      else if (q == 1) LDX(x_[j], vx0, sb + (j >> 1) * 4096, (j & 1) * 2048 + kk * 1024);
      else if (q == 2) LDW(w_[4 + j], vw1, sb + kk * 8192, j * 1024);
      else LDX(x_[4 + j], vx1, sb + (j >> 1) * 4096, (j & 1) * 2048 + kk * 1024);
    };
    auto half = [&](u32x4* w_, u32x4* x_, bool doload, unsigned sb, int kk, u32x4* wn, u32x4* xn) {
#pragma unroll
      for (int i = 0; i < 16; ++i) {
        const int qa = i >> 3, qb = (i >> 2) & 1, m = i & 3;
#pragma unroll
        for (int t = 0; t < 4; ++t) MF((qa ^ qb) * 4 + t, qa * 4 + m, w_, x_);
        if (doload) load1(i, sb, kk, wn, xn);
      }
    };
    const unsigned foff = IF_OFF + (unsigned)(((lane & 16) ? ct : 8 + rt) * 64 + ((((wv - lane) & 3) + ((lane & 8) >> 1)) << 3) + (lane & 7)) * 4;
    unsigned todo = 0xffffu, fv = 0;
    auto pick = [&]() -> unsigned {
      u64 rd = __ballot(fv >= call);
      unsigned m = (unsigned)rd & (unsigned)(rd >> 16) & todo;
      if (m == 0) {
        u64 t0 = __builtin_amdgcn_s_memrealtime();
        for (;;) {
          const unsigned v = __hip_atomic_load((unsigned*)(myws + foff), __ATOMIC_RELAXED, __HIP_MEMORY_SCOPE_SYSTEM);
          rd = __ballot(v >= call);
          m = (unsigned)rd & (unsigned)(rd >> 16) & todo;
          if (m) break;
          if (__builtin_amdgcn_readfirstlane(timed_out(tmo, call, t0))) { m = todo; break; }
          __builtin_amdgcn_s_sleep(4);
        }
      }
      const unsigned j = __builtin_ctz(m);
      todo &= ~(1u << j);
      return ((((((unsigned)wv - j) & 3u) + ((j & 8u) >> 1)) << 3) + (j & 7u)) * KSB;
    };
#define LDF() asm volatile("buffer_load_dword %0, %1, %2, 0 offen sc0 sc1" : "=v"(fv) : "v"(foff), "s"(mrs))
    {
      const unsigned sb = pick();
      asm volatile("s_waitcnt vmcnt(0)\n\ts_nop 4" ::: "memory");
      LDF();
#pragma unroll
      for (int i = 0; i < 16; ++i) load1(i, sb, 0, fw[0], fx[0]);
#pragma unroll
      for (int i = 0; i < 16; ++i) load1(i, sb, 1, fw[1], fx[1]);
    }
    for (int s = 0; s < 5; ++s) {
#pragma unroll
      for (int u = 0; u < 6; u += 2) {
        WAITF(fw[u % 3], fx[u % 3], 16);
        asm volatile("" : "+v"(fv));
        const unsigned sb = pick();
        asm volatile("s_nop 4");
        LDF();
        half(fw[u % 3], fx[u % 3], true, sb, 0, fw[(u + 2) % 3], fx[(u + 2) % 3]);
        WAITF(fw[(u + 1) % 3], fx[(u + 1) % 3], 17);
        half(fw[(u + 1) % 3], fx[(u + 1) % 3], true, sb, 1, fw[u % 3], fx[u % 3]);
      }
    }
    WAITF(fw[0], fx[0], 16);
    half(fw[0], fx[0], false, 0, 0, fw[0], fx[0]);
    WAITF(fw[1], fx[1], 0);
    half(fw[1], fx[1], false, 0, 0, fw[0], fx[0]);
    asm volatile("" : "+v"(fv));
    asm volatile("s_nop 15\n\ts_nop 15" ::: "memory");
    __builtin_amdgcn_sched_barrier(0);
  }
  {
    LDS_AS f32x4* ex = (LDS_AS f32x4*)lds;
#pragma unroll
    for (int t = 0; t < 4; ++t)
#pragma unroll
      for (int m = 0; m < 8; ++m) ex[(wv * 32 + t * 8 + m) * 64 + lane] = acc[4 + t][m];
    if (__hip_atomic_load(tmo, __ATOMIC_RELAXED, __HIP_MEMORY_SCOPE_AGENT) == call) cnt[4] = 1;
    __syncthreads();
    if (cnt[4]) {
      const u32x4 nanv = {0x7fc07fc0u, 0x7fc07fc0u, 0x7fc07fc0u, 0x7fc07fc0u};
#pragma unroll
      for (int c = 0; c < 8; ++c) *(u32x4*)(o + (size_t)(c >> 1) * 16 * DIM + (c & 1) * 32) = nanv;
      return;
    }
#pragma unroll
    for (int t = 0; t < 4; ++t)
#pragma unroll
      for (int m = 0; m < 8; ++m) acc[t][m] += ex[((wv ^ 1) * 32 + t * 8 + m) * 64 + lane];
    __syncthreads();
#pragma unroll
    for (int t = 0; t < 4; ++t)
#pragma unroll
      for (int m = 0; m < 4; ++m) ex[(64 + wv * 16 + t * 4 + m) * 64 + lane] = acc[t][4 + m];
    __syncthreads();
#pragma unroll
    for (int t = 0; t < 4; ++t)
#pragma unroll
      for (int m = 0; m < 4; ++m) acc[t][m] += ex[(64 + (wv ^ 2) * 16 + t * 4 + m) * 64 + lane];
  }

  {
    LDS_AS char* seg = (LDS_AS char*)(lds + wv * SEG);
    u32x4 pk[8];
    unsigned E[2] = {0u, 0u};
#pragma unroll
    for (int c = 0; c < 8; ++c) {
      const int mt = c >> 1, pc = c & 1;
      float v[8];
#pragma unroll
      for (int e = 0; e < 8; ++e) {
        const float a = acc[e & 3][mt][2 * pc + (e >> 2)];
        v[e] = (a <= 0.f) ? 0.f : a;
      }
      pk[c] = u32x4{pack_bf16(v[0], v[1]), pack_bf16(v[2], v[3]), pack_bf16(v[4], v[5]), pack_bf16(v[6], v[7])};
      pk[c] &= 0x7fff7fffu;
      __builtin_amdgcn_raw_buffer_store_b128(pk[c], ors, (unsigned)((char*)o - (char*)p.out) + pc * 64, mt * (16 * DIM * 2), OUT_AUX);
#pragma unroll
      for (int dd = 0; dd < 4; ++dd) {
        const unsigned t = pk[c][dd];
        unsigned nz;
        asm("v_pk_min_u16 %0, %1, 1 op_sel_hi:[1,0]" : "=v"(nz) : "v"(t));
        E[c >> 2] |= nz << ((c & 3) * 4 + dd);
      }
    }
    const unsigned cn = __builtin_popcount(E[0]) + __builtin_popcount(E[1]);
    const unsigned inc = wave_scan(cn);
    const unsigned total = __builtin_amdgcn_readlane(inc, 63);
    const unsigned H = total >> 5;
    {
      const unsigned xo = ((2 * total + 3) & ~3u) - 4 * H;
      *(LDS_AS u32x2*)(seg + SEG_TBL + lane * 8) = u32x2{2u * lane < H ? E[0] : 0u, 2u * lane + 1 < H ? E[1] : 0u};
      if (lane == 63) *(LDS_AS unsigned short*)(seg + 2 * total) = 0;
      if (2u * lane >= H) *(LDS_AS unsigned*)(seg + xo + lane * 8) = E[0];
      if (2u * lane + 1 >= H) *(LDS_AS unsigned*)(seg + xo + lane * 8 + 4) = E[1];
    }
    unsigned wa = (unsigned)(size_t)seg + 2 * (inc - cn);
#pragma unroll
    for (int c = 0; c < 8; ++c)
#pragma unroll
      for (int dd = 0; dd < 4; ++dd)
        asm volatile("v_cmpx_ne_u16 vcc, 0, %1\n\t"
                     "ds_write_b16 %0, %1\n\t"
                     "v_add_u32 %0, 2, %0\n\t"
                     "s_mov_b64 exec, -1\n\t"
                     "v_cmpx_lt_u32 vcc, %2, %1\n\t"
                     "ds_write_b16_d16_hi %0, %1\n\t"
                     "v_add_u32 %0, 2, %0\n\t"
                     "s_mov_b64 exec, -1"
                     : "+v"(wa) : "v"(pk[c][dd]), "s"(0xffffu) : "vcc", "memory");
    asm volatile("s_waitcnt lgkmcnt(0)" ::: "memory");
    const unsigned nb = seg_bytes(total);
    const unsigned so = OB_OFF + (unsigned)(gb * 4 + wv) * SEG + lane * 16;
#pragma unroll
    for (int i = 0; i < 9; ++i) {
      u32x4 v = *(LDS_AS u32x4*)(seg + i * 1024 + lane * 16);
      if (i < 8) {
        const unsigned e = *(LDS_AS unsigned*)(seg + SEG_TBL + i * 64 + (lane >> 2) * 4) >> ((lane & 3) * 4);
#pragma unroll
        for (int d = 0; d < 4; ++d) v[d] |= (e << (15 - d)) & 0x80008000u;
      }
      __builtin_amdgcn_raw_buffer_store_b128(v, mrs, (unsigned)(i * 1024 + lane * 16) < nb ? so + i * 1024 : 0xF0000000u, 0, AUX_SYS);
    }
    asm volatile("s_waitcnt vmcnt(0)" ::: "memory");
    u64 old = 0;
    const u64 mine = (1ull << 56) | ((u64)total << (14 * wv));
    if (lane == 0) old = __hip_atomic_fetch_add(ctl, mine, __ATOMIC_RELAXED, __HIP_MEMORY_SCOPE_WORKGROUP);
    old = (((u64)(unsigned)__builtin_amdgcn_readfirstlane((unsigned)(old >> 32)) << 32) | (unsigned)__builtin_amdgcn_readfirstlane((unsigned)old)) + mine;
    if ((old >> 56) == 4 && lane < 16 && (lane & 7) != rank) {
      const u64 f = ((u64)call << 32) | (unsigned)((old >> (28 * (lane >> 3))) & 0xfffffffu);
      __hip_atomic_store((u64*)(p.ws[lane & 7] + OF_OFF + (unsigned)(rank * 128 + gb) * 64 + (lane >> 3) * 8), f, __ATOMIC_RELAXED, __HIP_MEMORY_SCOPE_SYSTEM);
    }
  }
}

}

extern "C" void kernel_launch(int rank, int nranks, ncclComm_t comm,
                              hipStream_t stream,
                              void* const* d_in, const int* in_sizes, int n_in,
                              void* d_out, int out_size,
                              const int* in_dtypes, int out_dtype,
                              void* const* d_out_all,
                              void* const* d_ws_all, size_t ws_bytes) {
  (void)comm; (void)in_sizes; (void)n_in; (void)out_size; (void)in_dtypes; (void)out_dtype; (void)d_out_all;
  if (nranks != NR || (unsigned)rank >= (unsigned)NR || ws_bytes < (size_t)WS_USED) {
    fprintf(stderr, "unsupported configuration\n");
    return;
  }
  Params p;
  p.x = (const unsigned short*)d_in[0];
  p.w = (const unsigned short*)d_in[1];
  p.out = (unsigned short*)d_out;
  for (int i = 0; i < NR; ++i) p.ws[i] = (char*)d_ws_all[i];
  p.myws = p.ws[rank];
  p.rank = rank;
  constexpr OrderTable kOrder = make_order();
  constexpr PrepTable kPrep = make_prep();
  p.order = kOrder;
  p.prep = kPrep;
  gemm_ar_kernel<<<256, 256, 0, stream>>>(p);
  HIP_CHECK(hipGetLastError());
}
```

```cpp
#include <hip/hip_runtime.h>
#include <rccl/rccl.h>
#include <cstdio>
#include <cstdint>

#define HIP_CHECK(expr) do { hipError_t _e = (expr); if (_e != hipSuccess) { \
    fprintf(stderr, "HIP error %s at %s:%d\n", hipGetErrorString(_e), __FILE__, __LINE__); \
    return; } } while (0)

namespace {

typedef float f32x4 __attribute__((ext_vector_type(4)));
typedef unsigned u32x4 __attribute__((ext_vector_type(4)));
typedef unsigned u32x2 __attribute__((ext_vector_type(2)));
typedef unsigned long long u64;

constexpr int NR = 8;
constexpr int DIM = 4096;
constexpr int KSL = DIM / NR;
constexpr int NKS = 32;
constexpr unsigned KSB = 16384;
constexpr unsigned SEG = 9216;
constexpr unsigned SEG_TBL = 8704;
__host__ __device__ constexpr unsigned seg_bytes(unsigned n) { return ((2 * n + 3) & ~3u) + 4 * (128 - (n >> 5)); }

constexpr unsigned WS_OFF = 0;
constexpr unsigned PS_OFF = WS_OFF + 32 * NKS * KSB;
constexpr unsigned SB_OFF = PS_OFF + 128 * 2 * KSB;
constexpr unsigned PB_OFF = SB_OFF + 8 * 2 * 32 * KSB;
constexpr unsigned OB_OFF = PB_OFF + 128 * 2 * KSB;
constexpr unsigned IF_OFF = OB_OFF + 128 * 4 * SEG;
constexpr unsigned PF_OFF = IF_OFF + 2560 * 4;
constexpr unsigned OF_OFF = PF_OFF + NR * 256 * 4;
constexpr unsigned TK_OFF = OF_OFF + NR * 640 * 8;
constexpr unsigned QC_OFF = TK_OFF + 32 * 128;
constexpr unsigned TMO_OFF = QC_OFF + 64 * 128;
constexpr unsigned DBG_OFF = TMO_OFF + 4096;
constexpr unsigned LOG_OFF = DBG_OFF + 256 * 4 * 16 * 8;
constexpr unsigned WS_USED = LOG_OFF + 112 * 128 * 8;

#define NINV 16
#define ODEPTHV 4
constexpr int NIN = NINV;
constexpr int ODEPTH = ODEPTHV;
constexpr int NMOV = 7 * NIN, NG = 256 - NMOV;
constexpr int NSLOT = 12;
#define NEARLY 64u
#define PDELAY 40
#define PSTEP 12
#define PSLEEP 4
#define PFN 4
#define MSPIN 8
#define PD2 60
#define MU 1

constexpr u64 DEADLINE_TICKS = 50000000ull;

constexpr int LDS_CTRL = 131072;
constexpr int LDS_BYTES = LDS_CTRL + 256;

__host__ __device__ constexpr int row_tile(int g, int h, int r) { return (h << (4 - g)) | ((r & 1) << (3 + g)) | (((r >> 3) ^ g) << 2) | ((r >> 1) & 3); }
__host__ __device__ constexpr int col_tile(int g, int h, int c) { return (h << (4 - g)) | ((c & 1) << (3 + g)) | (c >> 1); }

#define SEQ 16,0,17,1,2,3,18,19,4,5,24,25,6,7,20,21,8,9,26,27,10,11,22,23,12,13,28,29,14,15,30,31
struct PosTable { int p[32]; };
constexpr PosTable make_pos() {
  const int seq[32] = {SEQ};
  PosTable o{};
  for (int i = 0; i < 32; ++i) o.p[seq[i]] = i;
  return o;
}
constexpr PosTable kPos = make_pos();
constexpr int unit_pos(int u) { return kPos.p[u]; }
constexpr int cmax(int a, int b) { return a > b ? a : b; }
#define PLAG 6
constexpr int send_key(int r, int c) { return cmax(unit_pos(16 + r), unit_pos(c)); }
constexpr int kept_key(int kt) {
  return cmax(cmax(unit_pos(24 + (kt >> 4)), unit_pos(kt & 15)),
              cmax(send_key((kt >> 4) | 1, (kt & 15) | 1), send_key((kt >> 4) & 6, kt & 14)) + PLAG);
}
struct alignas(4) TileTable { unsigned char wg[2 * NG]; };
constexpr TileTable make_tiles() {
  TileTable o{};
  int nj[8] = {};
  for (int key = 0; key < 64; ++key)
    for (int t = 0; t < 256; ++t) {
      const int x = t & 7;
      if ((t < 128 ? send_key(t >> 4, t & 15) : kept_key(t & 127)) == key) {
        o.wg[2 * ((nj[x] % (NG / 8)) * 8 + x) + nj[x] / (NG / 8)] = (unsigned char)t;
        ++nj[x];
      }
    }
  return o;
}
struct alignas(4) OrderTable { unsigned char t[2][160]; };
constexpr OrderTable make_order() {
  OrderTable o{};
  for (int x = 0; x < 2; ++x) {
    int n = 0;
    const int nk = x ? 3 : 5;
    for (int key = 0; key <= 31 + MU * 4; ++key)
      for (int k = nk - 1; k >= 0; --k)
        for (int u = 0; u < 32; ++u)
          if (unit_pos(u) + MU * k == key) o.t[x][n++] = (unsigned char)(u * 8 + k);
  }
  return o;
}
#define PK 2
constexpr int npush(int nk) {
  int n = 0;
  for (int u = 0; u < 32; ++u)
    for (int k = 0; k < nk; ++k) n += unit_pos(u) + MU * k <= PK;
  return n;
}
struct alignas(4) PrepTable { unsigned char t[256]; unsigned push[16]; };
constexpr PrepTable make_prep() {
  PrepTable o{};
  int n = 0;
  for (int key = 0; key <= 5 * (31 + MU * 4); ++key)
    for (int j = 0; j < 256; ++j) {
      const int ksl = j >> 5, t = j & 15, a = unit_pos(t), b = unit_pos(16 + t), k = MU * (ksl < 5 ? ksl : ksl - 5);
      if (((a < b ? a : b) + k) * (ksl < 5 ? 5 : 3) == key) {
        o.push[n >> 4] |= (unsigned)((a + k <= PK) + 2 * (b + k <= PK)) << (2 * (n & 15));
        o.t[n++] = (unsigned char)j;
      }
    }
  return o;
}

struct Params {
  const unsigned short* x;
  const unsigned short* w;
  unsigned short* out;
  char* myws;
  char* ws[NR];
  int rank;
  OrderTable order;
  PrepTable prep;
  TileTable tiles;
};

#define LDS_AS __attribute__((address_space(3)))

__device__ __forceinline__ __amdgpu_buffer_rsrc_t make_rsrc(const void* p, unsigned bytes) {
  return __builtin_amdgcn_make_buffer_rsrc((void*)p, 0, bytes, 0x00020000);
}
constexpr int AUX_SYS = 17;
constexpr int OUT_AUX = 16;

__device__ __forceinline__ bool timed_out(unsigned* tmo, unsigned call, u64 t0) {
  if (__hip_atomic_load(tmo, __ATOMIC_RELAXED, __HIP_MEMORY_SCOPE_AGENT) == call ||
      __builtin_amdgcn_s_memrealtime() - t0 > DEADLINE_TICKS) {
    __hip_atomic_store(tmo, call, __ATOMIC_RELAXED, __HIP_MEMORY_SCOPE_AGENT);
    return true;
  }
  return false;
}

__device__ __forceinline__ unsigned pack_bf16(float a, float b) {
  typedef __bf16 bf2 __attribute__((ext_vector_type(2)));
  typedef float f2 __attribute__((ext_vector_type(2)));
  f2 v = {a, b};
  bf2 r = __builtin_convertvector(v, bf2);
  return __builtin_bit_cast(unsigned, r);
}

__device__ __forceinline__ unsigned wave_scan(unsigned x) {
  x += __builtin_amdgcn_update_dpp(0u, x, 0x111, 0xf, 0xf, false);
  x += __builtin_amdgcn_update_dpp(0u, x, 0x112, 0xf, 0xf, false);
  x += __builtin_amdgcn_update_dpp(0u, x, 0x114, 0xf, 0xf, false);
  x += __builtin_amdgcn_update_dpp(0u, x, 0x118, 0xf, 0xf, false);
  x += __builtin_amdgcn_update_dpp(0u, x, 0x142, 0xa, 0xf, false);
  x += __builtin_amdgcn_update_dpp(0u, x, 0x143, 0xc, 0xf, false);
  return x;
}
typedef unsigned short u16x2 __attribute__((ext_vector_type(2)));

#define STAMP(i) do { } while (0)

__global__ void __launch_bounds__(256, 1) gemm_ar_kernel(Params p) {
  __shared__ __attribute__((aligned(16))) char lds[LDS_BYTES];
  const int tid = threadIdx.x;
  const int lane = tid & 63;
  const int wv = __builtin_amdgcn_readfirstlane(tid >> 6);
  const int rank = p.rank;
  char* myws = p.myws;
  unsigned* tmo = (unsigned*)(myws + TMO_OFF);
  const int r16 = lane & 15, g4 = lane >> 4;

  const int gq = rank >> 2, ri = (rank >> 1) & 1, rj = rank & 1;
  STAMP(0);
  __amdgpu_buffer_rsrc_t mrs = make_rsrc(myws, WS_USED);
  __amdgpu_buffer_rsrc_t ors = make_rsrc(p.out, (unsigned)DIM * DIM * 2);
  LDS_AS unsigned* cnt = (LDS_AS unsigned*)(lds + LDS_CTRL);
  unsigned call;
  {
    const unsigned job = (blockIdx.x + NG) & 255u;
    unsigned jobw = ((const unsigned*)p.prep.t)[job >> 2];
    const unsigned jobw2 = ((const unsigned*)p.prep.t)[((job + NMOV) & 255u) >> 2];
    unsigned push = p.prep.push[job >> 4];
    const unsigned short* const px = p.x;
    const unsigned short* const pw = p.w;
    asm volatile("" :: "s"(px), "s"(pw), "s"(jobw), "s"(jobw2), "s"(push), "s"(rank), "s"(myws));
    unsigned ticket = 1;
    asm volatile("buffer_atomic_add %0, %1, %2, 0 offen sc0" : "+v"(ticket) : "v"(lane == 0 ? TK_OFF + (blockIdx.x & 31) * 128 : 0xF0000000u), "s"(mrs) : "memory");
    char* wsl = p.ws[0];
#pragma unroll
    for (int i = 1; i < NR; ++i) wsl = (lane & 7) == i ? p.ws[i] : wsl;
    const int cbr = 8 * ((r16 & 3) >> 1) + 2 * (r16 >> 2) + (r16 & 1);
    auto load_w = [&](u32x4* r, int t, int k, int qt) {
      const unsigned short* q = pw + (size_t)(k * 64 + (qt >> 1) * 32 + 8 * g4) * DIM + t * 128 + 64 * (qt & 1) + 4 * cbr;
#pragma unroll
      for (int h = 0; h < 4; ++h) {
        const u32x2 a = *(const u32x2*)(q + (size_t)(2 * h) * DIM), b = *(const u32x2*)(q + (size_t)(2 * h + 1) * DIM);
        r[h] = u32x4{a.x, a.y, b.x, b.y};
      }
    };
    auto load_x = [&](u32x4* r, int t, int k, int qt) {
      const unsigned short* q = px + (size_t)(t * 128 + qt * 32 + r16) * KSL + k * 64 + 8 * g4;
#pragma unroll
      for (int h = 0; h < 4; ++h) r[h] = *(const u32x4*)(q + (size_t)(h >> 1) * 16 * KSL + (h & 1) * 32);
    };
    auto transpose_w = [&](u32x4* r) {
      u32x4 v[4];
#pragma unroll
      for (int t = 0; t < 4; ++t)
#pragma unroll
        for (int h = 0; h < 4; ++h) {
          const unsigned lo = r[h][t >> 1], hi = r[h][2 + (t >> 1)];
          v[t][h] = __builtin_amdgcn_perm(hi, lo, (t & 1) ? 0x07060302u : 0x05040100u);
        }
#pragma unroll
      for (int t = 0; t < 4; ++t) r[t] = v[t];
    };
    if (tid < 64) cnt[tid] = 0;
    asm volatile("s_waitcnt lgkmcnt(0)" ::: "memory");
    __builtin_amdgcn_s_barrier();
    const int njob = blockIdx.x < NMOV ? 0 : job + NMOV >= NG ? 2 : 1;
    push = (push >> (2 * (job & 15))) & 3u;
#pragma unroll 1
    for (int jn = 0; jn < njob; ++jn) {
      const int jb = (jobw >> (8 * (job & 3))) & 255, ksl = jb >> 5, hh = (jb >> 4) & 1, tt = jb & 15;
      const int tg = ksl < 5 ? gq : gq ^ 1, kq = ksl < 5 ? ksl : ksl - 5;
      const int gtw = col_tile(tg, hh, tt), gtx = row_tile(tg, hh, tt);
      if (jn == 0 && job >= NEARLY) {
        __builtin_amdgcn_s_sleep(PDELAY);
        for (unsigned i = NEARLY; i < job; i += 8) __builtin_amdgcn_s_sleep(PSTEP);
      }
      if (jn) __builtin_amdgcn_s_sleep(PD2);
      u32x4 wr[4], xr[4];
      load_w(wr, gtw, ksl, wv);
      load_x(xr, gtx, ksl, wv);
      const unsigned sb = SB_OFF + (unsigned)((ksl * 2 + hh) * 32 + tt) * KSB + wv * 4096 + lane * 16;
      const unsigned lb = WS_OFF + (unsigned)(tt * NKS + (rank & 3) * 5 + kq) * KSB + wv * 4096 + lane * 16;
      const unsigned lw = (ksl < 5 && hh == rj) ? lb : 0xF0000000u, lx = (ksl < 5 && hh == ri) ? lb + 16 * NKS * KSB : 0xF0000000u;
      transpose_w(wr);
#pragma unroll
      for (int t = 0; t < 4; ++t) __builtin_amdgcn_raw_buffer_store_b128(wr[t], mrs, sb + t * 1024, 0, AUX_SYS);
#pragma unroll
      for (int h = 0; h < 4; ++h) __builtin_amdgcn_raw_buffer_store_b128(xr[h], mrs, sb + 16 * KSB + h * 1024, 0, AUX_SYS);
#pragma unroll
      for (int t = 0; t < 4; ++t) __builtin_amdgcn_raw_buffer_store_b128(wr[t], mrs, lw + t * 1024, 0, AUX_SYS);
#pragma unroll
      for (int h = 0; h < 4; ++h) __builtin_amdgcn_raw_buffer_store_b128(xr[h], mrs, lx + h * 1024, 0, AUX_SYS);
      const unsigned ks = ksl < 5 ? (rank & 3) * 5 + kq : 20 + (rank & 3) * 3 + kq;
      if (push) {
        asm volatile("" ::: "memory");
#pragma unroll
        for (int i = 0; i < 4; ++i) {
          const int q = tg * 4 + (i < 2 ? i * 2 + hh : hh * 2 + i - 2);
          const __amdgpu_buffer_rsrc_t prs = make_rsrc(p.ws[q], WS_USED);
          const unsigned po = ((push >> (i >> 1)) & 1) && q != rank ? WS_OFF + (unsigned)((tt + (i >> 1) * 16) * NKS + ks) * KSB + wv * 4096 + lane * 16 : 0xF0000000u;
#pragma unroll
          for (int h = 0; h < 4; ++h) __builtin_amdgcn_raw_buffer_store_b128(i < 2 ? wr[h] : xr[h], prs, po + h * 1024, 0, AUX_SYS);
        }
        asm volatile("s_waitcnt vmcnt(16)" ::: "memory");
      } else {
        asm volatile("s_waitcnt vmcnt(0)" ::: "memory");
      }
      asm volatile("" : "+v"(ticket) :: "memory");
      call = (unsigned)__builtin_amdgcn_readfirstlane(ticket) / 32u + 1u;
      const int q = lane & 7;
      const bool need = lane < 16 && (q >> 2) == tg && (lane < 8 ? (q & 1) : ((q >> 1) & 1)) == hh;
      const int unit = tt + (lane & 8) * 2;
      auto last = [&](int ci) -> bool {
        unsigned old = 0;
        if (lane == 0) old = __hip_atomic_fetch_add(cnt + ci, 1u, __ATOMIC_RELAXED, __HIP_MEMORY_SCOPE_WORKGROUP);
        return __builtin_amdgcn_readfirstlane(old) == 3;
      };
      if (last(jn) && need) {
        if (q == rank)
          __hip_atomic_store((u64*)(myws + IF_OFF) + unit * 32 + ks, ((u64)call << 32) | call, __ATOMIC_RELAXED, __HIP_MEMORY_SCOPE_SYSTEM);
        else
          __hip_atomic_store((unsigned*)(wsl + PF_OFF) + rank * 256 + unit * 8 + kq, call, __ATOMIC_RELAXED, __HIP_MEMORY_SCOPE_SYSTEM);
      }
      if (push) {
        asm volatile("s_waitcnt vmcnt(0)" ::: "memory");
        if (last(2) && need && q != rank && ((push >> (lane >> 3)) & 1))
          __hip_atomic_store((u64*)(wsl + IF_OFF) + unit * 32 + ks, ((u64)call << 32) | call, __ATOMIC_RELAXED, __HIP_MEMORY_SCOPE_SYSTEM);
      }
      push = 0;
      jobw = jobw2;
    }
    asm volatile("s_waitcnt vmcnt(0)" : "+v"(ticket) :: "memory");
    call = (unsigned)__builtin_amdgcn_readfirstlane(ticket) / 32u + 1u;
    STAMP(1);
  }
  if (blockIdx.x < NMOV) {
    const int mb = blockIdx.x;
    const int bi = __builtin_amdgcn_readfirstlane((mb * 9363) >> 16);
    const int lk = __builtin_amdgcn_readfirstlane(mb - 7 * bi);
    const int src = (rank + 1 + lk) & 7;
    const int wi = bi;
    const int sgp = src >> 2, si = (src >> 1) & 1, sj = src & 1;
    unsigned short* const mark = p.out + (size_t)(row_tile(sgp, si, 8) * 128) * DIM + col_tile(sgp, sj, 0) * 128 + wi;
    const unsigned l16 = lane * 16;
    const unsigned oob = 0xF0000000u;
    LDS_AS u64* rdesc = (LDS_AS u64*)(lds + LDS_CTRL + 64);
    LDS_AS unsigned* rfree = (LDS_AS unsigned*)(lds + LDS_CTRL + 160);
    LDS_AS unsigned* rland = (LDS_AS unsigned*)(lds + LDS_CTRL + 208);
#define LDS_LD(ptr) __hip_atomic_load(ptr, __ATOMIC_RELAXED, __HIP_MEMORY_SCOPE_WORKGROUP)
#define LDS_ST(ptr, val) __hip_atomic_store(ptr, val, __ATOMIC_RELAXED, __HIP_MEMORY_SCOPE_WORKGROUP)
    const u64 tstart = __builtin_amdgcn_s_memrealtime();
    auto ring_dead = [&]() -> bool {
      if (__builtin_amdgcn_s_memrealtime() - tstart <= 3 * DEADLINE_TICKS) return false;
      __hip_atomic_store(tmo, call, __ATOMIC_RELAXED, __HIP_MEMORY_SCOPE_AGENT);
      if (lane == 0) *mark = 0x7fc0;
      return true;
    };
    const int cross = sgp != gq;
    LDS_AS unsigned* mbox = (LDS_AS unsigned*)(lds + LDS_CTRL + 216);
    if (wv == 3) {
      const unsigned ps = cross ? 2 * npush(3) : 2 * npush(5);
      const unsigned np = cross ? 128u : 0u, no = (cross ? 192u : 320u) - ps;
      const unsigned otab = ((const unsigned*)p.order.t[cross])[lane < 40 ? lane : 0];
      const unsigned ctr = QC_OFF + (unsigned)((call & 1) * 32 + lk + (lane & 3) * 8) * 128;
      unsigned tn = 1;
      asm volatile("buffer_atomic_add %0, %1, %2, 0 offen sc0\n\ts_waitcnt vmcnt(0)" : "+v"(tn) : "v"(lane < 3 ? ctr : oob), "s"(mrs) : "memory");
      unsigned cp = __builtin_amdgcn_readlane(tn, 0), co = __builtin_amdgcn_readlane(tn, 1), cq = __builtin_amdgcn_readlane(tn, 2);
      unsigned nx = 1;
      asm volatile("buffer_atomic_add %0, %1, %2, 0 offen sc0\n\ts_waitcnt vmcnt(0)" : "+v"(nx) : "v"(lane < 3 ? ctr : oob), "s"(mrs) : "memory");
      bool bad = false;
      int refill = -1;
      while (cq < 512u || co < no || cp < np) {
        const unsigned ob_ = (__builtin_amdgcn_readlane(otab, ((co + ps) >> 3) & 63) >> (8 * (((co + ps) >> 1) & 3))) & 255u;
        const unsigned fa = lane == 0 ? (cp < np ? OF_OFF + (unsigned)(src * 640 + 512 + cp) * 8 : oob)
                          : lane == 1 ? (co < no ? PF_OFF + (unsigned)(src * 256 + (ob_ & ~1u)) * 4 : oob)
                          : lane == 2 ? (cq < 512u ? OF_OFF + (unsigned)(src * 640 + cq) * 8 : oob) : oob;
        tn = 1;
        asm volatile("buffer_atomic_add %0, %1, %2, 0 offen sc0" : "+v"(tn) : "v"(lane == refill ? ctr : oob), "s"(mrs) : "memory");
        u64 t0 = __builtin_amdgcn_s_memrealtime();
        unsigned rd, fx;
        for (unsigned spin = 1;; ++spin) {
          const u32x2 f = __builtin_amdgcn_raw_buffer_load_b64(mrs, fa, 0, AUX_SYS);
          const unsigned ex = bad ? 4u : (unsigned)__ballot(((lane != 1 || (ob_ & 1u)) ? f.y : f.x) >= call) & 7u;
          fx = f.x;
          rd = 0;
          if (ex) {
            for (int w = cp < np ? MSPIN : 64; w > 0 && !rd; --w) {
              const u64 mb_ = LDS_LD((LDS_AS u64*)mbox);
              rd = __builtin_amdgcn_readfirstlane(ex & (((unsigned)mb_ ? 0u : 1u) | ((mb_ >> 32) ? 0u : 6u)));
            }
            if (rd) break;
          }
          if ((spin & 7) == 0 && !ex && __builtin_amdgcn_readfirstlane(timed_out(tmo, call, t0))) bad = true;
          if ((spin & 7) == 0 && ring_dead()) return;
          asm volatile("" ::: "memory");
        }
        asm volatile("s_waitcnt vmcnt(0)" : "+v"(tn) :: "memory");
        nx = lane == refill ? tn : nx;
        refill = -1;
        if (bad) {
          co = no; cp = np;
          fx = (cq >> 2) << 14;
          if (cq >= 512u) break;
        }
        unsigned item;
        if (rd & 1) {
          const unsigned tile = (__builtin_amdgcn_readlane(fx, 0) >> 14) & 127u, sw = cp & 3u;
          item = (1u << 30) | ((PB_OFF / 8192 + tile * 4 + sw) << 12) | (PS_OFF / 8192 + ((tile & 0x6eu) + si * 16 + sj) * 4 + sw);
          cp = __builtin_amdgcn_readlane(nx, 0); refill = 0;
        } else if (rd & 2) {
          const unsigned u = ob_ >> 3, kb = ob_ & 7u;
          item = (1u << 30) | ((SB_OFF / 8192 + (((kb + 5 * cross) * 2 + (u < 16 ? rj : ri)) * 32 + u) * 2 + (co & 1u)) << 12)
                 | ((u * 32 + (cross ? 20 + (src & 3) * 3 : (src & 3) * 5) + kb) * 2 + (co & 1u));
          co = __builtin_amdgcn_readlane(nx, 1); refill = 1;
        } else {
          const unsigned flo = __builtin_amdgcn_readlane(fx, 2);
          item = (min(flo & 0x3fffu, 4096u) << 12) | (((flo >> 14) & 127u) * 4 + (cq & 3u)) | (bad ? 1u << 29 : 0u);
          cq = __builtin_amdgcn_readlane(nx, 2); refill = 2;
        }
        LDS_ST(mbox + ((rd & 1) ^ 1), item | (1u << 28));
      }
      while (LDS_LD((LDS_AS u64*)mbox) != 0) {
        if (ring_dead()) return;
        __builtin_amdgcn_s_sleep(1);
      }
      LDS_ST(mbox + 1, 0x80000000u);
      if (bad && lane == 0) *mark = 0x7fc0;
      return;
    }
    if (wv != 0) {
      const unsigned ob = (unsigned)(r16 * DIM + 8 * g4) * 2;
      int slot = wv - 1;
      for (unsigned sq = wv - 1;; sq += 2) {
        u64 d;
        for (;;) {
          d = LDS_LD(rdesc + slot);
          if ((unsigned)(d >> 48) == sq + 1) break;
          if (ring_dead()) return;
          __builtin_amdgcn_s_sleep(4);
        }
        const unsigned dhi = __builtin_amdgcn_readfirstlane((unsigned)(d >> 32)), dend = __builtin_amdgcn_readfirstlane((unsigned)d);
        const unsigned sg = dhi & 0xfffu;
        if (sg == 0xfffu) break;
        const unsigned nv = dend >> 16, H = nv >> 5;
        while (LDS_LD(rland) < (dend & 0xffffu)) {
          if (ring_dead()) return;
          __builtin_amdgcn_s_sleep(1);
        }
        asm volatile("" ::: "memory");
        LDS_AS char* seg = (LDS_AS char*)(lds + 16 + slot * SEG);
        if (dhi & 0x4000u) {
          u32x4 v[8];
#pragma unroll
          for (int i = 0; i < 8; ++i) v[i] = *(LDS_AS u32x4*)(seg + i * 1024 + l16);
          LDS_ST(rfree + slot, sq + 1);
#pragma unroll
          for (int i = 0; i < 8; ++i) __builtin_amdgcn_raw_buffer_store_b128(v[i], mrs, sg * 8192 + i * 1024 + l16, 0, 16);
          asm volatile("s_waitcnt vmcnt(0)" ::: "memory");
          if (lane == 0) __hip_atomic_store((unsigned*)(myws + IF_OFF) + sg, call, __ATOMIC_RELAXED, __HIP_MEMORY_SCOPE_AGENT);
          slot += 2;
          if (slot >= NSLOT) slot -= NSLOT;
          continue;
        }
        auto unhide = [&](int i0, int n) {
          u32x4 v[5];
#pragma unroll
          for (int i = 0; i < n; ++i) v[i] = *(LDS_AS u32x4*)(seg + (i0 + i) * 1024 + l16);
#pragma unroll
          for (int i = 0; i < n; ++i) {
            unsigned e = ((v[i][0] >> 15) & 0x00010001u) | ((v[i][1] >> 14) & 0x00020002u) | ((v[i][2] >> 13) & 0x00040004u) | ((v[i][3] >> 12) & 0x00080008u);
            e <<= (lane & 3) * 4;
            e |= __builtin_amdgcn_update_dpp(0u, e, 0xb1, 0xf, 0xf, true);
            e |= __builtin_amdgcn_update_dpp(0u, e, 0x4e, 0xf, 0xf, true);
            *(LDS_AS unsigned*)(seg + SEG_TBL + (i0 + i) * 64 + (lane >> 2) * 4) = e;
          }
        };
        unhide(0, 5);
        if (H > 80) unhide(5, 3);
        const unsigned xo = ((2 * nv + 3) & ~3u) - 4 * H;
        u32x2 mk;
        mk.x = *(LDS_AS unsigned*)(seg + (2u * lane < H ? SEG_TBL : xo) + lane * 8);
        mk.y = *(LDS_AS unsigned*)(seg + (2u * lane + 1 < H ? SEG_TBL : xo) + lane * 8 + 4);
        const unsigned c0 = __builtin_popcount(mk.x), cn = c0 + __builtin_popcount(mk.y);
        const int b0 = 2 * (int)(wave_scan(cn) - cn) - 2, b1 = b0 + 2 * (int)c0;
        u32x4 ov[8], oh[8];
#pragma unroll
        for (int c = 0; c < 8; ++c)
#pragma unroll
          for (int dd = 0; dd < 4; ++dd) {
            const int dq = (c & 3) * 4 + dd;
            const unsigned E = c < 4 ? mk.x : mk.y;
            const unsigned pp = __builtin_popcount(E & (((2u << dq) - 1u) | (((1u << dq) - 1u) << 16)));
            const volatile LDS_AS unsigned short* q = (const volatile LDS_AS unsigned short*)(seg + (c < 4 ? b0 : b1) + 2 * (int)pp);
            ov[c][dd] = q[0];
            oh[c][dd] = q[1];
          }
        LDS_ST(rfree + slot, sq + 1);
        const int tile = sg >> 2, sw = sg & 3;
        const unsigned oo = ob + (unsigned)((row_tile(sgp, si, 8 + (tile >> 4)) * 128 + (sw >> 1) * 64) * DIM + col_tile(sgp, sj, tile & 15) * 128 + 64 * (sw & 1)) * 2;
        const unsigned poison = (dhi & 0x8000u) ? 0x7fc07fc0u : 0u;
#pragma unroll
        for (int c = 0; c < 8; ++c) {
#pragma unroll
          for (int dd = 0; dd < 4; ++dd) {
            const unsigned m = ((c < 4 ? mk.x : mk.y) >> ((c & 3) * 4 + dd)) & 0x00010001u;
            const unsigned t = (ov[c][dd] | (oh[c][dd] << 16)) & 0x7fff7fffu;
            const u16x2 r = __builtin_bit_cast(u16x2, t) * __builtin_bit_cast(u16x2, m);
            ov[c][dd] = __builtin_bit_cast(unsigned, r);
          }
          ov[c][0] |= poison;
          __builtin_amdgcn_raw_buffer_store_b128(ov[c], ors, oo + (c & 1) * 64, (c >> 1) * (16 * DIM * 2), OUT_AUX);
          __builtin_amdgcn_sched_barrier(0);
        }
        slot += 2;
        if (slot >= NSLOT) slot -= NSLOT;
      }
      STAMP(11);
      return;
    }
    __amdgpu_buffer_rsrc_t srs = make_rsrc(p.ws[src], WS_USED);
    bool bad = false;
    if (mb == 0 && tid < 32)
      __hip_atomic_store((unsigned*)(myws + QC_OFF + (((call + 1) & 1) * 32 + tid) * 128), 0u, __ATOMIC_RELAXED, __HIP_MEMORY_SCOPE_AGENT);
    {
      const unsigned ldsbase = (unsigned)(size_t)(LDS_AS char*)lds + 16;
      int k = 0, land = 0;
      unsigned sq = 0;
      int slot = 0;
      auto drain = [&]() {
        asm volatile("s_waitcnt vmcnt(0)" ::: "memory");
        land = k;
        LDS_ST(rland, (unsigned)k);
      };
      auto next_slot = [&]() {
        if (sq >= (unsigned)NSLOT && LDS_LD(rfree + slot) != sq - NSLOT + 1) {
          drain();
          while (LDS_LD(rfree + slot) != sq - NSLOT + 1) {
            if (ring_dead()) { bad = true; break; }
            __builtin_amdgcn_s_sleep(1);
          }
        }
      };
      for (;;) {
        u64 m = LDS_LD((LDS_AS u64*)mbox);
        if (m == 0) {
          asm volatile("s_waitcnt vmcnt(2)" ::: "memory");
          land = max(land, k - 2);
          LDS_ST(rland, (unsigned)land);
          m = LDS_LD((LDS_AS u64*)mbox);
          if (m == 0) {
            drain();
            for (;;) {
              m = LDS_LD((LDS_AS u64*)mbox);
              if (m != 0) break;
              if (ring_dead()) { m = 0x80000000u; break; }
              __builtin_amdgcn_s_sleep(1);
            }
          }
        }
        const unsigned mu = __builtin_amdgcn_readfirstlane((unsigned)m);
        const unsigned flo = mu ? mu : __builtin_amdgcn_readfirstlane((unsigned)(m >> 32));
        LDS_ST(mbox + (mu ? 0 : 1), 0u);
        if (flo >> 31) break;
        next_slot();
        const unsigned copy = (flo >> 30) & 1u, id = flo & 0xfffu, hi = (flo >> 12) & 0x1fffu;
        const unsigned nb = copy ? 8192u : seg_bytes(hi);
        const unsigned np = (nb + 1023) >> 10;
        LDS_ST(rdesc + slot, ((u64)(((sq + 1) << 16) | ((bad || ((flo >> 29) & 1u)) ? 0x8000u : 0u) | (copy << 14) | id) << 32) | (hi << 16) | (unsigned)(k + np));
        const unsigned sb = copy ? hi * 8192u : OB_OFF + id * SEG;
        unsigned vo = sb + l16;
        const unsigned vend = sb + nb;
        unsigned la = ldsbase + slot * SEG;
        for (unsigned i = 0; i < np; ++i) {
          unsigned keep;
          asm volatile("s_waitcnt vmcnt(%0)" :: "i"(ODEPTH - 1) : "memory");
          land = max(land, k - (ODEPTH - 1));
          LDS_ST(rland, (unsigned)land);
          asm volatile("s_mov_b32 %0, m0\n\ts_mov_b32 m0, %3\n\ts_nop 0\n\tbuffer_load_dwordx4 %1, %2, 0 offen sc0 sc1 lds\n\ts_mov_b32 m0, %0"
                       : "=&s"(keep) : "v"(vo < vend ? vo : oob), "s"(srs), "s"(la) : "memory");
          vo += 1024; la += 1024; ++k;
        }
        ++sq;
        if (++slot == NSLOT) slot = 0;
      }
      STAMP(10);
      drain();
      for (int e = 0; e < 2; ++e) {
        next_slot();
        LDS_ST(rdesc + slot, (u64)(((sq + 1) << 16) | 0xfffu) << 32);
        ++sq;
        if (++slot == NSLOT) slot = 0;
      }
    }
    if (__hip_atomic_load(tmo, __ATOMIC_RELAXED, __HIP_MEMORY_SCOPE_AGENT) == call && lane == 0)
      *mark = 0x7fc0;
    return;
  }

  const int gb = blockIdx.x - NMOV;
  const int wm = wv >> 1, hf = wv & 1;
  const unsigned wgt = ((const unsigned*)p.tiles.wg)[gb >> 1] >> (16 * (gb & 1));

#pragma unroll 1
  for (int jn = 0; jn < (gb < 256 - NG ? 2 : 1); ++jn) {
  const int pass = (wgt >> (8 * jn + 7)) & 1, kt = (wgt >> (8 * jn)) & 127, st = kt;
  LDS_AS u64* ctl = (LDS_AS u64*)(lds + LDS_CTRL + 32) + jn;
  f32x4 acc[8][8];
  u32x4 pt[8];
  unsigned early;
  int lane = tid & 63;
  asm volatile("" : "+v"(lane));
  const int rt = pass ? 8 + (kt >> 4) : st >> 4, ct = (pass ? kt : st) & 15;
  {
    const unsigned bx = WS_OFF + (unsigned)(16 + rt) * (NKS * KSB) + lane * 16, bw = WS_OFF + (unsigned)ct * (NKS * KSB) + lane * 16;
    const unsigned vx0 = bx + wm * 8192, vx1 = bx + (wm ^ 1) * 8192;
    const unsigned vw0 = bw + hf * 4096, vw1 = bw + (hf ^ 1) * 4096;
#pragma unroll
    for (int a = 0; a < 8; ++a)
#pragma unroll
      for (int b = 0; b < 8; ++b) acc[a][b] = f32x4{0.f, 0.f, 0.f, 0.f};
    u32x4 fw[3][8], fx[3][8];
#define LDW(dst, vo, so, imm) asm volatile("buffer_load_dwordx4 %0, %1, %2, %3 offen offset:%4 sc1" : "=v"(dst) : "v"(vo), "s"(mrs), "s"(so), "i"(imm))
#define LDX(dst, vo, so, imm) asm volatile("buffer_load_dwordx4 %0, %1, %2, %3 offen offset:%4 sc1" : "=v"(dst) : "v"(vo), "s"(mrs), "s"(so), "i"(imm))
#define MF(T, M, w_, x_) asm volatile("v_mfma_f32_16x16x32_bf16 %0, %1, %2, %0" : "+a"(acc[T][M]) : "v"(w_[T]), "v"(x_[M]))
#define WAITF(w_, x_, n) asm volatile("s_waitcnt vmcnt(" #n ")" : "+v"(w_[0]), "+v"(w_[1]), "+v"(w_[2]), "+v"(w_[3]), "+v"(w_[4]), "+v"(w_[5]), "+v"(w_[6]), "+v"(w_[7]), \
                   "+v"(x_[0]), "+v"(x_[1]), "+v"(x_[2]), "+v"(x_[3]), "+v"(x_[4]), "+v"(x_[5]), "+v"(x_[6]), "+v"(x_[7]))
    auto load1 = [&](int i, unsigned sb, int kk, u32x4* w_, u32x4* x_) {
      const int q = i >> 2, j = i & 3;
      if (q == 0) LDW(w_[j], vw0, sb + kk * 8192, j * 1024);
      else if (q == 1) LDX(x_[j], vx0, sb + (j >> 1) * 4096, (j & 1) * 2048 + kk * 1024);
      else if (q == 2) LDW(w_[4 + j], vw1, sb + kk * 8192, j * 1024);
      else LDX(x_[4 + j], vx1, sb + (j >> 1) * 4096, (j & 1) * 2048 + kk * 1024);
    };
    auto half = [&](u32x4* w_, u32x4* x_, bool doload, unsigned sb, int kk, u32x4* wn, u32x4* xn) {
#pragma unroll
      for (int i = 0; i < 16; ++i) {
        const int qa = i >> 3, qb = (i >> 2) & 1, m = i & 3;
#pragma unroll
        for (int t = 0; t < 4; ++t) MF((qa ^ qb) * 4 + t, qa * 4 + m, w_, x_);
        if (doload) load1(i, sb, kk, wn, xn);
      }
    };
    const int lj = lane & 7;
    const unsigned ksv = (unsigned)(lj < 5 ? ((wv - lj) & 3) * 5 + lj : 15 + ((wv - lj) & 3) * 3 + lj);
    const unsigned foff = IF_OFF + (lane < 32 ? ((unsigned)((lane & 8) ? ct : 16 + rt) * 32 + ksv) * 2 + (lane >> 4)
                                              : PS_OFF / 8192 + (unsigned)(kt * 4 + wv)) * 4;
    unsigned todo = 0xffu, fv = 0, warm = 0xffffu;
    auto pick = [&]() -> unsigned {
      u64 rd = __ballot(fv >= call);
      rd &= rd >> 16;
      unsigned m = (unsigned)rd & (unsigned)(rd >> 8) & todo;
      if (m == 0) {
        u64 t0 = __builtin_amdgcn_s_memrealtime();
        for (unsigned spin = 1;; ++spin) {
          const unsigned v = __hip_atomic_load((unsigned*)(myws + foff), __ATOMIC_RELAXED, __HIP_MEMORY_SCOPE_SYSTEM);
          rd = __ballot(v >= call);
          rd &= rd >> 16;
          m = (unsigned)rd & (unsigned)(rd >> 8) & todo;
          if (m) break;
          if ((spin & 15) == 0 && __builtin_amdgcn_readfirstlane(timed_out(tmo, call, t0))) { m = todo; cnt[4] = 1; break; }
          const unsigned cand = (unsigned)rd & warm & (todo * 0x101u);
          if (cand) {
            const unsigned i = __builtin_ctz(cand);
            warm &= ~(1u << i);
            const unsigned so = WS_OFF + ((unsigned)(i < 8 ? 16 + rt : ct) * NKS + (unsigned)__builtin_amdgcn_readlane(ksv, i)) * KSB;
            unsigned a = 0;
#pragma unroll
            for (int q = 0; q < PFN; ++q) a |= __builtin_amdgcn_raw_buffer_load_b32(mrs, (tid & 63) * (256 / PFN), so + q * (KSB / PFN), 16);
            asm volatile("" :: "v"(a));
          } else
          __builtin_amdgcn_s_sleep(PSLEEP);
        }
      }
      const unsigned j = __builtin_ctz(m);
      todo &= ~(1u << j);
      return (unsigned)__builtin_amdgcn_readlane(ksv, j) * KSB;
    };
#define LDF() asm volatile("buffer_load_dword %0, %1, %2, 0 offen sc0 sc1" : "=v"(fv) : "v"(foff), "s"(mrs))
    {
      const unsigned sb = pick();
      asm volatile("s_waitcnt vmcnt(0)\n\ts_nop 4" ::: "memory");
      LDF();
#pragma unroll
      for (int i = 0; i < 16; ++i) load1(i, sb, 0, fw[0], fx[0]);
#pragma unroll
      for (int i = 0; i < 16; ++i) load1(i, sb, 1, fw[1], fx[1]);
    }
#define KSTEP(u) do { \
        WAITF(fw[(u) % 3], fx[(u) % 3], 16); \
        asm volatile("" : "+v"(fv)); \
        const unsigned sb = pick(); \
        asm volatile("s_nop 4"); \
        LDF(); \
        half(fw[(u) % 3], fx[(u) % 3], true, sb, 0, fw[((u) + 2) % 3], fx[((u) + 2) % 3]); \
        WAITF(fw[((u) + 1) % 3], fx[((u) + 1) % 3], 17); \
        half(fw[((u) + 1) % 3], fx[((u) + 1) % 3], true, sb, 1, fw[(u) % 3], fx[(u) % 3]); } while (0)
#pragma unroll 1
    for (int s = 0; s < 2; ++s) { KSTEP(0); KSTEP(2); KSTEP(4); }
    KSTEP(0);
    WAITF(fw[2], fx[2], 16);
    half(fw[2], fx[2], false, 0, 0, fw[0], fx[0]);
    WAITF(fw[0], fx[0], 0);
    half(fw[0], fx[0], false, 0, 0, fw[0], fx[0]);
    asm volatile("" : "+v"(fv));
    early = (unsigned)(__ballot(fv >= call) >> 32) & (unsigned)pass;
    asm volatile("s_nop 15\n\ts_nop 15" ::: "memory");
    __builtin_amdgcn_sched_barrier(0);
  }
  STAMP(2 + 3 * jn);
  asm volatile("" : "+v"(lane));
  f32x4 r[4][4];
  const unsigned po = PS_OFF + (unsigned)(kt * 4 + wv) * 8192 + lane * 16;
  char* wq = p.ws[pass ? lane & 7 : (gq ^ 1) * 4 + (rt & 1) * 2 + (ct & 1)];
  {
    if (early) {
#pragma unroll
      for (int c = 0; c < 8; ++c) pt[c] = __builtin_amdgcn_raw_buffer_load_b128(mrs, po + c * 1024, 0, 16);
    }
    unsigned a0 = (unsigned)(size_t)(LDS_AS char*)lds + wv * 32768 + lane * 16;
    asm volatile("" : "+v"(a0));
    const unsigned a1 = a0 ^ 32768u, a3 = a0 ^ (3u * 32768u);
#pragma unroll
    for (int m = 7; m >= 0; --m)
#pragma unroll
      for (int t = 0; t < 4; ++t) *(LDS_AS f32x4*)(size_t)(a0 + (m * 4 + t) * 1024) = acc[4 + t][m];
    f32x4 hi[16], u[16], v[16];
#pragma unroll
    for (int i = 0; i < 16; ++i) { hi[i] = acc[i & 3][4 + (i >> 2)]; r[i & 3][i >> 2] = acc[i & 3][i >> 2]; asm volatile("" : "+v"(hi[i]), "+v"(r[i & 3][i >> 2])); }
#define O16(x) "=&v"(x[0]), "=&v"(x[1]), "=&v"(x[2]), "=&v"(x[3]), "=&v"(x[4]), "=&v"(x[5]), "=&v"(x[6]), "=&v"(x[7]), \
               "=&v"(x[8]), "=&v"(x[9]), "=&v"(x[10]), "=&v"(x[11]), "=&v"(x[12]), "=&v"(x[13]), "=&v"(x[14]), "=&v"(x[15])
#define READ16(o) "ds_read_b128 %0, %16 offset:" #o "\n\tds_read_b128 %1, %16 offset:" #o "+1024\n\tds_read_b128 %2, %16 offset:" #o "+2048\n\t" \
      "ds_read_b128 %3, %16 offset:" #o "+3072\n\tds_read_b128 %4, %16 offset:" #o "+4096\n\tds_read_b128 %5, %16 offset:" #o "+5120\n\t" \
      "ds_read_b128 %6, %16 offset:" #o "+6144\n\tds_read_b128 %7, %16 offset:" #o "+7168\n\tds_read_b128 %8, %16 offset:" #o "+8192\n\t" \
      "ds_read_b128 %9, %16 offset:" #o "+9216\n\tds_read_b128 %10, %16 offset:" #o "+10240\n\tds_read_b128 %11, %16 offset:" #o "+11264\n\t" \
      "ds_read_b128 %12, %16 offset:" #o "+12288\n\tds_read_b128 %13, %16 offset:" #o "+13312\n\tds_read_b128 %14, %16 offset:" #o "+14336\n\t" \
      "ds_read_b128 %15, %16 offset:" #o "+15360\n\ts_waitcnt lgkmcnt(0)"
    asm volatile("s_waitcnt lgkmcnt(0)\n\ts_barrier\n\t" READ16(16384) : O16(u) : "v"(a1) : "memory");
#pragma unroll
    for (int i = 0; i < 16; ++i) *(LDS_AS f32x4*)(size_t)(a1 + 16384 + i * 1024) = u[i] + hi[i];
    asm volatile(READ16(0) "\n\ts_barrier" : O16(u) : "v"(a1) : "memory");
    asm volatile(READ16(16384) : O16(v) : "v"(a3) : "memory");
#pragma unroll
    for (int i = 0; i < 16; ++i) r[i & 3][i >> 2] = (r[i & 3][i >> 2] + u[i]) + v[i];
    if (cnt[4]) r[0][0][0] = __uint_as_float(0x7fc00000u);
  }
  STAMP(3 + 3 * jn);
  if (pass == 0) {
    unsigned slot = 1;
    asm volatile("buffer_atomic_add %0, %1, %2, 0 offen sc0" : "+v"(slot)
                 : "v"(tid == 0 ? QC_OFF + (unsigned)((call & 1) * 32 + 24 + (rt & 1) * 2 + (ct & 1)) * 128 : 0xF0000000u), "s"(mrs) : "memory");
#pragma unroll
    for (int c = 0; c < 8; ++c) {
      const f32x4 a = r[c >> 1][(c & 1) * 2], b = r[c >> 1][(c & 1) * 2 + 1];
      const u32x4 pk = {pack_bf16(a[0], a[1]), pack_bf16(a[2], a[3]), pack_bf16(b[0], b[1]), pack_bf16(b[2], b[3])};
      __builtin_amdgcn_raw_buffer_store_b128(pk, mrs, PB_OFF + (unsigned)(st * 4 + wv) * 8192 + c * 1024 + lane * 16, 0, AUX_SYS);
    }
    asm volatile("s_waitcnt vmcnt(0)" : "+v"(slot), "+v"(wq) :: "memory");
    __syncthreads();
    slot = __builtin_amdgcn_readfirstlane(slot);
    if (tid < 4 && slot < 32u)
      __hip_atomic_store((u64*)(wq + OF_OFF) + rank * 640 + 512 + slot * 4 + tid, ((u64)call << 32) | ((unsigned)st << 14), __ATOMIC_RELAXED, __HIP_MEMORY_SCOPE_SYSTEM);
  } else {
    if (!early) {
      unsigned* pf = (unsigned*)(myws + IF_OFF) + PS_OFF / 8192 + kt * 4 + wv;
      u64 t0 = __builtin_amdgcn_s_memrealtime();
      for (unsigned spin = 1; __hip_atomic_load(pf, __ATOMIC_RELAXED, __HIP_MEMORY_SCOPE_SYSTEM) < call; ++spin) {
        if ((spin & 15) == 0 && __builtin_amdgcn_readfirstlane(timed_out(tmo, call, t0))) { r[0][0][0] = __uint_as_float(0x7fc00000u); break; }
        __builtin_amdgcn_s_sleep(1);
      }
      asm volatile("" ::: "memory");
#pragma unroll
      for (int c = 0; c < 8; ++c) pt[c] = __builtin_amdgcn_raw_buffer_load_b128(mrs, po + c * 1024, 0, 16);
    }
#pragma unroll
    for (int i = 0; i < 16; ++i) {
      const unsigned a = pt[i >> 1][(i & 1) * 2], b = pt[i >> 1][(i & 1) * 2 + 1];
      r[i >> 2][i & 3] += __builtin_bit_cast(f32x4, u32x4{a << 16, a & 0xffff0000u, b << 16, b & 0xffff0000u});
      asm volatile("" : "+v"(r[i >> 2][i & 3]));
    }
    STAMP(10 + 3 * jn);
    LDS_AS char* seg = (LDS_AS char*)(lds + (wv ^ 1) * 32768);
    u32x4 pk[8];
    unsigned E[2] = {0u, 0u};
#pragma unroll
    for (int c = 0; c < 8; ++c)
#pragma unroll
      for (int dd = 0; dd < 4; ++dd) {
        unsigned t = pack_bf16(r[(dd & 1) * 2][c >> 1][(c & 1) * 2 + (dd >> 1)], r[(dd & 1) * 2 + 1][c >> 1][(c & 1) * 2 + (dd >> 1)]);
        unsigned nn, nz;
        asm("v_pk_sub_u16 %1, %0, %3 clamp\n\t"
            "v_pk_max_i16 %0, %0, 0\n\t"
            "v_pk_mad_i16 %0, %1, %4, %0 clamp\n\t"
            "v_pk_min_u16 %2, %0, 1 op_sel_hi:[1,0]"
            : "+v"(t), "=&v"(nn), "=v"(nz) : "s"(0xff80ff80u), "s"(0x7fff7fffu));
        pk[c][dd] = t;
        E[c >> 2] |= nz << ((c & 3) * 4 + dd);
      }
    const unsigned cn = __builtin_popcount(E[0]) + __builtin_popcount(E[1]);
    const unsigned inc = wave_scan(cn);
    const unsigned total = __builtin_amdgcn_readlane(inc, 63);
    const unsigned H = total >> 5;
    {
      const unsigned xo = ((2 * total + 3) & ~3u) - 4 * H;
      *(LDS_AS u32x2*)(seg + SEG_TBL + 512 + lane * 8) = u32x2{2u * lane < H ? E[0] : 0u, 2u * lane + 1 < H ? E[1] : 0u};
      if (lane == 63) *(LDS_AS unsigned short*)(seg + 2 * total) = 0;
      if (2u * lane >= H) *(LDS_AS unsigned*)(seg + xo + lane * 8) = E[0];
      if (2u * lane + 1 >= H) *(LDS_AS unsigned*)(seg + xo + lane * 8 + 4) = E[1];
    }
    unsigned wa = (unsigned)(size_t)seg + 2 * (inc - cn);
#pragma unroll
    for (int c = 0; c < 8; ++c)
#pragma unroll
      for (int dd = 0; dd < 4; ++dd)
        asm volatile("v_cmpx_ne_u16 vcc, 0, %1\n\t"
                     "ds_write_b16 %0, %1\n\t"
                     "v_add_u32 %0, 2, %0\n\t"
                     "s_mov_b64 exec, -1\n\t"
                     "v_cmpx_lt_u32 vcc, %2, %1\n\t"
                     "ds_write_b16_d16_hi %0, %1\n\t"
                     "v_add_u32 %0, 2, %0\n\t"
                     "s_mov_b64 exec, -1"
                     : "+v"(wa) : "v"(pk[c][dd]), "s"(0xffffu) : "vcc", "memory");
    STAMP(11 + 3 * jn);
    const unsigned nb = seg_bytes(total);
    const unsigned so = OB_OFF + (unsigned)(kt * 4 + wv) * SEG + lane * 16;
    unsigned slot = 1;
    asm volatile("buffer_atomic_add %0, %1, %2, 0 offen sc0" : "+v"(slot)
                 : "v"(tid == 0 ? QC_OFF + (unsigned)((call & 1) * 32 + 28) * 128 : 0xF0000000u), "s"(mrs) : "memory");
    {
      u32x4 v[9];
      u32x2 e[4];
      asm volatile("ds_read2_b32 %9, %14 offset0:0 offset1:16\n\tds_read_b128 %0, %13\n\tds_read_b128 %1, %13 offset:1024\n\t"
                   "ds_read2_b32 %10, %14 offset0:32 offset1:48\n\tds_read_b128 %2, %13 offset:2048\n\tds_read_b128 %3, %13 offset:3072\n\t"
                   "ds_read2_b32 %11, %14 offset0:64 offset1:80\n\tds_read_b128 %4, %13 offset:4096\n\tds_read_b128 %5, %13 offset:5120\n\t"
                   "ds_read2_b32 %12, %14 offset0:96 offset1:112\n\tds_read_b128 %6, %13 offset:6144\n\tds_read_b128 %7, %13 offset:7168\n\t"
                   "ds_read_b128 %8, %13 offset:8192\n\ts_waitcnt lgkmcnt(0)"
                   : "=&v"(v[0]), "=&v"(v[1]), "=&v"(v[2]), "=&v"(v[3]), "=&v"(v[4]), "=&v"(v[5]), "=&v"(v[6]), "=&v"(v[7]), "=&v"(v[8]),
                     "=&v"(e[0]), "=&v"(e[1]), "=&v"(e[2]), "=&v"(e[3])
                   : "v"((unsigned)(size_t)seg + lane * 16), "v"((unsigned)(size_t)seg + SEG_TBL + 512 + (lane >> 2) * 4) : "memory");
#pragma unroll
      for (int i = 0; i < 9; ++i) {
        if (i < 8) {
          const unsigned x = e[i >> 1][i & 1] >> ((lane & 3) * 4);
#pragma unroll
          for (int d = 0; d < 4; ++d) v[i][d] |= (x << (15 - d)) & 0x80008000u;
        }
        __builtin_amdgcn_raw_buffer_store_b128(v[i], mrs, (unsigned)(i * 1024 + lane * 16) < nb ? so + i * 1024 : 0xF0000000u, 0, AUX_SYS);
      }
    }
    asm volatile("s_waitcnt vmcnt(0)" : "+v"(slot), "+v"(wq) :: "memory");
    STAMP(12 + 3 * jn);
    if (tid == 0) cnt[16] = slot;
    asm volatile("" ::: "memory");
    u64 old = 0;
    const u64 mine = (1ull << 56) | ((u64)total << (14 * wv));
    if (lane == 0) old = __hip_atomic_fetch_add(ctl, mine, __ATOMIC_RELAXED, __HIP_MEMORY_SCOPE_WORKGROUP);
    slot = *(volatile LDS_AS unsigned*)(cnt + 16);
    old = (((u64)(unsigned)__builtin_amdgcn_readfirstlane((unsigned)(old >> 32)) << 32) | (unsigned)__builtin_amdgcn_readfirstlane((unsigned)old)) + mine;
    if ((old >> 56) == 4 && lane < 32 && (lane & 7) != rank && slot < 128u) {
      const u64 f = ((u64)call << 32) | ((unsigned)kt << 14) | (unsigned)((old >> (14 * (lane >> 3))) & 0x3fffu);
      __hip_atomic_store((u64*)(wq + OF_OFF) + rank * 640 + slot * 4 + (lane >> 3), f, __ATOMIC_RELAXED, __HIP_MEMORY_SCOPE_SYSTEM);
    }
    const unsigned oo = (unsigned)((row_tile(gq, ri, 8 + (kt >> 4)) * 128 + wm * 64 + (lane & 15)) * DIM + col_tile(gq, rj, kt & 15) * 128 + 64 * hf + 8 * (lane >> 4)) * 2;
#pragma unroll
    for (int c = 0; c < 8; ++c) __builtin_amdgcn_raw_buffer_store_b128(pk[c], ors, oo + (c & 1) * 64, (c >> 1) * (16 * DIM * 2), OUT_AUX);
    __builtin_amdgcn_s_barrier();
  }
  STAMP(4 + 3 * jn);
  }
}

}

extern "C" void kernel_launch(int rank, int nranks, ncclComm_t comm,
                              hipStream_t stream,
                              void* const* d_in, const int* in_sizes, int n_in,
                              void* d_out, int out_size,
                              const int* in_dtypes, int out_dtype,
                              void* const* d_out_all,
                              void* const* d_ws_all, size_t ws_bytes) {
  (void)comm; (void)in_sizes; (void)n_in; (void)out_size; (void)in_dtypes; (void)out_dtype; (void)d_out_all;
  if (nranks != NR || (unsigned)rank >= (unsigned)NR || ws_bytes < (size_t)WS_USED) {
    fprintf(stderr, "unsupported configuration\n");
    return;
  }
  Params p;
  p.x = (const unsigned short*)d_in[0];
  p.w = (const unsigned short*)d_in[1];
  p.out = (unsigned short*)d_out;
  for (int i = 0; i < NR; ++i) p.ws[i] = (char*)d_ws_all[i];
  p.myws = p.ws[rank];
  p.rank = rank;
  constexpr OrderTable kOrder = make_order();
  constexpr PrepTable kPrep = make_prep();
  p.order = kOrder;
  p.prep = kPrep;
  constexpr TileTable kTiles = make_tiles();
  p.tiles = kTiles;
  gemm_ar_kernel<<<256, 256, 0, stream>>>(p);
  HIP_CHECK(hipGetLastError());
}
```

```cpp
#include <hip/hip_runtime.h>
#include <rccl/rccl.h>
#include <cstdio>
#include <cstdint>

#define HIP_CHECK(expr) do { hipError_t _e = (expr); if (_e != hipSuccess) { \
    fprintf(stderr, "HIP error %s at %s:%d\n", hipGetErrorString(_e), __FILE__, __LINE__); \
    return; } } while (0)

namespace {

typedef float f32x4 __attribute__((ext_vector_type(4)));
typedef unsigned u32x4 __attribute__((ext_vector_type(4)));
typedef unsigned u32x2 __attribute__((ext_vector_type(2)));
typedef unsigned long long u64;

constexpr int NR = 8;
constexpr int DIM = 4096;
constexpr int KSL = DIM / NR;
constexpr int NKS = 32;
constexpr unsigned KSB = 16384;
constexpr unsigned SEG = 9216;
constexpr unsigned SEG_TBL = 8704;
__host__ __device__ constexpr unsigned seg_bytes(unsigned n) { return ((2 * n + 3) & ~3u) + 4 * (128 - (n >> 5)); }

constexpr unsigned WS_OFF = 0;
constexpr unsigned PS_OFF = WS_OFF + 32 * NKS * KSB;
constexpr unsigned SB_OFF = PS_OFF + 128 * 2 * KSB;
constexpr unsigned PB_OFF = SB_OFF + 8 * 2 * 32 * KSB;
constexpr unsigned OB_OFF = PB_OFF + 128 * 2 * KSB;
#define FS 128
#define PFS 128
#define OFS 128
constexpr unsigned IF_OFF = OB_OFF + 128 * 4 * SEG;
constexpr unsigned PF_OFF = IF_OFF + 1280 * FS;
constexpr unsigned OF_OFF = PF_OFF + NR * 256 * PFS;
constexpr unsigned TK_OFF = OF_OFF + NR * 160 * OFS;
constexpr unsigned QC_OFF = TK_OFF + 32 * 128;
constexpr unsigned TMO_OFF = QC_OFF + 64 * 128;
constexpr unsigned DBG_OFF = TMO_OFF + 4096;
__host__ __device__ constexpr unsigned iflag(unsigned i) { return IF_OFF + (i >> 1) * FS + (i & 1) * 4; }
__host__ __device__ constexpr unsigned oflag(unsigned src, unsigned e) { return OF_OFF + (src * 160 + (e >> 2)) * OFS + (e & 3) * 8; }
constexpr unsigned LOG_OFF = DBG_OFF + 256 * 4 * 16 * 8;
constexpr unsigned WS_USED = LOG_OFF + 112 * 128 * 8;

#define NINV 16
#define ODEPTHV 4
constexpr int NIN = NINV;
constexpr int ODEPTH = ODEPTHV;
constexpr int NMOV = 7 * NIN, NG = 256 - NMOV;
constexpr int NSLOT = 12;
#define NEARLY 64u
#define PDELAY 40
#define PSTEP 12
#define PSLEEP 4
#define PFN 4
#define MSPIN 8
#define CPAUX 16
#define SKIPF 1
#define PSC 1
#define PD2 60
#define MU 1

constexpr u64 DEADLINE_TICKS = 50000000ull;

constexpr int LDS_CTRL = 131072;
constexpr int LDS_BYTES = LDS_CTRL + 256;

__host__ __device__ constexpr int row_tile(int g, int h, int r) { return (h << (4 - g)) | ((r & 1) << (3 + g)) | (((r >> 3) ^ g) << 2) | ((r >> 1) & 3); }
__host__ __device__ constexpr int col_tile(int g, int h, int c) { return (h << (4 - g)) | ((c & 1) << (3 + g)) | (c >> 1); }

#define SEQ 16,0,17,1,2,3,18,19,4,5,24,25,6,7,20,21,8,9,26,27,10,11,22,23,12,13,28,29,14,15,30,31
struct PosTable { int p[32]; };
constexpr PosTable make_pos() {
  const int seq[32] = {SEQ};
  PosTable o{};
  for (int i = 0; i < 32; ++i) o.p[seq[i]] = i;
  return o;
}
constexpr PosTable kPos = make_pos();
constexpr int unit_pos(int u) { return kPos.p[u]; }
constexpr int cmax(int a, int b) { return a > b ? a : b; }
#define PLAG 6
constexpr int send_key(int r, int c) { return cmax(unit_pos(16 + r), unit_pos(c)); }
constexpr int kept_key(int kt) {
  return cmax(cmax(unit_pos(24 + (kt >> 4)), unit_pos(kt & 15)),
              cmax(send_key((kt >> 4) | 1, (kt & 15) | 1), send_key((kt >> 4) & 6, kt & 14)) + PLAG);
}
struct alignas(4) TileTable { unsigned char wg[2 * NG]; };
constexpr TileTable make_tiles() {
  TileTable o{};
  int nj[8] = {};
  for (int key = 0; key < 64; ++key)
    for (int t = 0; t < 256; ++t) {
      const int x = t & 7;
      if ((t < 128 ? send_key(t >> 4, t & 15) : kept_key(t & 127)) == key) {
        o.wg[2 * ((nj[x] % (NG / 8)) * 8 + x) + nj[x] / (NG / 8)] = (unsigned char)t;
        ++nj[x];
      }
    }
  return o;
}
#define DLAG 4
#define DMAX 13
struct DirectTable { unsigned mask[4]; unsigned queued[4]; };
constexpr DirectTable make_direct() {
  DirectTable o{};
  for (int t = 0; t < 128; ++t) {
    const int own = cmax(unit_pos(24 + (t >> 4)), unit_pos(t & 15)), part = send_key((t >> 4) | 1, (t & 15) | 1);
    if (own <= DMAX && own <= part + DLAG) o.mask[t >> 5] |= 1u << (t & 31);
    else o.queued[((t >> 4) & 1) * 2 + (t & 1)] += 4;
  }
  return o;
}
struct alignas(4) OrderTable { unsigned char t[2][160]; };
constexpr OrderTable make_order() {
  OrderTable o{};
  for (int x = 0; x < 2; ++x) {
    int n = 0;
    const int nk = x ? 3 : 5;
    for (int key = 0; key <= 31 * PSC + MU * 4; ++key)
      for (int k = nk - 1; k >= 0; --k)
        for (int u = 0; u < 32; ++u)
          if (PSC * unit_pos(u) + MU * k == key) o.t[x][n++] = (unsigned char)(u * 8 + k);
  }
  return o;
}
#define PK 2
constexpr int npush(int nk) {
  int n = 0;
  for (int u = 0; u < 32; ++u)
    for (int k = 0; k < nk; ++k) n += PSC * unit_pos(u) + MU * k <= PK;
  return n;
}
struct alignas(4) PrepTable { unsigned char t[256]; unsigned push[16]; };
constexpr PrepTable make_prep() {
  PrepTable o{};
  int n = 0;
  int keys[256] = {};
  for (int j = 0; j < 256; ++j) {
    const int ksl = j >> 5, t = j & 15, a = PSC * unit_pos(t), b = PSC * unit_pos(16 + t), k = MU * (ksl < 5 ? ksl : ksl - 5);
    keys[j] = ((a < b ? a : b) + k) * (ksl < 5 ? 5 : 3);
  }
  for (int key = 0; key <= 5 * (31 * PSC + MU * 4); ++key)
    for (int j = 0; j < 256; ++j) {
      if (keys[j] == key) {
        const int ksl = j >> 5, t = j & 15, a = PSC * unit_pos(t), b = PSC * unit_pos(16 + t), k = MU * (ksl < 5 ? ksl : ksl - 5);
        o.push[n >> 4] |= (unsigned)((a + k <= PK) + 2 * (b + k <= PK)) << (2 * (n & 15));
        o.t[n++] = (unsigned char)j;
      }
    }
  return o;
}

struct Params {
  const unsigned short* x;
  const unsigned short* w;
  unsigned short* out;
  char* myws;
  char* ws[NR];
  int rank;
  OrderTable order;
  PrepTable prep;
  TileTable tiles;
  DirectTable direct;
};

#define LDS_AS __attribute__((address_space(3)))

__device__ __forceinline__ __amdgpu_buffer_rsrc_t make_rsrc(const void* p, unsigned bytes) {
  return __builtin_amdgcn_make_buffer_rsrc((void*)p, 0, bytes, 0x00020000);
}
constexpr int AUX_SYS = 17;
constexpr int OUT_AUX = 16;

__device__ __forceinline__ bool timed_out(unsigned* tmo, unsigned call, u64 t0) {
  if (__hip_atomic_load(tmo, __ATOMIC_RELAXED, __HIP_MEMORY_SCOPE_AGENT) == call ||
      __builtin_amdgcn_s_memrealtime() - t0 > DEADLINE_TICKS) {
    __hip_atomic_store(tmo, call, __ATOMIC_RELAXED, __HIP_MEMORY_SCOPE_AGENT);
    return true;
  }
  return false;
}

__device__ __forceinline__ unsigned pack_bf16(float a, float b) {
  typedef __bf16 bf2 __attribute__((ext_vector_type(2)));
  typedef float f2 __attribute__((ext_vector_type(2)));
  f2 v = {a, b};
  bf2 r = __builtin_convertvector(v, bf2);
  return __builtin_bit_cast(unsigned, r);
}

__device__ __forceinline__ unsigned wave_scan(unsigned x) {
  x += __builtin_amdgcn_update_dpp(0u, x, 0x111, 0xf, 0xf, false);
  x += __builtin_amdgcn_update_dpp(0u, x, 0x112, 0xf, 0xf, false);
  x += __builtin_amdgcn_update_dpp(0u, x, 0x114, 0xf, 0xf, false);
  x += __builtin_amdgcn_update_dpp(0u, x, 0x118, 0xf, 0xf, false);
  x += __builtin_amdgcn_update_dpp(0u, x, 0x142, 0xa, 0xf, false);
  x += __builtin_amdgcn_update_dpp(0u, x, 0x143, 0xc, 0xf, false);
  return x;
}
typedef unsigned short u16x2 __attribute__((ext_vector_type(2)));

#define STAMP(i) do { } while (0)

__global__ void __launch_bounds__(256, 1) gemm_ar_kernel(Params p) {
  __shared__ __attribute__((aligned(16))) char lds[LDS_BYTES];
  const int tid = threadIdx.x;
  const int lane = tid & 63;
  const int wv = __builtin_amdgcn_readfirstlane(tid >> 6);
  const int rank = p.rank;
  char* myws = p.myws;
  unsigned* tmo = (unsigned*)(myws + TMO_OFF);
  const int r16 = lane & 15, g4 = lane >> 4;

  const int gq = rank >> 2, ri = (rank >> 1) & 1, rj = rank & 1;
  STAMP(0);
  __amdgpu_buffer_rsrc_t mrs = make_rsrc(myws, WS_USED);
  __amdgpu_buffer_rsrc_t ors = make_rsrc(p.out, (unsigned)DIM * DIM * 2);
  LDS_AS unsigned* cnt = (LDS_AS unsigned*)(lds + LDS_CTRL);
  unsigned call;
  {
    const unsigned job = blockIdx.x >= 224u ? blockIdx.x - 224u : (blockIdx.x - 80u) & 255u;
    unsigned jobw = ((const unsigned*)p.prep.t)[job >> 2];
    const unsigned jobw2 = ((const unsigned*)p.prep.t)[blockIdx.x >> 2];
    unsigned push = p.prep.push[job >> 4];
    const unsigned short* const px = p.x;
    const unsigned short* const pw = p.w;
    asm volatile("" :: "s"(px), "s"(pw), "s"(jobw), "s"(jobw2), "s"(push), "s"(rank), "s"(myws));
    unsigned ticket = 1;
    asm volatile("buffer_atomic_add %0, %1, %2, 0 offen sc0" : "+v"(ticket) : "v"(lane == 0 ? TK_OFF + (blockIdx.x & 31) * 128 : 0xF0000000u), "s"(mrs) : "memory");
    char* wsl = p.ws[0];
#pragma unroll
    for (int i = 1; i < NR; ++i) wsl = (lane & 7) == i ? p.ws[i] : wsl;
    const int cbr = 8 * ((r16 & 3) >> 1) + 2 * (r16 >> 2) + (r16 & 1);
    auto load_w = [&](u32x4* r, int t, int k, int qt) {
      const unsigned short* q = pw + (size_t)(k * 64 + (qt >> 1) * 32 + 8 * g4) * DIM + t * 128 + 64 * (qt & 1) + 4 * cbr;
#pragma unroll
      for (int h = 0; h < 4; ++h) {
        const u32x2 a = *(const u32x2*)(q + (size_t)(2 * h) * DIM), b = *(const u32x2*)(q + (size_t)(2 * h + 1) * DIM);
        r[h] = u32x4{a.x, a.y, b.x, b.y};
      }
    };
    auto load_x = [&](u32x4* r, int t, int k, int qt) {
      const unsigned short* q = px + (size_t)(t * 128 + qt * 32 + r16) * KSL + k * 64 + 8 * g4;
#pragma unroll
      for (int h = 0; h < 4; ++h) r[h] = *(const u32x4*)(q + (size_t)(h >> 1) * 16 * KSL + (h & 1) * 32);
    };
    auto transpose_w = [&](u32x4* r) {
      u32x4 v[4];
#pragma unroll
      for (int t = 0; t < 4; ++t)
#pragma unroll
        for (int h = 0; h < 4; ++h) {
          const unsigned lo = r[h][t >> 1], hi = r[h][2 + (t >> 1)];
          v[t][h] = __builtin_amdgcn_perm(hi, lo, (t & 1) ? 0x07060302u : 0x05040100u);
        }
#pragma unroll
      for (int t = 0; t < 4; ++t) r[t] = v[t];
    };
    if (tid < 64) cnt[tid] = 0;
    asm volatile("s_waitcnt lgkmcnt(0)" ::: "memory");
    __builtin_amdgcn_s_barrier();
    const int njob = blockIdx.x < NMOV ? 0 : blockIdx.x >= NG ? 2 : 1;
    push = (push >> (2 * (job & 15))) & 3u;
#pragma unroll 1
    for (int jn = 0; jn < njob; ++jn) {
      const int jb = (jobw >> (8 * (job & 3))) & 255, ksl = jb >> 5, hh = (jb >> 4) & 1, tt = jb & 15;
      const int tg = ksl < 5 ? gq : gq ^ 1, kq = ksl < 5 ? ksl : ksl - 5;
      const int gtw = col_tile(tg, hh, tt), gtx = row_tile(tg, hh, tt);
      if (jn == 0 && job >= NEARLY) {
        __builtin_amdgcn_s_sleep(PDELAY);
        for (unsigned i = NEARLY; i < job; i += 8) __builtin_amdgcn_s_sleep(PSTEP);
      }
      if (jn) __builtin_amdgcn_s_sleep(PD2);
      u32x4 wr[4], xr[4];
      load_w(wr, gtw, ksl, wv);
      load_x(xr, gtx, ksl, wv);
      const unsigned sb = SB_OFF + (unsigned)((ksl * 2 + hh) * 32 + tt) * KSB + wv * 4096 + lane * 16;
      const unsigned lb = WS_OFF + (unsigned)(tt * NKS + (rank & 3) * 5 + kq) * KSB + wv * 4096 + lane * 16;
      const unsigned lw = (ksl < 5 && hh == rj) ? lb : 0xF0000000u, lx = (ksl < 5 && hh == ri) ? lb + 16 * NKS * KSB : 0xF0000000u;
      transpose_w(wr);
#pragma unroll
      for (int t = 0; t < 4; ++t) __builtin_amdgcn_raw_buffer_store_b128(wr[t], mrs, sb + t * 1024, 0, AUX_SYS);
#pragma unroll
      for (int h = 0; h < 4; ++h) __builtin_amdgcn_raw_buffer_store_b128(xr[h], mrs, sb + 16 * KSB + h * 1024, 0, AUX_SYS);
#pragma unroll
      for (int t = 0; t < 4; ++t) __builtin_amdgcn_raw_buffer_store_b128(wr[t], mrs, lw + t * 1024, 0, AUX_SYS);
#pragma unroll
      for (int h = 0; h < 4; ++h) __builtin_amdgcn_raw_buffer_store_b128(xr[h], mrs, lx + h * 1024, 0, AUX_SYS);
      const unsigned ks = ksl < 5 ? (rank & 3) * 5 + kq : 20 + (rank & 3) * 3 + kq;
      if (push) {
        asm volatile("" ::: "memory");
#pragma unroll
        for (int i = 0; i < 4; ++i) {
          const int q = tg * 4 + (i < 2 ? i * 2 + hh : hh * 2 + i - 2);
          const __amdgpu_buffer_rsrc_t prs = make_rsrc(p.ws[q], WS_USED);
          const unsigned po = ((push >> (i >> 1)) & 1) && q != rank ? WS_OFF + (unsigned)((tt + (i >> 1) * 16) * NKS + ks) * KSB + wv * 4096 + lane * 16 : 0xF0000000u;
#pragma unroll
          for (int h = 0; h < 4; ++h) __builtin_amdgcn_raw_buffer_store_b128(i < 2 ? wr[h] : xr[h], prs, po + h * 1024, 0, AUX_SYS);
        }
        asm volatile("s_waitcnt vmcnt(16)" ::: "memory");
      } else {
        asm volatile("s_waitcnt vmcnt(0)" ::: "memory");
      }
      asm volatile("" : "+v"(ticket) :: "memory");
      call = (unsigned)__builtin_amdgcn_readfirstlane(ticket) / 32u + 1u;
      const int q = lane & 7;
      const bool need = lane < 16 && (q >> 2) == tg && (lane < 8 ? (q & 1) : ((q >> 1) & 1)) == hh;
      const int unit = tt + (lane & 8) * 2;
      auto last = [&](int ci) -> bool {
        unsigned old = 0;
        if (lane == 0) old = __hip_atomic_fetch_add(cnt + ci, 1u, __ATOMIC_RELAXED, __HIP_MEMORY_SCOPE_WORKGROUP);
        return __builtin_amdgcn_readfirstlane(old) == 3;
      };
      if (last(jn) && need) {
        if (q == rank)
          __hip_atomic_store((u64*)(myws + iflag((unit * 32 + ks) * 2)), ((u64)call << 32) | call, __ATOMIC_RELAXED, __HIP_MEMORY_SCOPE_SYSTEM);
        else
          __hip_atomic_store((unsigned*)(wsl + PF_OFF + (rank * 256 + unit * 8 + kq) * PFS + 4), call, __ATOMIC_RELAXED, __HIP_MEMORY_SCOPE_SYSTEM);
      }
      if (push) {
        asm volatile("s_waitcnt vmcnt(0)" ::: "memory");
        if (last(2) && need && q != rank && ((push >> (lane >> 3)) & 1))
          __hip_atomic_store((u64*)(wsl + iflag((unit * 32 + ks) * 2)), ((u64)call << 32) | call, __ATOMIC_RELAXED, __HIP_MEMORY_SCOPE_SYSTEM);
      }
      push = 0;
      jobw = jobw2;
    }
    asm volatile("s_waitcnt vmcnt(0)" : "+v"(ticket) :: "memory");
    call = (unsigned)__builtin_amdgcn_readfirstlane(ticket) / 32u + 1u;
    STAMP(1);
  }
  if (blockIdx.x < NMOV) {
    const int mb = blockIdx.x;
    const int bi = __builtin_amdgcn_readfirstlane((mb * 9363) >> 16);
    const int lk = __builtin_amdgcn_readfirstlane(mb - 7 * bi);
    const int src = (rank + 1 + lk) & 7;
    const int wi = bi;
    const int sgp = src >> 2, si = (src >> 1) & 1, sj = src & 1;
    unsigned short* const mark = p.out + (size_t)(row_tile(sgp, si, 8) * 128) * DIM + col_tile(sgp, sj, 0) * 128 + wi;
    const unsigned l16 = lane * 16;
    const unsigned oob = 0xF0000000u;
    LDS_AS u64* rdesc = (LDS_AS u64*)(lds + LDS_CTRL + 64);
    LDS_AS unsigned* rfree = (LDS_AS unsigned*)(lds + LDS_CTRL + 160);
    LDS_AS unsigned* rland = (LDS_AS unsigned*)(lds + LDS_CTRL + 208);
#define LDS_LD(ptr) __hip_atomic_load(ptr, __ATOMIC_RELAXED, __HIP_MEMORY_SCOPE_WORKGROUP)
#define LDS_ST(ptr, val) __hip_atomic_store(ptr, val, __ATOMIC_RELAXED, __HIP_MEMORY_SCOPE_WORKGROUP)
    const u64 tstart = __builtin_amdgcn_s_memrealtime();
    auto ring_dead = [&]() -> bool {
      if (__builtin_amdgcn_s_memrealtime() - tstart <= 3 * DEADLINE_TICKS) return false;
      __hip_atomic_store(tmo, call, __ATOMIC_RELAXED, __HIP_MEMORY_SCOPE_AGENT);
      if (lane == 0) *mark = 0x7fc0;
      return true;
    };
    const int cross = sgp != gq;
    LDS_AS unsigned* mbox = (LDS_AS unsigned*)(lds + LDS_CTRL + 216);
    if (wv == 3) {
      const unsigned ps = cross ? 2 * npush(3) : 2 * npush(5);
      const unsigned np = cross ? p.direct.queued[src & 3] : 0u, no = (cross ? 192u : 320u) - ps;
      const unsigned otab = ((const unsigned*)p.order.t[cross])[lane < 40 ? lane : 0];
      const unsigned ctr = QC_OFF + (unsigned)((call & 1) * 32 + lk + (lane & 3) * 8) * 128;
      unsigned tn = 1;
      asm volatile("buffer_atomic_add %0, %1, %2, 0 offen sc0\n\ts_waitcnt vmcnt(0)" : "+v"(tn) : "v"(lane < 3 ? ctr : oob), "s"(mrs) : "memory");
      unsigned cp = __builtin_amdgcn_readlane(tn, 0), co = __builtin_amdgcn_readlane(tn, 1), cq = __builtin_amdgcn_readlane(tn, 2);
      unsigned nq = 0;
      bool bad = false, refill = true;
      while (cq < 512u || co < no || cp < np) {
        const unsigned ob_ = (__builtin_amdgcn_readlane(otab, ((co + ps) >> 3) & 63) >> (8 * (((co + ps) >> 1) & 3))) & 255u;
        const unsigned fa = lane == 0 ? (cp < np ? oflag(src, 512 + cp) : oob)
                          : lane == 1 ? (co < no ? PF_OFF + (unsigned)(src * 256 + ob_) * PFS : oob)
                          : lane == 2 ? (cq < 512u ? oflag(src, cq) : oob) : oob;
        tn = 1;
        asm volatile("buffer_atomic_add %0, %1, %2, 0 offen sc0" : "+v"(tn) : "v"(lane == 2 && refill ? ctr : oob), "s"(mrs) : "memory");
        u64 t0 = __builtin_amdgcn_s_memrealtime();
        unsigned rd, fx;
        for (unsigned spin = 1;; ++spin) {
          const u32x2 f = __builtin_amdgcn_raw_buffer_load_b64(mrs, fa, 0, AUX_SYS);
          const unsigned ex = bad ? 4u : (unsigned)__ballot(f.y >= call) & 7u;
          fx = f.x;
          rd = 0;
          if (ex) {
            for (int w = cp < np ? MSPIN : 64; w > 0 && !rd; --w) {
              const u64 mb_ = LDS_LD((LDS_AS u64*)mbox);
              rd = __builtin_amdgcn_readfirstlane(ex & (((unsigned)mb_ ? 0u : 1u) | ((mb_ >> 32) ? 0u : 6u)));
            }
            if (rd) break;
          }
          if ((spin & 7) == 0 && !ex && __builtin_amdgcn_readfirstlane(timed_out(tmo, call, t0))) bad = true;
          if ((spin & 7) == 0 && ring_dead()) return;
          asm volatile("" ::: "memory");
        }
        asm volatile("s_waitcnt vmcnt(0)" : "+v"(tn) :: "memory");
        if (refill) nq = __builtin_amdgcn_readlane(tn, 2);
        refill = false;
        if (bad) {
          co = no; cp = np;
          fx = (cq >> 2) << 14;
          if (cq >= 512u) break;
        }
        if (rd & 1) {
          const unsigned tile = (__builtin_amdgcn_readlane(fx, 0) >> 14) & 127u, sw = cp & 3u;
          LDS_ST(mbox, (5u << 28) | ((PB_OFF / 8192 + tile * 4 + sw) << 12) | (PS_OFF / 8192 + ((tile & 0x6eu) + si * 16 + sj) * 4 + sw));
        } else if (rd & 2) {
          const unsigned u = ob_ >> 3, kb = ob_ & 7u;
          LDS_ST(mbox + 1, (5u << 28) | ((SB_OFF / 8192 + (((kb + 5 * cross) * 2 + (u < 16 ? rj : ri)) * 32 + u) * 2 + (co & 1u)) << 12)
                           | ((u * 32 + (cross ? 20 + (src & 3) * 3 : (src & 3) * 5) + kb) * 2 + (co & 1u)));
        } else {
          const unsigned flo = __builtin_amdgcn_readlane(fx, 2);
          LDS_ST(mbox + 1, (1u << 28) | (min(flo & 0x3fffu, 4096u) << 12) | (((flo >> 14) & 127u) * 4 + (cq & 3u)) | (bad ? 1u << 29 : 0u));
          cq = nq; refill = true;
        }
        if (rd & 3) {
          tn = 1;
          asm volatile("buffer_atomic_add %0, %1, %2, 0 offen sc0\n\ts_waitcnt vmcnt(0)" : "+v"(tn) : "v"(lane == (int)(~rd & 1) ? ctr : oob), "s"(mrs) : "memory");
          if (rd & 1) cp = __builtin_amdgcn_readlane(tn, 0);
          else co = __builtin_amdgcn_readlane(tn, 1);
        }
      }
      while (LDS_LD((LDS_AS u64*)mbox) != 0) {
        if (ring_dead()) return;
        __builtin_amdgcn_s_sleep(1);
      }
      LDS_ST(mbox + 1, 0x80000000u);
      if (bad && lane == 0) *mark = 0x7fc0;
      return;
    }
    if (wv != 0) {
      const unsigned ob = (unsigned)(r16 * DIM + 8 * g4) * 2;
      int slot = wv - 1;
      for (unsigned sq = wv - 1;; sq += 2) {
        u64 d;
        for (;;) {
          d = LDS_LD(rdesc + slot);
          if ((unsigned)(d >> 48) == sq + 1) break;
          if (ring_dead()) return;
          __builtin_amdgcn_s_sleep(4);
        }
        const unsigned dhi = __builtin_amdgcn_readfirstlane((unsigned)(d >> 32)), dend = __builtin_amdgcn_readfirstlane((unsigned)d);
        const unsigned sg = dhi & 0xfffu;
        if (sg == 0xfffu) break;
        const unsigned nv = dend >> 16, H = nv >> 5;
        while (LDS_LD(rland) < (dend & 0xffffu)) {
          if (ring_dead()) return;
          __builtin_amdgcn_s_sleep(1);
        }
        asm volatile("" ::: "memory");
        LDS_AS char* seg = (LDS_AS char*)(lds + 16 + slot * SEG);
        if (dhi & 0x4000u) {
          u32x4 v[8];
#pragma unroll
          for (int i = 0; i < 8; ++i) v[i] = *(LDS_AS u32x4*)(seg + i * 1024 + l16);
          LDS_ST(rfree + slot, sq + 1);
#pragma unroll
          for (int i = 0; i < 8; ++i) __builtin_amdgcn_raw_buffer_store_b128(v[i], mrs, sg * 8192 + i * 1024 + l16, 0, CPAUX);
          asm volatile("s_waitcnt vmcnt(0)" ::: "memory");
          if (lane == 0) __hip_atomic_store((unsigned*)(myws + iflag(sg)), call, __ATOMIC_RELAXED, __HIP_MEMORY_SCOPE_AGENT);
          slot += 2;
          if (slot >= NSLOT) slot -= NSLOT;
          continue;
        }
        auto unhide = [&](int i0, int n) {
          u32x4 v[5];
#pragma unroll
          for (int i = 0; i < n; ++i) v[i] = *(LDS_AS u32x4*)(seg + (i0 + i) * 1024 + l16);
#pragma unroll
          for (int i = 0; i < n; ++i) {
            unsigned e = ((v[i][0] >> 15) & 0x00010001u) | ((v[i][1] >> 14) & 0x00020002u) | ((v[i][2] >> 13) & 0x00040004u) | ((v[i][3] >> 12) & 0x00080008u);
            e <<= (lane & 3) * 4;
            e |= __builtin_amdgcn_update_dpp(0u, e, 0xb1, 0xf, 0xf, true);
            e |= __builtin_amdgcn_update_dpp(0u, e, 0x4e, 0xf, 0xf, true);
            *(LDS_AS unsigned*)(seg + SEG_TBL + (i0 + i) * 64 + (lane >> 2) * 4) = e;
          }
        };
        unhide(0, 5);
        if (H > 80) unhide(5, 3);
        const unsigned xo = ((2 * nv + 3) & ~3u) - 4 * H;
        u32x2 mk;
        mk.x = *(LDS_AS unsigned*)(seg + (2u * lane < H ? SEG_TBL : xo) + lane * 8);
        mk.y = *(LDS_AS unsigned*)(seg + (2u * lane + 1 < H ? SEG_TBL : xo) + lane * 8 + 4);
        const unsigned c0 = __builtin_popcount(mk.x), cn = c0 + __builtin_popcount(mk.y);
        const int b0 = 2 * (int)(wave_scan(cn) - cn) - 2, b1 = b0 + 2 * (int)c0;
        u32x4 ov[8], oh[8];
#pragma unroll
        for (int c = 0; c < 8; ++c)
#pragma unroll
          for (int dd = 0; dd < 4; ++dd) {
            const int dq = (c & 3) * 4 + dd;
            const unsigned E = c < 4 ? mk.x : mk.y;
            const unsigned pp = __builtin_popcount(E & (((2u << dq) - 1u) | (((1u << dq) - 1u) << 16)));
            const volatile LDS_AS unsigned short* q = (const volatile LDS_AS unsigned short*)(seg + (c < 4 ? b0 : b1) + 2 * (int)pp);
            ov[c][dd] = q[0];
            oh[c][dd] = q[1];
          }
        LDS_ST(rfree + slot, sq + 1);
        const int tile = sg >> 2, sw = sg & 3;
        const unsigned oo = ob + (unsigned)((row_tile(sgp, si, 8 + (tile >> 4)) * 128 + (sw >> 1) * 64) * DIM + col_tile(sgp, sj, tile & 15) * 128 + 64 * (sw & 1)) * 2;
        const unsigned poison = (dhi & 0x8000u) ? 0x7fc07fc0u : 0u;
#pragma unroll
        for (int c = 0; c < 8; ++c) {
#pragma unroll
          for (int dd = 0; dd < 4; ++dd) {
            const unsigned m = ((c < 4 ? mk.x : mk.y) >> ((c & 3) * 4 + dd)) & 0x00010001u;
            const unsigned t = (ov[c][dd] | (oh[c][dd] << 16)) & 0x7fff7fffu;
            const u16x2 r = __builtin_bit_cast(u16x2, t) * __builtin_bit_cast(u16x2, m);
            ov[c][dd] = __builtin_bit_cast(unsigned, r);
          }
          ov[c][0] |= poison;
          __builtin_amdgcn_raw_buffer_store_b128(ov[c], ors, oo + (c & 1) * 64, (c >> 1) * (16 * DIM * 2), OUT_AUX);
          __builtin_amdgcn_sched_barrier(0);
        }
        slot += 2;
        if (slot >= NSLOT) slot -= NSLOT;
      }
      STAMP(11);
      return;
    }
    __amdgpu_buffer_rsrc_t srs = make_rsrc(p.ws[src], WS_USED);
    bool bad = false;
    if (mb == 0 && tid < 32)
      __hip_atomic_store((unsigned*)(myws + QC_OFF + (((call + 1) & 1) * 32 + tid) * 128), 0u, __ATOMIC_RELAXED, __HIP_MEMORY_SCOPE_AGENT);
    {
      const unsigned ldsbase = (unsigned)(size_t)(LDS_AS char*)lds + 16;
      int k = 0, land = 0;
      unsigned sq = 0;
      int slot = 0;
      auto drain = [&]() {
        asm volatile("s_waitcnt vmcnt(0)" ::: "memory");
        land = k;
        LDS_ST(rland, (unsigned)k);
      };
      auto next_slot = [&]() {
        if (sq >= (unsigned)NSLOT && LDS_LD(rfree + slot) != sq - NSLOT + 1) {
          drain();
          while (LDS_LD(rfree + slot) != sq - NSLOT + 1) {
            if (ring_dead()) { bad = true; break; }
            __builtin_amdgcn_s_sleep(1);
          }
        }
      };
      for (;;) {
        u64 m = LDS_LD((LDS_AS u64*)mbox);
        if (m == 0) {
          asm volatile("s_waitcnt vmcnt(2)" ::: "memory");
          land = max(land, k - 2);
          LDS_ST(rland, (unsigned)land);
          m = LDS_LD((LDS_AS u64*)mbox);
          if (m == 0) {
            drain();
            for (;;) {
              m = LDS_LD((LDS_AS u64*)mbox);
              if (m != 0) break;
              if (ring_dead()) { m = 0x80000000u; break; }
              __builtin_amdgcn_s_sleep(1);
            }
          }
        }
        const unsigned mu = __builtin_amdgcn_readfirstlane((unsigned)m);
        const unsigned flo = mu ? mu : __builtin_amdgcn_readfirstlane((unsigned)(m >> 32));
        LDS_ST(mbox + (mu ? 0 : 1), 0u);
        if (flo >> 31) break;
        next_slot();
        const unsigned copy = (flo >> 30) & 1u, id = flo & 0xfffu, hi = (flo >> 12) & 0x1fffu;
        const unsigned nb = copy ? 8192u : seg_bytes(hi);
        const unsigned np = (nb + 1023) >> 10;
        LDS_ST(rdesc + slot, ((u64)(((sq + 1) << 16) | ((bad || ((flo >> 29) & 1u)) ? 0x8000u : 0u) | (copy << 14) | id) << 32) | (hi << 16) | (unsigned)(k + np));
        const unsigned sb = copy ? hi * 8192u : OB_OFF + id * SEG;
        unsigned vo = sb + l16;
        const unsigned vend = sb + nb;
        unsigned la = ldsbase + slot * SEG;
        for (unsigned i = 0; i < np; ++i) {
          unsigned keep;
          asm volatile("s_waitcnt vmcnt(%0)" :: "i"(ODEPTH - 1) : "memory");
          land = max(land, k - (ODEPTH - 1));
          LDS_ST(rland, (unsigned)land);
          asm volatile("s_mov_b32 %0, m0\n\ts_mov_b32 m0, %3\n\ts_nop 0\n\tbuffer_load_dwordx4 %1, %2, 0 offen sc0 sc1 lds\n\ts_mov_b32 m0, %0"
                       : "=&s"(keep) : "v"(vo < vend ? vo : oob), "s"(srs), "s"(la) : "memory");
          vo += 1024; la += 1024; ++k;
        }
        ++sq;
        if (++slot == NSLOT) slot = 0;
      }
      STAMP(10);
      drain();
      for (int e = 0; e < 2; ++e) {
        next_slot();
        LDS_ST(rdesc + slot, (u64)(((sq + 1) << 16) | 0xfffu) << 32);
        ++sq;
        if (++slot == NSLOT) slot = 0;
      }
    }
    if (__hip_atomic_load(tmo, __ATOMIC_RELAXED, __HIP_MEMORY_SCOPE_AGENT) == call && lane == 0)
      *mark = 0x7fc0;
    return;
  }

  const int gb = blockIdx.x - NMOV;
  const int wm = wv >> 1, hf = wv & 1;
  const unsigned wgt = ((const unsigned*)p.tiles.wg)[gb >> 1] >> (16 * (gb & 1));

#pragma unroll 1
  for (int jn = 0; jn < (gb < 256 - NG ? 2 : 1); ++jn) {
  const int pass = (wgt >> (8 * jn + 7)) & 1, kt = (wgt >> (8 * jn)) & 127, st = kt;
  LDS_AS u64* ctl = (LDS_AS u64*)(lds + LDS_CTRL + 32) + jn;
  f32x4 acc[8][8];
  u32x4 pt[8];
  unsigned early;
  int lane = tid & 63;
  asm volatile("" : "+v"(lane));
  const int rt = pass ? 8 + (kt >> 4) : st >> 4, ct = (pass ? kt : st) & 15;
  {
    const unsigned bx = WS_OFF + (unsigned)(16 + rt) * (NKS * KSB) + lane * 16, bw = WS_OFF + (unsigned)ct * (NKS * KSB) + lane * 16;
    const unsigned vx0 = bx + wm * 8192, vx1 = bx + (wm ^ 1) * 8192;
    const unsigned vw0 = bw + hf * 4096, vw1 = bw + (hf ^ 1) * 4096;
#pragma unroll
    for (int a = 0; a < 8; ++a)
#pragma unroll
      for (int b = 0; b < 8; ++b) acc[a][b] = f32x4{0.f, 0.f, 0.f, 0.f};
    u32x4 fw[3][8], fx[3][8];
#define LDW(dst, vo, so, imm) asm volatile("buffer_load_dwordx4 %0, %1, %2, %3 offen offset:%4 sc1" : "=v"(dst) : "v"(vo), "s"(mrs), "s"(so), "i"(imm))
#define LDX(dst, vo, so, imm) asm volatile("buffer_load_dwordx4 %0, %1, %2, %3 offen offset:%4 sc1" : "=v"(dst) : "v"(vo), "s"(mrs), "s"(so), "i"(imm))
#define MF(T, M, w_, x_) asm volatile("v_mfma_f32_16x16x32_bf16 %0, %1, %2, %0" : "+a"(acc[T][M]) : "v"(w_[T]), "v"(x_[M]))
#define WAITF(w_, x_, n) asm volatile("s_waitcnt vmcnt(" #n ")" : "+v"(w_[0]), "+v"(w_[1]), "+v"(w_[2]), "+v"(w_[3]), "+v"(w_[4]), "+v"(w_[5]), "+v"(w_[6]), "+v"(w_[7]), \
                   "+v"(x_[0]), "+v"(x_[1]), "+v"(x_[2]), "+v"(x_[3]), "+v"(x_[4]), "+v"(x_[5]), "+v"(x_[6]), "+v"(x_[7]))
    auto load1 = [&](int i, unsigned sb, int kk, u32x4* w_, u32x4* x_) {
      const int q = i >> 2, j = i & 3;
      if (q == 0) LDW(w_[j], vw0, sb + kk * 8192, j * 1024);
      else if (q == 1) LDX(x_[j], vx0, sb + (j >> 1) * 4096, (j & 1) * 2048 + kk * 1024);
      else if (q == 2) LDW(w_[4 + j], vw1, sb + kk * 8192, j * 1024);
      else LDX(x_[4 + j], vx1, sb + (j >> 1) * 4096, (j & 1) * 2048 + kk * 1024);
    };
    auto half = [&](u32x4* w_, u32x4* x_, bool doload, unsigned sb, int kk, u32x4* wn, u32x4* xn) {
#pragma unroll
      for (int i = 0; i < 16; ++i) {
        const int qa = i >> 3, qb = (i >> 2) & 1, m = i & 3;
#pragma unroll
        for (int t = 0; t < 4; ++t) MF((qa ^ qb) * 4 + t, qa * 4 + m, w_, x_);
        if (doload) load1(i, sb, kk, wn, xn);
      }
    };
    const int lj = lane & 7;
    const unsigned ksv = (unsigned)(lj < 5 ? ((wv - lj) & 3) * 5 + lj : 15 + ((wv - lj) & 3) * 3 + lj);
    const unsigned foff = iflag(lane < 32 ? ((unsigned)((lane & 8) ? ct : 16 + rt) * 32 + ksv) * 2 + (lane >> 4)
                                          : PS_OFF / 8192 + (unsigned)(kt * 4 + wv));
    unsigned todo = 0xffu, known = 0, fv = 0, warm = 0xffffu;
    auto pick = [&]() -> unsigned {
      u64 rd = __ballot(fv >= call);
      rd &= rd >> 16;
      unsigned m = (((unsigned)rd & (unsigned)(rd >> 8)) | known) & todo;
      if (m == 0) {
        u64 t0 = __builtin_amdgcn_s_memrealtime();
        for (unsigned spin = 1;; ++spin) {
          const unsigned v = __hip_atomic_load((unsigned*)(myws + foff), __ATOMIC_RELAXED, __HIP_MEMORY_SCOPE_SYSTEM);
          rd = __ballot(v >= call);
          rd &= rd >> 16;
          m = (unsigned)rd & (unsigned)(rd >> 8) & todo;
          if (m) break;
          if ((spin & 15) == 0 && __builtin_amdgcn_readfirstlane(timed_out(tmo, call, t0))) { m = todo; cnt[4] = 1; break; }
          unsigned cand = (unsigned)rd & warm & (todo * 0x101u);
          if (cand) {
            warm &= ~cand;
            unsigned a = 0;
            const unsigned vo = (tid & 63) * (256 / PFN);
            for (; cand; cand &= cand - 1) {
              const unsigned i = __builtin_ctz(cand);
              const unsigned so = WS_OFF + ((unsigned)(i < 8 ? 16 + rt : ct) * NKS + (unsigned)__builtin_amdgcn_readlane(ksv, i)) * KSB;
#pragma unroll
              for (int q = 0; q < PFN; ++q)
                asm volatile("buffer_load_dword %0, %1, %2, %3 offen sc1" : "+v"(a) : "v"(vo), "s"(mrs), "s"(so + q * (KSB / PFN)));
            }
            asm volatile("s_waitcnt vmcnt(0)" : "+v"(a));
          } else
          __builtin_amdgcn_s_sleep(PSLEEP);
        }
      }
      const unsigned j = __builtin_ctz(m);
      todo &= ~(1u << j);
      known = SKIPF ? m & todo : 0u;
      return (unsigned)__builtin_amdgcn_readlane(ksv, j) * KSB;
    };
#define LDF() asm volatile("buffer_load_dword %0, %1, %2, 0 offen sc0 sc1" : "=v"(fv) : "v"(foff | (known ? 0xF0000000u : 0u)), "s"(mrs))
    {
      const unsigned sb = pick();
      asm volatile("s_waitcnt vmcnt(0)\n\ts_nop 4" ::: "memory");
      LDF();
#pragma unroll
      for (int i = 0; i < 16; ++i) load1(i, sb, 0, fw[0], fx[0]);
#pragma unroll
      for (int i = 0; i < 16; ++i) load1(i, sb, 1, fw[1], fx[1]);
    }
#define KSTEP(u) do { \
        WAITF(fw[(u) % 3], fx[(u) % 3], 16); \
        asm volatile("" : "+v"(fv)); \
        const unsigned sb = pick(); \
        asm volatile("s_nop 4"); \
        LDF(); \
        half(fw[(u) % 3], fx[(u) % 3], true, sb, 0, fw[((u) + 2) % 3], fx[((u) + 2) % 3]); \
        WAITF(fw[((u) + 1) % 3], fx[((u) + 1) % 3], 17); \
        half(fw[((u) + 1) % 3], fx[((u) + 1) % 3], true, sb, 1, fw[(u) % 3], fx[(u) % 3]); } while (0)
#pragma unroll 1
    for (int s = 0; s < 2; ++s) { KSTEP(0); KSTEP(2); KSTEP(4); }
    KSTEP(0);
    WAITF(fw[2], fx[2], 16);
    half(fw[2], fx[2], false, 0, 0, fw[0], fx[0]);
    WAITF(fw[0], fx[0], 0);
    half(fw[0], fx[0], false, 0, 0, fw[0], fx[0]);
    asm volatile("" : "+v"(fv));
    early = (unsigned)(__ballot(fv >= call) >> 32) & (unsigned)pass;
    asm volatile("s_nop 15\n\ts_nop 15" ::: "memory");
    __builtin_amdgcn_sched_barrier(0);
  }
  STAMP(2 + 3 * jn);
  asm volatile("" : "+v"(lane));
  f32x4 r[4][4];
  const int prk = (gq ^ 1) * 4 + (rt & 1) * 2 + (ct & 1), ptl = (kt & 0x6e) + ri * 16 + rj, ftl = pass ? kt : ptl;
  const bool direct = (p.direct.mask[ftl >> 5] >> (ftl & 31)) & 1u;
  const __amdgpu_buffer_rsrc_t prs = make_rsrc(direct ? p.ws[prk] : myws, WS_USED);
  const unsigned po = (direct ? PB_OFF + (unsigned)(ptl * 4 + wv) * 8192 : PS_OFF + (unsigned)(kt * 4 + wv) * 8192) + lane * 16;
  char* wq = p.ws[pass ? lane & 7 : prk];
  {
    if (early) {
#pragma unroll
      for (int c = 0; c < 8; ++c) pt[c] = __builtin_amdgcn_raw_buffer_load_b128(prs, po + c * 1024, 0, AUX_SYS);
    }
    unsigned a0 = (unsigned)(size_t)(LDS_AS char*)lds + wv * 32768 + lane * 16;
    asm volatile("" : "+v"(a0));
    const unsigned a1 = a0 ^ 32768u, a3 = a0 ^ (3u * 32768u);
#pragma unroll
    for (int m = 7; m >= 0; --m)
#pragma unroll
      for (int t = 0; t < 4; ++t) *(LDS_AS f32x4*)(size_t)(a0 + (m * 4 + t) * 1024) = acc[4 + t][m];
    f32x4 hi[16], u[16], v[16];
#pragma unroll
    for (int i = 0; i < 16; ++i) { hi[i] = acc[i & 3][4 + (i >> 2)]; r[i & 3][i >> 2] = acc[i & 3][i >> 2]; asm volatile("" : "+v"(hi[i]), "+v"(r[i & 3][i >> 2])); }
#define O16(x) "=&v"(x[0]), "=&v"(x[1]), "=&v"(x[2]), "=&v"(x[3]), "=&v"(x[4]), "=&v"(x[5]), "=&v"(x[6]), "=&v"(x[7]), \
               "=&v"(x[8]), "=&v"(x[9]), "=&v"(x[10]), "=&v"(x[11]), "=&v"(x[12]), "=&v"(x[13]), "=&v"(x[14]), "=&v"(x[15])
#define READ16(o) "ds_read_b128 %0, %16 offset:" #o "\n\tds_read_b128 %1, %16 offset:" #o "+1024\n\tds_read_b128 %2, %16 offset:" #o "+2048\n\t" \
      "ds_read_b128 %3, %16 offset:" #o "+3072\n\tds_read_b128 %4, %16 offset:" #o "+4096\n\tds_read_b128 %5, %16 offset:" #o "+5120\n\t" \
      "ds_read_b128 %6, %16 offset:" #o "+6144\n\tds_read_b128 %7, %16 offset:" #o "+7168\n\tds_read_b128 %8, %16 offset:" #o "+8192\n\t" \
      "ds_read_b128 %9, %16 offset:" #o "+9216\n\tds_read_b128 %10, %16 offset:" #o "+10240\n\tds_read_b128 %11, %16 offset:" #o "+11264\n\t" \
      "ds_read_b128 %12, %16 offset:" #o "+12288\n\tds_read_b128 %13, %16 offset:" #o "+13312\n\tds_read_b128 %14, %16 offset:" #o "+14336\n\t" \
      "ds_read_b128 %15, %16 offset:" #o "+15360\n\ts_waitcnt lgkmcnt(0)"
    asm volatile("s_waitcnt lgkmcnt(0)\n\ts_barrier\n\t" READ16(16384) : O16(u) : "v"(a1) : "memory");
#pragma unroll
    for (int i = 0; i < 16; ++i) *(LDS_AS f32x4*)(size_t)(a1 + 16384 + i * 1024) = u[i] + hi[i];
    asm volatile(READ16(0) "\n\ts_barrier" : O16(u) : "v"(a1) : "memory");
    asm volatile(READ16(16384) : O16(v) : "v"(a3) : "memory");
#pragma unroll
    for (int i = 0; i < 16; ++i) r[i & 3][i >> 2] = (r[i & 3][i >> 2] + u[i]) + v[i];
    if (cnt[4]) r[0][0][0] = __uint_as_float(0x7fc00000u);
  }
  STAMP(3 + 3 * jn);
  if (pass == 0) {
    unsigned slot = 1;
    asm volatile("buffer_atomic_add %0, %1, %2, 0 offen sc0" : "+v"(slot)
                 : "v"(tid == 0 && !direct ? QC_OFF + (unsigned)((call & 1) * 32 + 24 + (prk & 3)) * 128 : 0xF0000000u), "s"(mrs) : "memory");
#pragma unroll
    for (int c = 0; c < 8; ++c) {
      const f32x4 a = r[c >> 1][(c & 1) * 2], b = r[c >> 1][(c & 1) * 2 + 1];
      const u32x4 pk = {pack_bf16(a[0], a[1]), pack_bf16(a[2], a[3]), pack_bf16(b[0], b[1]), pack_bf16(b[2], b[3])};
      __builtin_amdgcn_raw_buffer_store_b128(pk, mrs, PB_OFF + (unsigned)(st * 4 + wv) * 8192 + c * 1024 + lane * 16, 0, AUX_SYS);
    }
    asm volatile("s_waitcnt vmcnt(0)" : "+v"(slot), "+v"(wq) :: "memory");
    if (direct && lane == 0)
      __hip_atomic_store((unsigned*)(wq + iflag(PS_OFF / 8192 + ptl * 4 + wv)), call, __ATOMIC_RELAXED, __HIP_MEMORY_SCOPE_SYSTEM);
    __syncthreads();
    slot = __builtin_amdgcn_readfirstlane(slot);
    if (!direct && tid < 4 && slot < 32u)
      __hip_atomic_store((u64*)(wq + oflag(rank, 512 + slot * 4 + tid)), ((u64)call << 32) | ((unsigned)st << 14), __ATOMIC_RELAXED, __HIP_MEMORY_SCOPE_SYSTEM);
  } else {
    if (!early) {
      unsigned* pf = (unsigned*)(myws + iflag(PS_OFF / 8192 + kt * 4 + wv));
      u64 t0 = __builtin_amdgcn_s_memrealtime();
      for (unsigned spin = 1; __hip_atomic_load(pf, __ATOMIC_RELAXED, __HIP_MEMORY_SCOPE_SYSTEM) < call; ++spin) {
        if ((spin & 15) == 0 && __builtin_amdgcn_readfirstlane(timed_out(tmo, call, t0))) { r[0][0][0] = __uint_as_float(0x7fc00000u); break; }
        __builtin_amdgcn_s_sleep(1);
      }
      asm volatile("" ::: "memory");
#pragma unroll
      for (int c = 0; c < 8; ++c) pt[c] = __builtin_amdgcn_raw_buffer_load_b128(prs, po + c * 1024, 0, AUX_SYS);
    }
#pragma unroll
    for (int i = 0; i < 16; ++i) {
      const unsigned a = pt[i >> 1][(i & 1) * 2], b = pt[i >> 1][(i & 1) * 2 + 1];
      r[i >> 2][i & 3] += __builtin_bit_cast(f32x4, u32x4{a << 16, a & 0xffff0000u, b << 16, b & 0xffff0000u});
      asm volatile("" : "+v"(r[i >> 2][i & 3]));
    }
    STAMP(10 + 3 * jn);
    LDS_AS char* seg = (LDS_AS char*)(lds + (wv ^ 1) * 32768);
    u32x4 pk[8];
    unsigned E[2] = {0u, 0u};
#pragma unroll
    for (int c = 0; c < 8; ++c) {
      unsigned t[4], s_[4];
#pragma unroll
      for (int dd = 0; dd < 4; ++dd)
        t[dd] = pack_bf16(r[(dd & 1) * 2][c >> 1][(c & 1) * 2 + (dd >> 1)], r[(dd & 1) * 2 + 1][c >> 1][(c & 1) * 2 + (dd >> 1)]);
      asm("v_pk_sub_u16 %4, %0, %8 clamp\n\tv_pk_sub_u16 %5, %1, %8 clamp\n\tv_pk_sub_u16 %6, %2, %8 clamp\n\tv_pk_sub_u16 %7, %3, %8 clamp\n\t"
          "v_pk_max_i16 %0, %0, 0\n\tv_pk_max_i16 %1, %1, 0\n\tv_pk_max_i16 %2, %2, 0\n\tv_pk_max_i16 %3, %3, 0\n\t"
          "v_pk_mad_i16 %0, %4, %9, %0 clamp\n\tv_pk_mad_i16 %1, %5, %9, %1 clamp\n\tv_pk_mad_i16 %2, %6, %9, %2 clamp\n\tv_pk_mad_i16 %3, %7, %9, %3 clamp\n\t"
          "v_pk_min_u16 %4, %0, 1 op_sel_hi:[1,0]\n\tv_pk_min_u16 %5, %1, 1 op_sel_hi:[1,0]\n\tv_pk_min_u16 %6, %2, 1 op_sel_hi:[1,0]\n\tv_pk_min_u16 %7, %3, 1 op_sel_hi:[1,0]"
          : "+v"(t[0]), "+v"(t[1]), "+v"(t[2]), "+v"(t[3]), "=&v"(s_[0]), "=&v"(s_[1]), "=&v"(s_[2]), "=&v"(s_[3]) : "s"(0xff80ff80u), "s"(0x7fff7fffu));
      pk[c] = u32x4{t[0], t[1], t[2], t[3]};
      E[c >> 2] |= ((s_[0] | (s_[1] << 1)) | ((s_[2] | (s_[3] << 1)) << 2)) << ((c & 3) * 4);
    }
    const unsigned cn = __builtin_popcount(E[0]) + __builtin_popcount(E[1]);
    const unsigned inc = wave_scan(cn);
    const unsigned total = __builtin_amdgcn_readlane(inc, 63);
    const unsigned H = total >> 5;
    {
      const unsigned xo = ((2 * total + 3) & ~3u) - 4 * H;
      *(LDS_AS u32x2*)(seg + SEG_TBL + 512 + lane * 8) = u32x2{2u * lane < H ? E[0] : 0u, 2u * lane + 1 < H ? E[1] : 0u};
      if (lane == 63) *(LDS_AS unsigned short*)(seg + 2 * total) = 0;
      if (2u * lane >= H) *(LDS_AS unsigned*)(seg + xo + lane * 8) = E[0];
      if (2u * lane + 1 >= H) *(LDS_AS unsigned*)(seg + xo + lane * 8 + 4) = E[1];
    }
    unsigned wa = (unsigned)(size_t)seg + 2 * (inc - cn);
    unsigned wb = wa + 2 * __builtin_popcount(E[0]);
#pragma unroll
    for (int c = 0; c < 4; ++c)
#pragma unroll
      for (int dd = 0; dd < 4; dd += 2) {
        u64 m[8];
#define CW(i, a, ins, v) "s_mov_b64 exec, %" #i "\n\t" ins " %" #a ", %" #v "\n\tv_add_u32 %" #a ", 2, %" #a "\n\t"
        asm volatile("v_cmp_ne_u16 %2, 0, %10\n\tv_cmp_ne_u16 %3, 0, %12\n\tv_cmp_lt_u32 %4, %14, %10\n\tv_cmp_lt_u32 %5, %14, %12\n\t"
                     "v_cmp_ne_u16 %6, 0, %11\n\tv_cmp_ne_u16 %7, 0, %13\n\tv_cmp_lt_u32 %8, %14, %11\n\tv_cmp_lt_u32 %9, %14, %13\n\t"
                     CW(2, 0, "ds_write_b16", 10) CW(3, 1, "ds_write_b16", 12) CW(4, 0, "ds_write_b16_d16_hi", 10) CW(5, 1, "ds_write_b16_d16_hi", 12)
                     CW(6, 0, "ds_write_b16", 11) CW(7, 1, "ds_write_b16", 13) CW(8, 0, "ds_write_b16_d16_hi", 11) CW(9, 1, "ds_write_b16_d16_hi", 13)
                     "s_mov_b64 exec, -1"
                     : "+v"(wa), "+v"(wb), "=&s"(m[0]), "=&s"(m[1]), "=&s"(m[2]), "=&s"(m[3]), "=&s"(m[4]), "=&s"(m[5]), "=&s"(m[6]), "=&s"(m[7])
                     : "v"(pk[c][dd]), "v"(pk[c][dd + 1]), "v"(pk[4 + c][dd]), "v"(pk[4 + c][dd + 1]), "s"(0xffffu) : "memory");
      }
    STAMP(11 + 3 * jn);
    const unsigned nb = seg_bytes(total);
    const unsigned so = OB_OFF + (unsigned)(kt * 4 + wv) * SEG + lane * 16;
    unsigned slot = 1;
    asm volatile("buffer_atomic_add %0, %1, %2, 0 offen sc0" : "+v"(slot)
                 : "v"(tid == 0 ? QC_OFF + (unsigned)((call & 1) * 32 + 28) * 128 : 0xF0000000u), "s"(mrs) : "memory");
    {
      u32x4 v[9];
      u32x2 e[4];
      asm volatile("ds_read2_b32 %9, %14 offset0:0 offset1:16\n\tds_read_b128 %0, %13\n\tds_read_b128 %1, %13 offset:1024\n\t"
                   "ds_read2_b32 %10, %14 offset0:32 offset1:48\n\tds_read_b128 %2, %13 offset:2048\n\tds_read_b128 %3, %13 offset:3072\n\t"
                   "ds_read2_b32 %11, %14 offset0:64 offset1:80\n\tds_read_b128 %4, %13 offset:4096\n\tds_read_b128 %5, %13 offset:5120\n\t"
                   "ds_read2_b32 %12, %14 offset0:96 offset1:112\n\tds_read_b128 %6, %13 offset:6144\n\tds_read_b128 %7, %13 offset:7168\n\t"
                   "ds_read_b128 %8, %13 offset:8192\n\ts_waitcnt lgkmcnt(0)"
                   : "=&v"(v[0]), "=&v"(v[1]), "=&v"(v[2]), "=&v"(v[3]), "=&v"(v[4]), "=&v"(v[5]), "=&v"(v[6]), "=&v"(v[7]), "=&v"(v[8]),
                     "=&v"(e[0]), "=&v"(e[1]), "=&v"(e[2]), "=&v"(e[3])
                   : "v"((unsigned)(size_t)seg + lane * 16), "v"((unsigned)(size_t)seg + SEG_TBL + 512 + (lane >> 2) * 4) : "memory");
#pragma unroll
      for (int i = 0; i < 9; ++i) {
        if (i < 8) {
          const unsigned x = e[i >> 1][i & 1] >> ((lane & 3) * 4);
#pragma unroll
          for (int d = 0; d < 4; ++d) v[i][d] |= (x << (15 - d)) & 0x80008000u;
        }
        __builtin_amdgcn_raw_buffer_store_b128(v[i], mrs, (unsigned)(i * 1024 + lane * 16) < nb ? so + i * 1024 : 0xF0000000u, 0, AUX_SYS);
      }
    }
    asm volatile("s_waitcnt vmcnt(0)" : "+v"(slot), "+v"(wq) :: "memory");
    STAMP(12 + 3 * jn);
    if (tid == 0) cnt[16] = slot;
    asm volatile("" ::: "memory");
    u64 old = 0;
    const u64 mine = (1ull << 56) | ((u64)total << (14 * wv));
    if (lane == 0) old = __hip_atomic_fetch_add(ctl, mine, __ATOMIC_RELAXED, __HIP_MEMORY_SCOPE_WORKGROUP);
    slot = *(volatile LDS_AS unsigned*)(cnt + 16);
    old = (((u64)(unsigned)__builtin_amdgcn_readfirstlane((unsigned)(old >> 32)) << 32) | (unsigned)__builtin_amdgcn_readfirstlane((unsigned)old)) + mine;
    if ((old >> 56) == 4 && lane < 32 && (lane & 7) != rank && slot < 128u) {
      const u64 f = ((u64)call << 32) | ((unsigned)kt << 14) | (unsigned)((old >> (14 * (lane >> 3))) & 0x3fffu);
      __hip_atomic_store((u64*)(wq + oflag(rank, slot * 4 + (lane >> 3))), f, __ATOMIC_RELAXED, __HIP_MEMORY_SCOPE_SYSTEM);
    }
    const unsigned oo = (unsigned)((row_tile(gq, ri, 8 + (kt >> 4)) * 128 + wm * 64 + (lane & 15)) * DIM + col_tile(gq, rj, kt & 15) * 128 + 64 * hf + 8 * (lane >> 4)) * 2;
#pragma unroll
    for (int c = 0; c < 8; ++c) __builtin_amdgcn_raw_buffer_store_b128(pk[c], ors, oo + (c & 1) * 64, (c >> 1) * (16 * DIM * 2), OUT_AUX);
    __builtin_amdgcn_s_barrier();
  }
  STAMP(4 + 3 * jn);
  }
}

}

extern "C" void kernel_launch(int rank, int nranks, ncclComm_t comm,
                              hipStream_t stream,
                              void* const* d_in, const int* in_sizes, int n_in,
                              void* d_out, int out_size,
                              const int* in_dtypes, int out_dtype,
                              void* const* d_out_all,
                              void* const* d_ws_all, size_t ws_bytes) {
  (void)comm; (void)in_sizes; (void)n_in; (void)out_size; (void)in_dtypes; (void)out_dtype; (void)d_out_all;
  if (nranks != NR || (unsigned)rank >= (unsigned)NR || ws_bytes < (size_t)WS_USED) {
    fprintf(stderr, "unsupported configuration\n");
    return;
  }
  Params p;
  p.x = (const unsigned short*)d_in[0];
  p.w = (const unsigned short*)d_in[1];
  p.out = (unsigned short*)d_out;
  for (int i = 0; i < NR; ++i) p.ws[i] = (char*)d_ws_all[i];
  p.myws = p.ws[rank];
  p.rank = rank;
  constexpr OrderTable kOrder = make_order();
  constexpr PrepTable kPrep = make_prep();
  p.order = kOrder;
  p.prep = kPrep;
  constexpr TileTable kTiles = make_tiles();
  p.tiles = kTiles;
  constexpr DirectTable kDirect = make_direct();
  p.direct = kDirect;
  gemm_ar_kernel<<<256, 256, 0, stream>>>(p);
  HIP_CHECK(hipGetLastError());
}
```

```cpp
#include <hip/hip_runtime.h>
#include <rccl/rccl.h>
#include <cstdio>
#include <cstdint>

#define HIP_CHECK(expr) do { hipError_t _e = (expr); if (_e != hipSuccess) { \
    fprintf(stderr, "HIP error %s at %s:%d\n", hipGetErrorString(_e), __FILE__, __LINE__); \
    return; } } while (0)

namespace {

typedef float f32x4 __attribute__((ext_vector_type(4)));
typedef unsigned u32x4 __attribute__((ext_vector_type(4)));
typedef unsigned u32x2 __attribute__((ext_vector_type(2)));
typedef unsigned long long u64;

constexpr int NR = 8;
constexpr int DIM = 4096;
constexpr int KSL = DIM / NR;
constexpr int NKS = 32;
constexpr unsigned KSB = 16384;
constexpr unsigned SEG = 9216;
constexpr unsigned SEG_TBL = 8704;
__host__ __device__ constexpr unsigned seg_bytes(unsigned n) { return ((2 * n + 3) & ~3u) + 4 * (128 - (n >> 5)); }

constexpr unsigned WS_OFF = 0;
constexpr unsigned PS_OFF = WS_OFF + 32 * NKS * KSB;
constexpr unsigned SB_OFF = PS_OFF + 128 * 2 * KSB;
constexpr unsigned PB_OFF = SB_OFF + 8 * 2 * 32 * KSB;
constexpr unsigned OB_OFF = PB_OFF + 128 * 2 * KSB;
constexpr unsigned IF_OFF = OB_OFF + 128 * 4 * SEG;
constexpr unsigned PF_OFF = IF_OFF + 1280 * 128;
constexpr unsigned OF_OFF = PF_OFF + NR * 256 * 128;
constexpr unsigned TK_OFF = OF_OFF + NR * 160 * 128;
constexpr unsigned QC_OFF = TK_OFF + 32 * 128;
constexpr unsigned TMO_OFF = QC_OFF + 64 * 128;
constexpr unsigned WS_USED = TMO_OFF + 128;
__host__ __device__ constexpr unsigned iflag(unsigned i) { return IF_OFF + (i >> 1) * 128 + (i & 1) * 4; }
__host__ __device__ constexpr unsigned oflag(unsigned src, unsigned e) { return OF_OFF + (src * 160 + (e >> 2)) * 128 + (e & 3) * 8; }

constexpr int NMOV = 7 * 16, NG = 256 - NMOV;
constexpr int ODEPTH = 4;
constexpr int NSLOT = 12;
constexpr unsigned NEARLY = 64;

constexpr u64 DEADLINE_TICKS = 50000000ull;

constexpr int LDS_CTRL = 131072;
constexpr int LDS_BYTES = LDS_CTRL + 256;

__host__ __device__ constexpr int row_tile(int g, int h, int r) { return (h << (4 - g)) | ((r & 1) << (3 + g)) | (((r >> 3) ^ g) << 2) | ((r >> 1) & 3); }
__host__ __device__ constexpr int col_tile(int g, int h, int c) { return (h << (4 - g)) | ((c & 1) << (3 + g)) | (c >> 1); }

struct PosTable { int p[32]; };
constexpr PosTable make_pos() {
  const int seq[32] = {16, 0, 17, 1, 2, 3, 18, 19, 4, 5, 24, 25, 6, 7, 20, 21, 8, 9, 26, 27, 10, 11, 22, 23, 12, 13, 28, 29, 14, 15, 30, 31};
  PosTable o{};
  for (int i = 0; i < 32; ++i) o.p[seq[i]] = i;
  return o;
}
constexpr PosTable kPos = make_pos();
constexpr int unit_pos(int u) { return kPos.p[u]; }
constexpr int cmax(int a, int b) { return a > b ? a : b; }
constexpr int send_key(int r, int c) { return cmax(unit_pos(16 + r), unit_pos(c)); }
constexpr int kept_key(int kt) {
  return cmax(cmax(unit_pos(24 + (kt >> 4)), unit_pos(kt & 15)),
              cmax(send_key((kt >> 4) | 1, (kt & 15) | 1), send_key((kt >> 4) & 6, kt & 14)) + 6);
}
struct alignas(4) TileTable { unsigned char wg[2 * NG]; };
constexpr TileTable make_tiles() {
  TileTable o{};
  int nj[8] = {};
  for (int key = 0; key < 64; ++key)
    for (int t = 0; t < 256; ++t) {
      const int x = t & 7;
      if ((t < 128 ? send_key(t >> 4, t & 15) : kept_key(t & 127)) == key) {
        o.wg[2 * ((nj[x] % (NG / 8)) * 8 + x) + nj[x] / (NG / 8)] = (unsigned char)t;
        ++nj[x];
      }
    }
  return o;
}
struct DirectTable { unsigned mask[4]; unsigned queued[4]; };
constexpr DirectTable make_direct() {
  DirectTable o{};
  for (int t = 0; t < 128; ++t) {
    const int own = cmax(unit_pos(24 + (t >> 4)), unit_pos(t & 15)), part = send_key((t >> 4) | 1, (t & 15) | 1);
    if (own <= 13 && own <= part + 4) o.mask[t >> 5] |= 1u << (t & 31);
    else o.queued[((t >> 4) & 1) * 2 + (t & 1)] += 4;
  }
  return o;
}
struct alignas(4) OrderTable { unsigned char t[2][160]; };
constexpr OrderTable make_order() {
  OrderTable o{};
  for (int x = 0; x < 2; ++x) {
    int n = 0;
    const int nk = x ? 3 : 5;
    for (int key = 0; key <= 31 + 4; ++key)
      for (int k = nk - 1; k >= 0; --k)
        for (int u = 0; u < 32; ++u)
          if (unit_pos(u) + k == key) o.t[x][n++] = (unsigned char)(u * 8 + k);
  }
  return o;
}
constexpr int NPUSH = 6;
struct alignas(4) PrepTable { unsigned char t[256]; unsigned push[16]; };
constexpr PrepTable make_prep() {
  PrepTable o{};
  int n = 0;
  for (int key = 0; key <= 5 * (31 + 4); ++key)
    for (int j = 0; j < 256; ++j) {
      const int ksl = j >> 5, t = j & 15, a = unit_pos(t), b = unit_pos(16 + t), k = ksl < 5 ? ksl : ksl - 5;
      if (((a < b ? a : b) + k) * (ksl < 5 ? 5 : 3) == key) {
        o.push[n >> 4] |= (unsigned)((a + k <= 2) + 2 * (b + k <= 2)) << (2 * (n & 15));
        o.t[n++] = (unsigned char)j;
      }
    }
  return o;
}

struct Params {
  const unsigned short* x;
  const unsigned short* w;
  unsigned short* out;
  char* myws;
  char* ws[NR];
  int rank;
  OrderTable order;
  PrepTable prep;
  TileTable tiles;
  DirectTable direct;
};

#define LDS_AS __attribute__((address_space(3)))

__device__ __forceinline__ __amdgpu_buffer_rsrc_t make_rsrc(const void* p, unsigned bytes) {
  return __builtin_amdgcn_make_buffer_rsrc((void*)p, 0, bytes, 0x00020000);
}
constexpr int AUX_SYS = 17;
constexpr int OUT_AUX = 16;

__device__ __forceinline__ bool timed_out(unsigned* tmo, unsigned call, u64 t0) {
  if (__hip_atomic_load(tmo, __ATOMIC_RELAXED, __HIP_MEMORY_SCOPE_AGENT) == call ||
      __builtin_amdgcn_s_memrealtime() - t0 > DEADLINE_TICKS) {
    __hip_atomic_store(tmo, call, __ATOMIC_RELAXED, __HIP_MEMORY_SCOPE_AGENT);
    return true;
  }
  return false;
}

__device__ __forceinline__ unsigned pack_bf16(float a, float b) {
  typedef __bf16 bf2 __attribute__((ext_vector_type(2)));
  typedef float f2 __attribute__((ext_vector_type(2)));
  f2 v = {a, b};
  bf2 r = __builtin_convertvector(v, bf2);
  return __builtin_bit_cast(unsigned, r);
}

__device__ __forceinline__ unsigned wave_scan(unsigned x) {
  x += __builtin_amdgcn_update_dpp(0u, x, 0x111, 0xf, 0xf, false);
  x += __builtin_amdgcn_update_dpp(0u, x, 0x112, 0xf, 0xf, false);
  x += __builtin_amdgcn_update_dpp(0u, x, 0x114, 0xf, 0xf, false);
  x += __builtin_amdgcn_update_dpp(0u, x, 0x118, 0xf, 0xf, false);
  x += __builtin_amdgcn_update_dpp(0u, x, 0x142, 0xa, 0xf, false);
  x += __builtin_amdgcn_update_dpp(0u, x, 0x143, 0xc, 0xf, false);
  return x;
}
typedef unsigned short u16x2 __attribute__((ext_vector_type(2)));

__global__ void __launch_bounds__(256, 1) gemm_ar_kernel(Params p) {
  __shared__ __attribute__((aligned(16))) char lds[LDS_BYTES];
  const int tid = threadIdx.x;
  const int lane = tid & 63;
  const int wv = __builtin_amdgcn_readfirstlane(tid >> 6);
  const int rank = p.rank;
  char* myws = p.myws;
  unsigned* tmo = (unsigned*)(myws + TMO_OFF);
  const int r16 = lane & 15, g4 = lane >> 4;

  const int gq = rank >> 2, ri = (rank >> 1) & 1, rj = rank & 1;
  __amdgpu_buffer_rsrc_t mrs = make_rsrc(myws, WS_USED);
  __amdgpu_buffer_rsrc_t ors = make_rsrc(p.out, (unsigned)DIM * DIM * 2);
  LDS_AS unsigned* cnt = (LDS_AS unsigned*)(lds + LDS_CTRL);
  unsigned call;
  {
    const unsigned job = blockIdx.x >= 224u ? blockIdx.x - 224u : (blockIdx.x - 80u) & 255u;
    unsigned jobw = ((const unsigned*)p.prep.t)[job >> 2];
    const unsigned jobw2 = ((const unsigned*)p.prep.t)[blockIdx.x >> 2];
    unsigned push = p.prep.push[job >> 4];
    const unsigned short* const px = p.x;
    const unsigned short* const pw = p.w;
    asm volatile("" :: "s"(px), "s"(pw), "s"(jobw), "s"(jobw2), "s"(push), "s"(rank), "s"(myws));
    unsigned ticket = 1;
    asm volatile("buffer_atomic_add %0, %1, %2, 0 offen sc0" : "+v"(ticket) : "v"(lane == 0 ? TK_OFF + (blockIdx.x & 31) * 128 : 0xF0000000u), "s"(mrs) : "memory");
    char* wsl = p.ws[0];
#pragma unroll
    for (int i = 1; i < NR; ++i) wsl = (lane & 7) == i ? p.ws[i] : wsl;
    const int cbr = 8 * ((r16 & 3) >> 1) + 2 * (r16 >> 2) + (r16 & 1);
    auto load_w = [&](u32x4* r, int t, int k, int qt) {
      const unsigned short* q = pw + (size_t)(k * 64 + (qt >> 1) * 32 + 8 * g4) * DIM + t * 128 + 64 * (qt & 1) + 4 * cbr;
#pragma unroll
      for (int h = 0; h < 4; ++h) {
        const u32x2 a = *(const u32x2*)(q + (size_t)(2 * h) * DIM), b = *(const u32x2*)(q + (size_t)(2 * h + 1) * DIM);
        r[h] = u32x4{a.x, a.y, b.x, b.y};
      }
    };
    auto load_x = [&](u32x4* r, int t, int k, int qt) {
      const unsigned short* q = px + (size_t)(t * 128 + (qt & 1) * 64 + r16) * KSL + k * 64 + (qt >> 1) * 32 + 8 * g4;
#pragma unroll
      for (int h = 0; h < 4; ++h) r[h] = *(const u32x4*)(q + (size_t)h * 16 * KSL);
    };
    auto transpose_w = [&](u32x4* r) {
      u32x4 v[4];
#pragma unroll
      for (int t = 0; t < 4; ++t)
#pragma unroll
        for (int h = 0; h < 4; ++h) {
          const unsigned lo = r[h][t >> 1], hi = r[h][2 + (t >> 1)];
          v[t][h] = __builtin_amdgcn_perm(hi, lo, (t & 1) ? 0x07060302u : 0x05040100u);
        }
#pragma unroll
      for (int t = 0; t < 4; ++t) r[t] = v[t];
    };
    if (tid < 64) cnt[tid] = 0;
    asm volatile("s_waitcnt lgkmcnt(0)" ::: "memory");
    __builtin_amdgcn_s_barrier();
    const int njob = blockIdx.x < NMOV ? 0 : blockIdx.x >= NG ? 2 : 1;
    push = (push >> (2 * (job & 15))) & 3u;
#pragma unroll 1
    for (int jn = 0; jn < njob; ++jn) {
      const int jb = (jobw >> (8 * (job & 3))) & 255, ksl = jb >> 5, hh = (jb >> 4) & 1, tt = jb & 15;
      const int tg = ksl < 5 ? gq : gq ^ 1, kq = ksl < 5 ? ksl : ksl - 5;
      const int gtw = col_tile(tg, hh, tt), gtx = row_tile(tg, hh, tt);
      if (jn == 0 && job >= NEARLY) {
        __builtin_amdgcn_s_sleep(40);
        for (unsigned i = NEARLY; i < job; i += 8) __builtin_amdgcn_s_sleep(12);
      }
      if (jn) __builtin_amdgcn_s_sleep(60);
      u32x4 wr[4], xr[4];
      load_w(wr, gtw, ksl, wv);
      load_x(xr, gtx, ksl, wv);
      const unsigned sb = SB_OFF + (unsigned)((ksl * 2 + hh) * 32 + tt) * KSB + wv * 4096 + lane * 16;
      const unsigned lb = WS_OFF + (unsigned)(tt * NKS + (rank & 3) * 5 + kq) * KSB + wv * 4096 + lane * 16;
      const unsigned lw = (ksl < 5 && hh == rj) ? lb : 0xF0000000u, lx = (ksl < 5 && hh == ri) ? lb + 16 * NKS * KSB : 0xF0000000u;
      transpose_w(wr);
#pragma unroll
      for (int t = 0; t < 4; ++t) __builtin_amdgcn_raw_buffer_store_b128(wr[t], mrs, sb + t * 1024, 0, AUX_SYS);
#pragma unroll
      for (int h = 0; h < 4; ++h) __builtin_amdgcn_raw_buffer_store_b128(xr[h], mrs, sb + 16 * KSB + h * 1024, 0, AUX_SYS);
#pragma unroll
      for (int t = 0; t < 4; ++t) __builtin_amdgcn_raw_buffer_store_b128(wr[t], mrs, lw + t * 1024, 0, AUX_SYS);
#pragma unroll
      for (int h = 0; h < 4; ++h) __builtin_amdgcn_raw_buffer_store_b128(xr[h], mrs, lx + h * 1024, 0, AUX_SYS);
      const unsigned ks = ksl < 5 ? (rank & 3) * 5 + kq : 20 + (rank & 3) * 3 + kq;
      if (push) {
        asm volatile("" ::: "memory");
#pragma unroll
        for (int i = 0; i < 4; ++i) {
          const int q = tg * 4 + (i < 2 ? i * 2 + hh : hh * 2 + i - 2);
          const __amdgpu_buffer_rsrc_t prs = make_rsrc(p.ws[q], WS_USED);
          const unsigned po = ((push >> (i >> 1)) & 1) && q != rank ? WS_OFF + (unsigned)((tt + (i >> 1) * 16) * NKS + ks) * KSB + wv * 4096 + lane * 16 : 0xF0000000u;
#pragma unroll
          for (int h = 0; h < 4; ++h) __builtin_amdgcn_raw_buffer_store_b128(i < 2 ? wr[h] : xr[h], prs, po + h * 1024, 0, AUX_SYS);
        }
        asm volatile("s_waitcnt vmcnt(16)" ::: "memory");
      } else {
        asm volatile("s_waitcnt vmcnt(0)" ::: "memory");
      }
      asm volatile("" : "+v"(ticket) :: "memory");
      call = (unsigned)__builtin_amdgcn_readfirstlane(ticket) / 32u + 1u;
      const int q = lane & 7;
      const bool need = lane < 16 && (q >> 2) == tg && (lane < 8 ? (q & 1) : ((q >> 1) & 1)) == hh;
      const int unit = tt + (lane & 8) * 2;
      auto last = [&](int ci) -> bool {
        unsigned old = 0;
        if (lane == 0) old = __hip_atomic_fetch_add(cnt + ci, 1u, __ATOMIC_RELAXED, __HIP_MEMORY_SCOPE_WORKGROUP);
        return __builtin_amdgcn_readfirstlane(old) == 3;
      };
      if (last(jn) && need) {
        if (q == rank)
          __hip_atomic_store((u64*)(myws + iflag((unit * 32 + ks) * 2)), ((u64)call << 32) | call, __ATOMIC_RELAXED, __HIP_MEMORY_SCOPE_SYSTEM);
        else
          __hip_atomic_store((unsigned*)(wsl + PF_OFF + (rank * 256 + unit * 8 + kq) * 128 + 4), call, __ATOMIC_RELAXED, __HIP_MEMORY_SCOPE_SYSTEM);
      }
      if (push) {
        asm volatile("s_waitcnt vmcnt(0)" ::: "memory");
        if (last(2) && need && q != rank && ((push >> (lane >> 3)) & 1))
          __hip_atomic_store((u64*)(wsl + iflag((unit * 32 + ks) * 2)), ((u64)call << 32) | call, __ATOMIC_RELAXED, __HIP_MEMORY_SCOPE_SYSTEM);
      }
      push = 0;
      jobw = jobw2;
    }
    asm volatile("s_waitcnt vmcnt(0)" : "+v"(ticket) :: "memory");
    call = (unsigned)__builtin_amdgcn_readfirstlane(ticket) / 32u + 1u;
  }
  if (blockIdx.x < NMOV) {
    const int mb = blockIdx.x;
    const int bi = __builtin_amdgcn_readfirstlane((mb * 9363) >> 16);
    const int lk = __builtin_amdgcn_readfirstlane(mb - 7 * bi);
    const int src = (rank + 1 + lk) & 7;
    const int wi = bi;
    const int sgp = src >> 2, si = (src >> 1) & 1, sj = src & 1;
    unsigned short* const mark = p.out + (size_t)(row_tile(sgp, si, 8) * 128) * DIM + col_tile(sgp, sj, 0) * 128 + wi;
    const unsigned l16 = lane * 16;
    const unsigned oob = 0xF0000000u;
    LDS_AS u64* rdesc = (LDS_AS u64*)(lds + LDS_CTRL + 64);
    LDS_AS unsigned* rfree = (LDS_AS unsigned*)(lds + LDS_CTRL + 160);
    LDS_AS unsigned* rland = (LDS_AS unsigned*)(lds + LDS_CTRL + 208);
#define LDS_LD(ptr) __hip_atomic_load(ptr, __ATOMIC_RELAXED, __HIP_MEMORY_SCOPE_WORKGROUP)
#define LDS_ST(ptr, val) __hip_atomic_store(ptr, val, __ATOMIC_RELAXED, __HIP_MEMORY_SCOPE_WORKGROUP)
    const u64 tstart = __builtin_amdgcn_s_memrealtime();
    auto ring_dead = [&]() -> bool {
      if (__builtin_amdgcn_s_memrealtime() - tstart <= 3 * DEADLINE_TICKS) return false;
      __hip_atomic_store(tmo, call, __ATOMIC_RELAXED, __HIP_MEMORY_SCOPE_AGENT);
      if (lane == 0) *mark = 0x7fc0;
      return true;
    };
    const int cross = sgp != gq;
    LDS_AS unsigned* mbox = (LDS_AS unsigned*)(lds + LDS_CTRL + 216);
    if (wv == 3) {
      const unsigned ps = 2 * NPUSH;
      const unsigned np = cross ? p.direct.queued[src & 3] : 0u, no = (cross ? 192u : 320u) - ps;
      const unsigned otab = ((const unsigned*)p.order.t[cross])[lane < 40 ? lane : 0];
      const unsigned ctr = QC_OFF + (unsigned)((call & 1) * 32 + lk + (lane & 3) * 8) * 128;
      unsigned tn = 1;
      asm volatile("buffer_atomic_add %0, %1, %2, 0 offen sc0\n\ts_waitcnt vmcnt(0)" : "+v"(tn) : "v"(lane < 3 ? ctr : oob), "s"(mrs) : "memory");
      unsigned cp = __builtin_amdgcn_readlane(tn, 0), co = __builtin_amdgcn_readlane(tn, 1), cq = __builtin_amdgcn_readlane(tn, 2);
      unsigned nq = 0;
      bool bad = false, refill = true;
      while (cq < 512u || co < no || cp < np) {
        const unsigned ob_ = (__builtin_amdgcn_readlane(otab, ((co + ps) >> 3) & 63) >> (8 * (((co + ps) >> 1) & 3))) & 255u;
        const unsigned fa = lane == 0 ? (cp < np ? oflag(src, 512 + cp) : oob)
                          : lane == 1 ? (co < no ? PF_OFF + (unsigned)(src * 256 + ob_) * 128 : oob)
                          : lane == 2 ? (cq < 512u ? oflag(src, cq) : oob) : oob;
        tn = 1;
        asm volatile("buffer_atomic_add %0, %1, %2, 0 offen sc0" : "+v"(tn) : "v"(lane == 2 && refill ? ctr : oob), "s"(mrs) : "memory");
        u64 t0 = __builtin_amdgcn_s_memrealtime();
        unsigned rd, fx;
        for (unsigned spin = 1;; ++spin) {
          const u32x2 f = __builtin_amdgcn_raw_buffer_load_b64(mrs, fa, 0, AUX_SYS);
          const unsigned ex = bad ? 4u : (unsigned)__ballot(f.y >= call) & 7u;
          fx = f.x;
          rd = 0;
          if (ex) {
            for (int w = cp < np ? 8 : 64; w > 0 && !rd; --w) {
              const u64 mb_ = LDS_LD((LDS_AS u64*)mbox);
              rd = __builtin_amdgcn_readfirstlane(ex & (((unsigned)mb_ ? 0u : 1u) | ((mb_ >> 32) ? 0u : 6u)));
            }
            if (rd) break;
          }
          if ((spin & 7) == 0 && !ex && __builtin_amdgcn_readfirstlane(timed_out(tmo, call, t0))) bad = true;
          if ((spin & 7) == 0 && ring_dead()) return;
          asm volatile("" ::: "memory");
        }
        asm volatile("s_waitcnt vmcnt(0)" : "+v"(tn) :: "memory");
        if (refill) nq = __builtin_amdgcn_readlane(tn, 2);
        refill = false;
        if (bad) {
          co = no; cp = np;
          fx = (cq >> 2) << 14;
          if (cq >= 512u) break;
        }
        if (rd & 1) {
          const unsigned tile = (__builtin_amdgcn_readlane(fx, 0) >> 14) & 127u, sw = cp & 3u;
          LDS_ST(mbox, (5u << 28) | ((PB_OFF / 8192 + tile * 4 + sw) << 12) | (PS_OFF / 8192 + ((tile & 0x6eu) + si * 16 + sj) * 4 + sw));
        } else if (rd & 2) {
          const unsigned u = ob_ >> 3, kb = ob_ & 7u;
          LDS_ST(mbox + 1, (5u << 28) | ((SB_OFF / 8192 + (((kb + 5 * cross) * 2 + (u < 16 ? rj : ri)) * 32 + u) * 2 + (co & 1u)) << 12)
                           | ((u * 32 + (cross ? 20 + (src & 3) * 3 : (src & 3) * 5) + kb) * 2 + (co & 1u)));
        } else {
          const unsigned flo = __builtin_amdgcn_readlane(fx, 2);
          LDS_ST(mbox + 1, (1u << 28) | (min(flo & 0x3fffu, 4096u) << 12) | (((flo >> 14) & 127u) * 4 + (cq & 3u)) | (bad ? 1u << 29 : 0u));
          cq = nq; refill = true;
        }
        if (rd & 3) {
          tn = 1;
          asm volatile("buffer_atomic_add %0, %1, %2, 0 offen sc0\n\ts_waitcnt vmcnt(0)" : "+v"(tn) : "v"(lane == (int)(~rd & 1) ? ctr : oob), "s"(mrs) : "memory");
          if (rd & 1) cp = __builtin_amdgcn_readlane(tn, 0);
          else co = __builtin_amdgcn_readlane(tn, 1);
        }
      }
      while (LDS_LD((LDS_AS u64*)mbox) != 0) {
        if (ring_dead()) return;
        __builtin_amdgcn_s_sleep(1);
      }
      LDS_ST(mbox + 1, 0x80000000u);
      if (bad && lane == 0) *mark = 0x7fc0;
      return;
    }
    if (wv != 0) {
      const unsigned ob = (unsigned)(r16 * DIM + 8 * g4) * 2;
      int slot = wv - 1;
      for (unsigned sq = wv - 1;; sq += 2) {
        u64 d;
        for (;;) {
          d = LDS_LD(rdesc + slot);
          if ((unsigned)(d >> 48) == sq + 1) break;
          if (ring_dead()) return;
          __builtin_amdgcn_s_sleep(4);
        }
        const unsigned dhi = __builtin_amdgcn_readfirstlane((unsigned)(d >> 32)), dend = __builtin_amdgcn_readfirstlane((unsigned)d);
        const unsigned sg = dhi & 0xfffu;
        if (sg == 0xfffu) break;
        const unsigned nv = dend >> 16, H = nv >> 5;
        while (LDS_LD(rland) < (dend & 0xffffu)) {
          if (ring_dead()) return;
          __builtin_amdgcn_s_sleep(1);
        }
        asm volatile("" ::: "memory");
        LDS_AS char* seg = (LDS_AS char*)(lds + 16 + slot * SEG);
        if (dhi & 0x4000u) {
          u32x4 v[8];
#pragma unroll
          for (int i = 0; i < 8; ++i) v[i] = *(LDS_AS u32x4*)(seg + i * 1024 + l16);
          LDS_ST(rfree + slot, sq + 1);
#pragma unroll
          for (int i = 0; i < 8; ++i) __builtin_amdgcn_raw_buffer_store_b128(v[i], mrs, sg * 8192 + i * 1024 + l16, 0, OUT_AUX);
          asm volatile("s_waitcnt vmcnt(0)" ::: "memory");
          if (lane == 0) __hip_atomic_store((unsigned*)(myws + iflag(sg)), call, __ATOMIC_RELAXED, __HIP_MEMORY_SCOPE_AGENT);
          slot += 2;
          if (slot >= NSLOT) slot -= NSLOT;
          continue;
        }
        auto unhide = [&](int i0, int n) {
          u32x4 v[5];
#pragma unroll
          for (int i = 0; i < n; ++i) v[i] = *(LDS_AS u32x4*)(seg + (i0 + i) * 1024 + l16);
#pragma unroll
          for (int i = 0; i < n; ++i) {
            unsigned e = ((v[i][0] >> 15) & 0x00010001u) | ((v[i][1] >> 14) & 0x00020002u) | ((v[i][2] >> 13) & 0x00040004u) | ((v[i][3] >> 12) & 0x00080008u);
            e <<= (lane & 3) * 4;
            e |= __builtin_amdgcn_update_dpp(0u, e, 0xb1, 0xf, 0xf, true);
            e |= __builtin_amdgcn_update_dpp(0u, e, 0x4e, 0xf, 0xf, true);
            *(LDS_AS unsigned*)(seg + SEG_TBL + (i0 + i) * 64 + (lane >> 2) * 4) = e;
          }
        };
        unhide(0, 5);
        if (H > 80) unhide(5, 3);
        const unsigned xo = ((2 * nv + 3) & ~3u) - 4 * H;
        u32x2 mk;
        mk.x = *(LDS_AS unsigned*)(seg + (2u * lane < H ? SEG_TBL : xo) + lane * 8);
        mk.y = *(LDS_AS unsigned*)(seg + (2u * lane + 1 < H ? SEG_TBL : xo) + lane * 8 + 4);
        const unsigned c0 = __builtin_popcount(mk.x), cn = c0 + __builtin_popcount(mk.y);
        const int b0 = 2 * (int)(wave_scan(cn) - cn) - 2, b1 = b0 + 2 * (int)c0;
        u32x4 ov[8], oh[8];
#pragma unroll
        for (int c = 0; c < 8; ++c)
#pragma unroll
          for (int dd = 0; dd < 4; ++dd) {
            const int dq = (c & 3) * 4 + dd;
            const unsigned E = c < 4 ? mk.x : mk.y;
            const unsigned pp = __builtin_popcount(E & (((2u << dq) - 1u) | (((1u << dq) - 1u) << 16)));
            const volatile LDS_AS unsigned short* q = (const volatile LDS_AS unsigned short*)(seg + (c < 4 ? b0 : b1) + 2 * (int)pp);
            ov[c][dd] = q[0];
            oh[c][dd] = q[1];
          }
        LDS_ST(rfree + slot, sq + 1);
        const int tile = sg >> 2, sw = sg & 3;
        const unsigned oo = ob + (unsigned)((row_tile(sgp, si, 8 + (tile >> 4)) * 128 + (sw >> 1) * 64) * DIM + col_tile(sgp, sj, tile & 15) * 128 + 64 * (sw & 1)) * 2;
        const unsigned poison = (dhi & 0x8000u) ? 0x7fc07fc0u : 0u;
#pragma unroll
        for (int c = 0; c < 8; ++c) {
#pragma unroll
          for (int dd = 0; dd < 4; ++dd) {
            const unsigned m = ((c < 4 ? mk.x : mk.y) >> ((c & 3) * 4 + dd)) & 0x00010001u;
            const unsigned t = (ov[c][dd] | (oh[c][dd] << 16)) & 0x7fff7fffu;
            const u16x2 r = __builtin_bit_cast(u16x2, t) * __builtin_bit_cast(u16x2, m);
            ov[c][dd] = __builtin_bit_cast(unsigned, r);
          }
          ov[c][0] |= poison;
          __builtin_amdgcn_raw_buffer_store_b128(ov[c], ors, oo + (c & 1) * 64, (c >> 1) * (16 * DIM * 2), OUT_AUX);
          __builtin_amdgcn_sched_barrier(0);
          asm volatile("s_nop 1");
          __builtin_amdgcn_sched_barrier(0);
        }
        slot += 2;
        if (slot >= NSLOT) slot -= NSLOT;
      }
      return;
    }
    __amdgpu_buffer_rsrc_t srs = make_rsrc(p.ws[src], WS_USED);
    bool bad = false;
    if (mb == 0 && tid < 32)
      __hip_atomic_store((unsigned*)(myws + QC_OFF + (((call + 1) & 1) * 32 + tid) * 128), 0u, __ATOMIC_RELAXED, __HIP_MEMORY_SCOPE_AGENT);
    {
      const unsigned ldsbase = (unsigned)(size_t)(LDS_AS char*)lds + 16;
      int k = 0, land = 0;
      unsigned sq = 0;
      int slot = 0;
      auto drain = [&]() {
        asm volatile("s_waitcnt vmcnt(0)" ::: "memory");
        land = k;
        LDS_ST(rland, (unsigned)k);
      };
      auto next_slot = [&]() {
        if (sq >= (unsigned)NSLOT && LDS_LD(rfree + slot) != sq - NSLOT + 1) {
          drain();
          while (LDS_LD(rfree + slot) != sq - NSLOT + 1) {
            if (ring_dead()) { bad = true; break; }
            __builtin_amdgcn_s_sleep(1);
          }
        }
      };
      for (;;) {
        u64 m = LDS_LD((LDS_AS u64*)mbox);
        if (m == 0) {
          asm volatile("s_waitcnt vmcnt(2)" ::: "memory");
          land = max(land, k - 2);
          LDS_ST(rland, (unsigned)land);
          m = LDS_LD((LDS_AS u64*)mbox);
          if (m == 0) {
            drain();
            for (;;) {
              m = LDS_LD((LDS_AS u64*)mbox);
              if (m != 0) break;
              if (ring_dead()) { m = 0x80000000u; break; }
              __builtin_amdgcn_s_sleep(1);
            }
          }
        }
        const unsigned mu = __builtin_amdgcn_readfirstlane((unsigned)m);
        const unsigned flo = mu ? mu : __builtin_amdgcn_readfirstlane((unsigned)(m >> 32));
        LDS_ST(mbox + (mu ? 0 : 1), 0u);
        if (flo >> 31) break;
        next_slot();
        const unsigned copy = (flo >> 30) & 1u, id = flo & 0xfffu, hi = (flo >> 12) & 0x1fffu;
        const unsigned nb = copy ? 8192u : seg_bytes(hi);
        const unsigned np = (nb + 1023) >> 10;
        LDS_ST(rdesc + slot, ((u64)(((sq + 1) << 16) | ((bad || ((flo >> 29) & 1u)) ? 0x8000u : 0u) | (copy << 14) | id) << 32) | (hi << 16) | (unsigned)(k + np));
        const unsigned sb = copy ? hi * 8192u : OB_OFF + id * SEG;
        unsigned vo = sb + l16;
        const unsigned vend = sb + nb;
        unsigned la = ldsbase + slot * SEG;
        for (unsigned i = 0; i < np; ++i) {
          unsigned keep;
          asm volatile("s_waitcnt vmcnt(%0)" :: "i"(ODEPTH - 1) : "memory");
          land = max(land, k - (ODEPTH - 1));
          LDS_ST(rland, (unsigned)land);
          asm volatile("s_mov_b32 %0, m0\n\ts_mov_b32 m0, %3\n\ts_nop 0\n\tbuffer_load_dwordx4 %1, %2, 0 offen sc0 sc1 lds\n\ts_mov_b32 m0, %0"
                       : "=&s"(keep) : "v"(vo < vend ? vo : oob), "s"(srs), "s"(la) : "memory");
          vo += 1024; la += 1024; ++k;
        }
        ++sq;
        if (++slot == NSLOT) slot = 0;
      }
      drain();
      for (int e = 0; e < 2; ++e) {
        next_slot();
        LDS_ST(rdesc + slot, (u64)(((sq + 1) << 16) | 0xfffu) << 32);
        ++sq;
        if (++slot == NSLOT) slot = 0;
      }
    }
    if (__hip_atomic_load(tmo, __ATOMIC_RELAXED, __HIP_MEMORY_SCOPE_AGENT) == call && lane == 0)
      *mark = 0x7fc0;
    return;
  }

  const int gb = blockIdx.x - NMOV;
  const int wm = wv >> 1, hf = wv & 1;
  const unsigned wgt = ((const unsigned*)p.tiles.wg)[gb >> 1] >> (16 * (gb & 1));

#pragma unroll 1
  for (int jn = 0; jn < (gb < 256 - NG ? 2 : 1); ++jn) {
  const int pass = (wgt >> (8 * jn + 7)) & 1, kt = (wgt >> (8 * jn)) & 127, st = kt;
  LDS_AS u64* ctl = (LDS_AS u64*)(lds + LDS_CTRL + 32) + jn;
  f32x4 acc[8][8];
  u32x4 pt[8];
  unsigned early;
  int lane = tid & 63;
  asm volatile("" : "+v"(lane));
  const int rt = pass ? 8 + (kt >> 4) : st >> 4, ct = (pass ? kt : st) & 15;
  {
    const unsigned bx = WS_OFF + (unsigned)(16 + rt) * (NKS * KSB) + lane * 16, bw = WS_OFF + (unsigned)ct * (NKS * KSB) + lane * 16;
    const unsigned vx0 = bx + wm * 4096, vx1 = bx + (wm ^ 1) * 4096;
    const unsigned vw0 = bw + hf * 4096, vw1 = bw + (hf ^ 1) * 4096;
#pragma unroll
    for (int a = 0; a < 8; ++a)
#pragma unroll
      for (int b = 0; b < 8; ++b) acc[a][b] = f32x4{0.f, 0.f, 0.f, 0.f};
    u32x4 fw[3][8], fx[3][8];
#define LDW(dst, vo, so, imm) asm volatile("buffer_load_dwordx4 %0, %1, %2, %3 offen offset:%4 sc1" : "=v"(dst) : "v"(vo), "s"(mrs), "s"(so), "i"(imm))
#define LDX(dst, vo, so, imm) asm volatile("buffer_load_dwordx4 %0, %1, %2, %3 offen offset:%4 sc1" : "=v"(dst) : "v"(vo), "s"(mrs), "s"(so), "i"(imm))
#define MF(T, M, w_, x_) asm volatile("v_mfma_f32_16x16x32_bf16 %0, %1, %2, %0" : "+a"(acc[T][M]) : "v"(w_[T]), "v"(x_[M]))
#define WAITF(w_, x_, n) asm volatile("s_waitcnt vmcnt(" #n ")" : "+v"(w_[0]), "+v"(w_[1]), "+v"(w_[2]), "+v"(w_[3]), "+v"(w_[4]), "+v"(w_[5]), "+v"(w_[6]), "+v"(w_[7]), \
                   "+v"(x_[0]), "+v"(x_[1]), "+v"(x_[2]), "+v"(x_[3]), "+v"(x_[4]), "+v"(x_[5]), "+v"(x_[6]), "+v"(x_[7]))
    auto load1 = [&](int i, unsigned sb, u32x4* w_, u32x4* x_) {
      const int q = i >> 2, j = i & 3;
      if (q == 0) LDW(w_[j], vw0, sb, j * 1024);
      else if (q == 1) LDX(x_[j], vx0, sb, j * 1024);
      else if (q == 2) LDW(w_[4 + j], vw1, sb, j * 1024);
      else LDX(x_[4 + j], vx1, sb, j * 1024);
    };
    auto half = [&](u32x4* w_, u32x4* x_, bool doload, unsigned sb, u32x4* wn, u32x4* xn) {
#pragma unroll
      for (int i = 0; i < 16; ++i) {
        const int qa = i >> 3, qb = (i >> 2) & 1, m = i & 3;
#pragma unroll
        for (int t = 0; t < 4; ++t) MF((qa ^ qb) * 4 + t, qa * 4 + m, w_, x_);
        if (doload) load1(i, sb, wn, xn);
      }
    };
    const int lb = (lane >> 1) & 7, ls = (wv - lb - 2 * (lane & 1)) & 3;
    const unsigned ksv = (unsigned)(lb < 5 ? ls * 5 + lb : 15 + ls * 3 + lb) * 2 + (lane & 1);
    const unsigned foff = iflag(lane < 32 ? (unsigned)((lane & 16) ? ct : 16 + rt) * 64 + ksv
                                          : PS_OFF / 8192 + (unsigned)(kt * 4 + wv));
    unsigned todo = 0xffffu, known = 0, fv = 0, warm = 0xffffffffu;
    auto pick = [&]() -> unsigned {
      unsigned rd = (unsigned)__ballot(fv >= call);
      unsigned m = ((rd & (rd >> 16)) | known) & todo;
      if (m == 0) {
        u64 t0 = __builtin_amdgcn_s_memrealtime();
        for (unsigned spin = 1;; ++spin) {
          const unsigned v = __hip_atomic_load((unsigned*)(myws + foff), __ATOMIC_RELAXED, __HIP_MEMORY_SCOPE_SYSTEM);
          rd = (unsigned)__ballot(v >= call);
          m = rd & (rd >> 16) & todo;
          if (m) break;
          if ((spin & 15) == 0 && __builtin_amdgcn_readfirstlane(timed_out(tmo, call, t0))) { m = todo; cnt[4] = 1; break; }
          unsigned cand = rd & warm & (todo * 0x10001u);
          if (cand) {
            warm &= ~cand;
            unsigned a = 0;
            const unsigned vo = (tid & 63) * 64;
            for (; cand; cand &= cand - 1) {
              const unsigned i = __builtin_ctz(cand);
              const unsigned so = WS_OFF + ((unsigned)(i < 16 ? 16 + rt : ct) * 64 + (unsigned)__builtin_amdgcn_readlane(ksv, i)) * 8192;
#pragma unroll
              for (int q = 0; q < 2; ++q)
                asm volatile("buffer_load_dword %0, %1, %2, %3 offen sc1" : "+v"(a) : "v"(vo), "s"(mrs), "s"(so + q * 4096));
            }
            asm volatile("s_waitcnt vmcnt(0)" : "+v"(a));
          } else {
            __builtin_amdgcn_s_sleep(4);
          }
        }
      }
      const unsigned j = __builtin_ctz(m);
      todo &= ~(1u << j);
      known = m & todo;
      return (unsigned)__builtin_amdgcn_readlane(ksv, j) * 8192;
    };
#define LDF() asm volatile("buffer_load_dword %0, %1, %2, 0 offen sc0 sc1" : "=v"(fv) : "v"(foff | (known ? 0xF0000000u : 0u)), "s"(mrs))
    {
      const unsigned s0 = pick();
      asm volatile("s_waitcnt vmcnt(0)\n\ts_nop 4" ::: "memory");
#pragma unroll
      for (int i = 0; i < 16; ++i) load1(i, s0, fw[0], fx[0]);
      const unsigned s1 = pick();
      asm volatile("s_nop 4");
      LDF();
#pragma unroll
      for (int i = 0; i < 16; ++i) load1(i, s1, fw[1], fx[1]);
    }
#define KSTEP(u) do { \
        WAITF(fw[u], fx[u], 16); \
        asm volatile("" : "+v"(fv)); \
        const unsigned sb = pick(); \
        asm volatile("s_nop 4"); \
        LDF(); \
        half(fw[u], fx[u], true, sb, fw[((u) + 2) % 3], fx[((u) + 2) % 3]); } while (0)
#pragma unroll 1
    for (int s = 0; s < 4; ++s) { KSTEP(0); KSTEP(1); KSTEP(2); }
    KSTEP(0);
    KSTEP(1);
    WAITF(fw[2], fx[2], 16);
    half(fw[2], fx[2], false, 0, fw[0], fx[0]);
    WAITF(fw[0], fx[0], 0);
    half(fw[0], fx[0], false, 0, fw[0], fx[0]);
    asm volatile("" : "+v"(fv));
    early = (unsigned)(__ballot(fv >= call) >> 32) & (unsigned)pass;
    asm volatile("s_nop 15\n\ts_nop 15" ::: "memory");
    __builtin_amdgcn_sched_barrier(0);
  }
  asm volatile("" : "+v"(lane));
  f32x4 r[4][4];
  const int prk = (gq ^ 1) * 4 + (rt & 1) * 2 + (ct & 1), ptl = (kt & 0x6e) + ri * 16 + rj, ftl = pass ? kt : ptl;
  const bool direct = (p.direct.mask[ftl >> 5] >> (ftl & 31)) & 1u;
  const __amdgpu_buffer_rsrc_t prs = make_rsrc(direct ? p.ws[prk] : myws, WS_USED);
  const unsigned po = (direct ? PB_OFF + (unsigned)(ptl * 4 + wv) * 8192 : PS_OFF + (unsigned)(kt * 4 + wv) * 8192) + lane * 16;
  char* wq = p.ws[pass ? lane & 7 : prk];
  {
    if (early) {
#pragma unroll
      for (int c = 0; c < 8; ++c) pt[c] = __builtin_amdgcn_raw_buffer_load_b128(prs, po + c * 1024, 0, AUX_SYS);
    }
    unsigned a0 = (unsigned)(size_t)(LDS_AS char*)lds + wv * 32768 + lane * 16;
    asm volatile("" : "+v"(a0));
    const unsigned a1 = a0 ^ 32768u, a3 = a0 ^ (3u * 32768u);
#pragma unroll
    for (int m = 7; m >= 0; --m)
#pragma unroll
      for (int t = 0; t < 4; ++t) *(LDS_AS f32x4*)(size_t)(a0 + (m * 4 + t) * 1024) = acc[4 + t][m];
    f32x4 hi[16], u[16], v[16];
#pragma unroll
    for (int i = 0; i < 16; ++i) { hi[i] = acc[i & 3][4 + (i >> 2)]; r[i & 3][i >> 2] = acc[i & 3][i >> 2]; asm volatile("" : "+v"(hi[i]), "+v"(r[i & 3][i >> 2])); }
#define O16(x) "=&v"(x[0]), "=&v"(x[1]), "=&v"(x[2]), "=&v"(x[3]), "=&v"(x[4]), "=&v"(x[5]), "=&v"(x[6]), "=&v"(x[7]), \
               "=&v"(x[8]), "=&v"(x[9]), "=&v"(x[10]), "=&v"(x[11]), "=&v"(x[12]), "=&v"(x[13]), "=&v"(x[14]), "=&v"(x[15])
#define READ16(o) "ds_read_b128 %0, %16 offset:" #o "\n\tds_read_b128 %1, %16 offset:" #o "+1024\n\tds_read_b128 %2, %16 offset:" #o "+2048\n\t" \
      "ds_read_b128 %3, %16 offset:" #o "+3072\n\tds_read_b128 %4, %16 offset:" #o "+4096\n\tds_read_b128 %5, %16 offset:" #o "+5120\n\t" \
      "ds_read_b128 %6, %16 offset:" #o "+6144\n\tds_read_b128 %7, %16 offset:" #o "+7168\n\tds_read_b128 %8, %16 offset:" #o "+8192\n\t" \
      "ds_read_b128 %9, %16 offset:" #o "+9216\n\tds_read_b128 %10, %16 offset:" #o "+10240\n\tds_read_b128 %11, %16 offset:" #o "+11264\n\t" \
      "ds_read_b128 %12, %16 offset:" #o "+12288\n\tds_read_b128 %13, %16 offset:" #o "+13312\n\tds_read_b128 %14, %16 offset:" #o "+14336\n\t" \
      "ds_read_b128 %15, %16 offset:" #o "+15360\n\ts_waitcnt lgkmcnt(0)"
    asm volatile("s_waitcnt lgkmcnt(0)\n\ts_barrier\n\t" READ16(16384) : O16(u) : "v"(a1) : "memory");
#pragma unroll
    for (int i = 0; i < 16; ++i) *(LDS_AS f32x4*)(size_t)(a1 + 16384 + i * 1024) = u[i] + hi[i];
    asm volatile(READ16(0) "\n\ts_barrier" : O16(u) : "v"(a1) : "memory");
    asm volatile(READ16(16384) : O16(v) : "v"(a3) : "memory");
#pragma unroll
    for (int i = 0; i < 16; ++i) r[i & 3][i >> 2] = (r[i & 3][i >> 2] + u[i]) + v[i];
    if (cnt[4]) r[0][0][0] = __uint_as_float(0x7fc00000u);
  }
  if (pass == 0) {
    unsigned slot = 1;
    asm volatile("buffer_atomic_add %0, %1, %2, 0 offen sc0" : "+v"(slot)
                 : "v"(tid == 0 && !direct ? QC_OFF + (unsigned)((call & 1) * 32 + 24 + (prk & 3)) * 128 : 0xF0000000u), "s"(mrs) : "memory");
#pragma unroll
    for (int c = 0; c < 8; ++c) {
      const f32x4 a = r[c >> 1][(c & 1) * 2], b = r[c >> 1][(c & 1) * 2 + 1];
      const u32x4 pk = {pack_bf16(a[0], a[1]), pack_bf16(a[2], a[3]), pack_bf16(b[0], b[1]), pack_bf16(b[2], b[3])};
      __builtin_amdgcn_raw_buffer_store_b128(pk, mrs, PB_OFF + (unsigned)(st * 4 + wv) * 8192 + c * 1024 + lane * 16, 0, AUX_SYS);
    }
    asm volatile("s_waitcnt vmcnt(0)" : "+v"(slot), "+v"(wq) :: "memory");
    if (direct && lane == 0)
      __hip_atomic_store((unsigned*)(wq + iflag(PS_OFF / 8192 + ptl * 4 + wv)), call, __ATOMIC_RELAXED, __HIP_MEMORY_SCOPE_SYSTEM);
    __syncthreads();
    slot = __builtin_amdgcn_readfirstlane(slot);
    if (!direct && tid < 4 && slot < 32u)
      __hip_atomic_store((u64*)(wq + oflag(rank, 512 + slot * 4 + tid)), ((u64)call << 32) | ((unsigned)st << 14), __ATOMIC_RELAXED, __HIP_MEMORY_SCOPE_SYSTEM);
  } else {
    if (!early) {
      unsigned* pf = (unsigned*)(myws + iflag(PS_OFF / 8192 + kt * 4 + wv));
      u64 t0 = __builtin_amdgcn_s_memrealtime();
      for (unsigned spin = 1; __hip_atomic_load(pf, __ATOMIC_RELAXED, __HIP_MEMORY_SCOPE_SYSTEM) < call; ++spin) {
        if ((spin & 15) == 0 && __builtin_amdgcn_readfirstlane(timed_out(tmo, call, t0))) { r[0][0][0] = __uint_as_float(0x7fc00000u); break; }
        __builtin_amdgcn_s_sleep(1);
      }
      asm volatile("" ::: "memory");
#pragma unroll
      for (int c = 0; c < 8; ++c) pt[c] = __builtin_amdgcn_raw_buffer_load_b128(prs, po + c * 1024, 0, AUX_SYS);
    }
#pragma unroll
    for (int i = 0; i < 16; ++i) {
      const unsigned a = pt[i >> 1][(i & 1) * 2], b = pt[i >> 1][(i & 1) * 2 + 1];
      r[i >> 2][i & 3] += __builtin_bit_cast(f32x4, u32x4{a << 16, a & 0xffff0000u, b << 16, b & 0xffff0000u});
      asm volatile("" : "+v"(r[i >> 2][i & 3]));
    }
    LDS_AS char* seg = (LDS_AS char*)(lds + (wv ^ 1) * 32768);
    u32x4 pk[8];
    unsigned E[2] = {0u, 0u};
#pragma unroll
    for (int c = 0; c < 8; ++c) {
      unsigned t[4], s_[4];
#pragma unroll
      for (int dd = 0; dd < 4; ++dd)
        t[dd] = pack_bf16(r[(dd & 1) * 2][c >> 1][(c & 1) * 2 + (dd >> 1)], r[(dd & 1) * 2 + 1][c >> 1][(c & 1) * 2 + (dd >> 1)]);
      asm("v_pk_sub_u16 %4, %0, %8 clamp\n\tv_pk_sub_u16 %5, %1, %8 clamp\n\tv_pk_sub_u16 %6, %2, %8 clamp\n\tv_pk_sub_u16 %7, %3, %8 clamp\n\t"
          "v_pk_max_i16 %0, %0, 0\n\tv_pk_max_i16 %1, %1, 0\n\tv_pk_max_i16 %2, %2, 0\n\tv_pk_max_i16 %3, %3, 0\n\t"
          "v_pk_mad_i16 %0, %4, %9, %0 clamp\n\tv_pk_mad_i16 %1, %5, %9, %1 clamp\n\tv_pk_mad_i16 %2, %6, %9, %2 clamp\n\tv_pk_mad_i16 %3, %7, %9, %3 clamp\n\t"
          "v_pk_min_u16 %4, %0, 1 op_sel_hi:[1,0]\n\tv_pk_min_u16 %5, %1, 1 op_sel_hi:[1,0]\n\tv_pk_min_u16 %6, %2, 1 op_sel_hi:[1,0]\n\tv_pk_min_u16 %7, %3, 1 op_sel_hi:[1,0]"
          : "+v"(t[0]), "+v"(t[1]), "+v"(t[2]), "+v"(t[3]), "=&v"(s_[0]), "=&v"(s_[1]), "=&v"(s_[2]), "=&v"(s_[3]) : "s"(0xff80ff80u), "s"(0x7fff7fffu));
      pk[c] = u32x4{t[0], t[1], t[2], t[3]};
      E[c >> 2] |= ((s_[0] | (s_[1] << 1)) | ((s_[2] | (s_[3] << 1)) << 2)) << ((c & 3) * 4);
    }
    const unsigned cn = __builtin_popcount(E[0]) + __builtin_popcount(E[1]);
    const unsigned inc = wave_scan(cn);
    const unsigned total = __builtin_amdgcn_readlane(inc, 63);
    const unsigned H = total >> 5;
    {
      const unsigned xo = ((2 * total + 3) & ~3u) - 4 * H;
      *(LDS_AS u32x2*)(seg + SEG_TBL + 512 + lane * 8) = u32x2{2u * lane < H ? E[0] : 0u, 2u * lane + 1 < H ? E[1] : 0u};
      if (lane == 63) *(LDS_AS unsigned short*)(seg + 2 * total) = 0;
      if (2u * lane >= H) *(LDS_AS unsigned*)(seg + xo + lane * 8) = E[0];
      if (2u * lane + 1 >= H) *(LDS_AS unsigned*)(seg + xo + lane * 8 + 4) = E[1];
    }
    unsigned draw = 0;
    if (lane == 0) draw = __hip_atomic_fetch_add(cnt + 17 + jn, 1u, __ATOMIC_RELAXED, __HIP_MEMORY_SCOPE_WORKGROUP);
    unsigned wa = (unsigned)(size_t)seg + 2 * (inc - cn);
    unsigned wb = wa + 2 * __builtin_popcount(E[0]);
#pragma unroll
    for (int c = 0; c < 4; ++c)
#pragma unroll
      for (int dd = 0; dd < 4; dd += 2) {
        u64 m[8];
#define CW(i, a, ins, v) "s_mov_b64 exec, %" #i "\n\t" ins " %" #a ", %" #v "\n\tv_add_u32 %" #a ", 2, %" #a "\n\t"
        asm volatile("v_cmp_ne_u16 %2, 0, %10\n\tv_cmp_ne_u16 %3, 0, %12\n\tv_cmp_lt_u32 %4, %14, %10\n\tv_cmp_lt_u32 %5, %14, %12\n\t"
                     "v_cmp_ne_u16 %6, 0, %11\n\tv_cmp_ne_u16 %7, 0, %13\n\tv_cmp_lt_u32 %8, %14, %11\n\tv_cmp_lt_u32 %9, %14, %13\n\t"
                     CW(2, 0, "ds_write_b16", 10) CW(3, 1, "ds_write_b16", 12) CW(4, 0, "ds_write_b16_d16_hi", 10) CW(5, 1, "ds_write_b16_d16_hi", 12)
                     CW(6, 0, "ds_write_b16", 11) CW(7, 1, "ds_write_b16", 13) CW(8, 0, "ds_write_b16_d16_hi", 11) CW(9, 1, "ds_write_b16_d16_hi", 13)
                     "s_mov_b64 exec, -1"
                     : "+v"(wa), "+v"(wb), "=&s"(m[0]), "=&s"(m[1]), "=&s"(m[2]), "=&s"(m[3]), "=&s"(m[4]), "=&s"(m[5]), "=&s"(m[6]), "=&s"(m[7])
                     : "v"(pk[c][dd]), "v"(pk[c][dd + 1]), "v"(pk[4 + c][dd]), "v"(pk[4 + c][dd + 1]), "s"(0xffffu) : "memory");
      }
    const unsigned nb = seg_bytes(total);
    const unsigned so = OB_OFF + (unsigned)(kt * 4 + wv) * SEG + lane * 16;
    unsigned slot = 1;
    asm volatile("buffer_atomic_add %0, %1, %2, 0 offen sc0" : "+v"(slot)
                 : "v"(draw == 3 ? QC_OFF + (unsigned)((call & 1) * 32 + 28) * 128 : 0xF0000000u), "s"(mrs) : "memory");
    {
      u32x4 v[9];
      u32x2 e[4];
      asm volatile("ds_read2_b32 %9, %14 offset0:0 offset1:16\n\tds_read_b128 %0, %13\n\tds_read_b128 %1, %13 offset:1024\n\t"
                   "ds_read2_b32 %10, %14 offset0:32 offset1:48\n\tds_read_b128 %2, %13 offset:2048\n\tds_read_b128 %3, %13 offset:3072\n\t"
                   "ds_read2_b32 %11, %14 offset0:64 offset1:80\n\tds_read_b128 %4, %13 offset:4096\n\tds_read_b128 %5, %13 offset:5120\n\t"
                   "ds_read2_b32 %12, %14 offset0:96 offset1:112\n\tds_read_b128 %6, %13 offset:6144\n\tds_read_b128 %7, %13 offset:7168\n\t"
                   "ds_read_b128 %8, %13 offset:8192\n\ts_waitcnt lgkmcnt(0)"
                   : "=&v"(v[0]), "=&v"(v[1]), "=&v"(v[2]), "=&v"(v[3]), "=&v"(v[4]), "=&v"(v[5]), "=&v"(v[6]), "=&v"(v[7]), "=&v"(v[8]),
                     "=&v"(e[0]), "=&v"(e[1]), "=&v"(e[2]), "=&v"(e[3])
                   : "v"((unsigned)(size_t)seg + lane * 16), "v"((unsigned)(size_t)seg + SEG_TBL + 512 + (lane >> 2) * 4) : "memory");
#pragma unroll
      for (int i = 0; i < 9; ++i) {
        if (i < 8) {
          const unsigned x = e[i >> 1][i & 1] >> ((lane & 3) * 4);
#pragma unroll
          for (int d = 0; d < 4; ++d) v[i][d] |= (x << (15 - d)) & 0x80008000u;
        }
        __builtin_amdgcn_raw_buffer_store_b128(v[i], mrs, (unsigned)(i * 1024 + lane * 16) < nb ? so + i * 1024 : 0xF0000000u, 0, AUX_SYS);
      }
    }
    asm volatile("s_waitcnt vmcnt(0)" : "+v"(slot), "+v"(wq) :: "memory");
    if (draw == 3) cnt[16] = slot;
    asm volatile("" ::: "memory");
    u64 old = 0;
    const u64 mine = (1ull << 56) | ((u64)total << (14 * wv));
    if (lane == 0) old = __hip_atomic_fetch_add(ctl, mine, __ATOMIC_RELAXED, __HIP_MEMORY_SCOPE_WORKGROUP);
    slot = *(volatile LDS_AS unsigned*)(cnt + 16);
    old = (((u64)(unsigned)__builtin_amdgcn_readfirstlane((unsigned)(old >> 32)) << 32) | (unsigned)__builtin_amdgcn_readfirstlane((unsigned)old)) + mine;
    if ((old >> 56) == 4 && lane < 32 && (lane & 7) != rank && slot < 128u) {
      const u64 f = ((u64)call << 32) | ((unsigned)kt << 14) | (unsigned)((old >> (14 * (lane >> 3))) & 0x3fffu);
      __hip_atomic_store((u64*)(wq + oflag(rank, slot * 4 + (lane >> 3))), f, __ATOMIC_RELAXED, __HIP_MEMORY_SCOPE_SYSTEM);
    }
    const unsigned oo = (unsigned)((row_tile(gq, ri, 8 + (kt >> 4)) * 128 + wm * 64 + (lane & 15)) * DIM + col_tile(gq, rj, kt & 15) * 128 + 64 * hf + 8 * (lane >> 4)) * 2;
#pragma unroll
    for (int c = 0; c < 8; ++c) __builtin_amdgcn_raw_buffer_store_b128(pk[c], ors, oo + (c & 1) * 64, (c >> 1) * (16 * DIM * 2), OUT_AUX);
    __builtin_amdgcn_s_barrier();
  }
  }
}

}

extern "C" void kernel_launch(int rank, int nranks, ncclComm_t comm,
                              hipStream_t stream,
                              void* const* d_in, const int* in_sizes, int n_in,
                              void* d_out, int out_size,
                              const int* in_dtypes, int out_dtype,
                              void* const* d_out_all,
                              void* const* d_ws_all, size_t ws_bytes) {
  (void)comm; (void)in_sizes; (void)n_in; (void)out_size; (void)in_dtypes; (void)out_dtype; (void)d_out_all;
  if (nranks != NR || (unsigned)rank >= (unsigned)NR || ws_bytes < (size_t)WS_USED) {
    fprintf(stderr, "unsupported configuration\n");
    return;
  }
  Params p;
  p.x = (const unsigned short*)d_in[0];
  p.w = (const unsigned short*)d_in[1];
  p.out = (unsigned short*)d_out;
  for (int i = 0; i < NR; ++i) p.ws[i] = (char*)d_ws_all[i];
  p.myws = p.ws[rank];
  p.rank = rank;
  constexpr OrderTable kOrder = make_order();
  constexpr PrepTable kPrep = make_prep();
  p.order = kOrder;
  p.prep = kPrep;
  constexpr TileTable kTiles = make_tiles();
  p.tiles = kTiles;
  constexpr DirectTable kDirect = make_direct();
  p.direct = kDirect;
  gemm_ar_kernel<<<256, 256, 0, stream>>>(p);
  HIP_CHECK(hipGetLastError());
}
```
